```python
import numpy as np

kernel_name = "gpu_gemm_rs_m4096_k4096_n16384_bf16_q8"


ABSMAX_RTOL = 0.02

COMPUTE_DTYPE = "bfloat16"
EPILOGUE = "quant_int8"
IN_QUANT = None
OUT_QUANT = "int8"
IN_DTYPES = {"x": "bfloat16", "w_mat": "bfloat16"}
WORLD_SIZE = 8

def _to_bf16(x):
    u = np.ascontiguousarray(x, dtype=np.float32).view(np.uint32)
    return ((u + 0x7FFF + ((u >> 16) & 1)) & 0xFFFF0000).view(np.float32)

def setup_inputs():

    x = _to_bf16(np.random.randn(4096, 4096).astype(np.float32))
    w_mat = _to_bf16((np.random.randn(4096, 16384) / np.sqrt(4096)).astype(np.float32))
    return {"x": x, "w_mat": w_mat}

def _quant_dequant(y, amax):
    scale = amax / 127.0
    q = np.clip(np.round(y / scale), -127.0, 127.0)
    return (q * scale).astype(np.float32)
def _epilogue(y):
    return _quant_dequant(y, np.abs(y).max())

def reference(x, w_mat):
    y = (x @ w_mat).astype(np.float32)
    return _epilogue(y)
```

```cpp
#include <hip/hip_runtime.h>
#include <rccl/rccl.h>
#include <cstdio>
#include <cstdint>

typedef unsigned u32;
typedef unsigned long long u64;
typedef u32 u32x4 __attribute__((ext_vector_type(4)));
typedef u32 u32x2 __attribute__((ext_vector_type(2)));
typedef float f32x16 __attribute__((ext_vector_type(16)));
typedef __bf16 bf16x8 __attribute__((ext_vector_type(8)));
typedef __amdgpu_buffer_rsrc_t rsrc_t;
#define LAS __attribute__((address_space(3)))
#define GAS __attribute__((address_space(1)))

constexpr int NR = 8;
constexpr int MM = 4096;
constexpr int KSH = 512;
constexpr int NN = 16384;
constexpr int NL = NN / NR;
constexpr u32 X_BYTES = MM * KSH * 2;
constexpr u32 W_BYTES = KSH * NN * 2;
constexpr u32 OUT_BYTES = (MM / NR) * NN;
constexpr u32 BLK = 4096;
constexpr u32 XST_OFF = 0;
constexpr u32 WST_OFF = XST_OFF + 64u * 128u * BLK;
constexpr u32 FLX_OFF = WST_OFF + 64u * 64u * BLK;
constexpr u32 FLW_OFF = FLX_OFF + 16u * 64u * 32u;
constexpr u32 LMX_OFF = FLW_OFF + 16u * 64u * 16u;
constexpr u32 AMX_OFF = LMX_OFF + 256u * 8u;
constexpr u32 DON_OFF = AMX_OFF + 8u * 128u;
constexpr u32 QUE_OFF = DON_OFF + 256u * 128u;
constexpr u32 CTR_OFF = QUE_OFF + 8u * 128u;
constexpr u32 TMO_OFF = CTR_OFF + 128u;
constexpr u32 NTASK = 768;
#define THR 2
#define OUTW 1
#define OUTT 2
constexpr u32 DBG_OFF = CTR_OFF + 4096u;
constexpr u32 WS_USED = DBG_OFF + 256u * 64u * 8u;
constexpr int AUX_SYS = 17;
#define LOAD_AUX 17
constexpr u64 DEADLINE = 100000000ull;

constexpr u32 SLOT = 12 * BLK;
constexpr u32 NSLOT = 3;
constexpr u32 L_CTL = NSLOT * SLOT;
constexpr u32 L_FULL = L_CTL;
constexpr u32 L_FREE = L_CTL + 16;
constexpr u32 L_OCNT = L_CTL + 32;
constexpr u32 L_WMAX = L_CTL + 36;
constexpr u32 L_WCNT = L_CTL + 40;
constexpr u32 L_GMAX = L_CTL + 44;
constexpr u32 L_ORD = L_CTL + 64;
constexpr u32 L_META = L_ORD + 256;
constexpr u32 LDS_BYTES = L_META + 64 * 32;

struct Params {
    const void* x;
    const void* w;
    char* ws[NR];
    char* out[NR];
    int rank;
    int pad;
};

__device__ __forceinline__ rsrc_t mkr(const void* p, u32 bytes) {
    return __builtin_amdgcn_make_buffer_rsrc((void*)p, 0, bytes, 0x00020000);
}
__device__ __forceinline__ u64 rt() { return __builtin_amdgcn_s_memrealtime(); }
__device__ __forceinline__ u32 rfl(u32 v) { return __builtin_amdgcn_readfirstlane(v); }
#define LGKM0() asm volatile("s_waitcnt lgkmcnt(0)" ::: "memory")
#define VM0() asm volatile("s_waitcnt vmcnt(0)" ::: "memory")
#define VMN(n) asm volatile("s_waitcnt vmcnt(" #n ")" ::: "memory")

__device__ __forceinline__ u32 lds_ld(const LAS char* l, u32 off) {
    return __hip_atomic_load((const LAS u32*)(l + off), __ATOMIC_RELAXED, __HIP_MEMORY_SCOPE_WORKGROUP);
}
__device__ __forceinline__ void lds_st(LAS char* l, u32 off, u32 v) {
    __hip_atomic_store((LAS u32*)(l + off), v, __ATOMIC_RELAXED, __HIP_MEMORY_SCOPE_WORKGROUP);
}
__device__ __forceinline__ void lds_add(LAS char* l, u32 off, u32 v) {
    (void)__hip_atomic_fetch_add((LAS u32*)(l + off), v, __ATOMIC_RELAXED, __HIP_MEMORY_SCOPE_WORKGROUP);
}
__device__ __forceinline__ void lds_wait_ge(const LAS char* l, u32 off, u32 want) {
    while (rfl(lds_ld(l, off)) < want) __builtin_amdgcn_s_sleep(1);
    asm volatile("" ::: "memory");
}

#define STAMP(i) do {} while (0)

__global__ void __launch_bounds__(1024) gemm_ag_q8(Params P) {
    __shared__ __attribute__((aligned(16))) char smem_[LDS_BYTES];
    LAS char* lds = (LAS char*)smem_;
    const int tid = threadIdx.x;
    const int lane = tid & 63;
    const int wave = rfl(tid >> 6);
    const int b = blockIdx.x;
    const int rank = P.rank;
    const int xcd = b & 7, idx = b >> 3;
    const int mt = (xcd >> 1) * 4 + (idx >> 3);
    const int nt = (xcd & 1) * 8 + (idx & 7);
    char* const myws = P.ws[rank];
    GAS u32* const ctr = (GAS u32*)(myws + CTR_OFF);
    GAS u32* const tmo = (GAS u32*)(myws + TMO_OFF);

    if (wave == 0) STAMP(0);
    if (tid < (int)((LDS_BYTES - L_CTL) / 4)) *(LAS u32*)(lds + L_CTL + tid * 4) = 0u;
    const u32 call = rfl(__hip_atomic_load(ctr, __ATOMIC_RELAXED, __HIP_MEMORY_SCOPE_AGENT)) + 1u;

    if (wave == 15) {
        const u32 home = (u32)(idx + xcd) & 7u;
        const u32 k = (u32)xcd * 4u + ((u32)idx >> 3);
        const u32 help = (home + 1u + k % 7u) & 7u;
        const rsrc_t xin = mkr(P.x, X_BYTES), win = mkr(P.w, W_BYTES);
        const u32 c = lane & 31, h = lane >> 5;
        const u32 xv = c * 1024u + h * 16u;
        const u32 pic = 16u * ((c >> 2) & 1u) + 4u * (c >> 3) + (c & 3u);
        const u32 wv = h * 8u * 32768u + pic * 4u;
        const u32 sv = lane * 16u;
        const u32 fv = lane == 0 ? 0u : 0x40000000u;
        u32 R[32];
        u32x4 O[8];
        auto is_w = [&](u32 u) { return (u % 96u) < 32u; };
        auto x_batch = [&](u32 d, u32 u) { const u32 v = u % 96u - 32u; return ((((v >> 2) + (((u32)rank - d) & 7u) * 2u) & 15u) << 2) | (v & 3u); };
        auto issue_loads = [&](u32 d, u32 u) {
            const u32 j = u / 96u;
            if (is_w(u)) {
                const u32 base = (64u * j * 16384u + 2048u * d + 64u * (u % 96u)) * 2u;
#pragma unroll
                for (int i = 0; i < 32; ++i) R[i] = __builtin_amdgcn_raw_buffer_load_b32(win, wv, base + ((i >> 3) * 16u + (i & 7)) * 32768u, 0);
            } else {
                const u32 base = x_batch(d, u) * 65536u + j * 128u;
#pragma unroll
                for (int i = 0; i < 8; ++i) {
                    const u32x4 v = __builtin_amdgcn_raw_buffer_load_b128(xin, xv, base + (i >> 2) * 32768u + (i & 3) * 32u, 0);
                    R[4 * i] = v.x; R[4 * i + 1] = v.y; R[4 * i + 2] = v.z; R[4 * i + 3] = v.w;
                }
            }
        };
        auto transform = [&](u32 u) {
            if (is_w(u)) {
#pragma unroll
                for (int t = 0; t < 4; ++t)
#pragma unroll
                    for (int e = 0; e < 4; ++e) {
                        O[t][e] = __builtin_amdgcn_perm(R[t * 8 + 2 * e + 1], R[t * 8 + 2 * e], 0x05040100u);
                        O[4 + t][e] = __builtin_amdgcn_perm(R[t * 8 + 2 * e + 1], R[t * 8 + 2 * e], 0x07060302u);
                    }
            } else {
#pragma unroll
                for (int i = 0; i < 8; ++i) { O[i].x = R[4 * i]; O[i].y = R[4 * i + 1]; O[i].z = R[4 * i + 2]; O[i].w = R[4 * i + 3]; }
            }
        };
        u32 cd = home, cu = k;
        u32 nd = home, nu = 32u + k;
        bool nvalid = true, helping = false;
        issue_loads(cd, cu);
        LGKM0();
        __builtin_amdgcn_s_barrier();
        STAMP(16);
        transform(cu);
        const u32 qbase = (call - 1u) * NTASK - 64u;
        for (;;) {
            u32 tk = 0;
            if (nvalid) {
                GAS u32* q = (GAS u32*)(myws + QUE_OFF + (helping ? help : home) * 128u);
                if (lane == 0) tk = __hip_atomic_fetch_add(q, 1u, __ATOMIC_RELAXED, __HIP_MEMORY_SCOPE_AGENT);
                issue_loads(nd, nu);
            }
            {
                const rsrc_t dst = mkr(P.ws[cd], WS_USED);
                const u32 j = cu / 96u, st = rank * 8u + j;
                u32 doff, foff;
                if (is_w(cu)) {
                    const u32 wb = cu % 96u;
                    doff = WST_OFF + (st * 64u + 2u * wb) * BLK;
                    foff = FLW_OFF + ((wb >> 1) * 64u + st) * 16u + (wb & 1u) * 8u;
                } else {
                    const u32 xb = x_batch(cd, cu);
                    doff = XST_OFF + (st * 128u + 2u * xb) * BLK;
                    foff = FLX_OFF + ((xb >> 2) * 64u + st) * 32u + (xb & 3u) * 8u;
                }
#pragma unroll
                for (int i = 0; i < 8; ++i) {
                    __builtin_amdgcn_raw_buffer_store_b128(O[i], dst, sv + (i & 3) * 1024, doff + (i >> 2) * BLK, AUX_SYS);
                    if (((i + 1) % THR) == 0) VM0();
                }
                u32x2 f; f.x = 0u; f.y = call;
                __builtin_amdgcn_raw_buffer_store_b64(f, dst, fv, foff, AUX_SYS);
            }
            if (!nvalid) break;
            transform(nu);
            cd = nd; cu = nu;
            u32 t = rfl(tk) - qbase;
            if (t >= NTASK && !helping) {
                helping = true;
                GAS u32* q = (GAS u32*)(myws + QUE_OFF + help * 128u);
                u32 tk2 = 0;
                if (lane == 0) tk2 = __hip_atomic_fetch_add(q, 1u, __ATOMIC_RELAXED, __HIP_MEMORY_SCOPE_AGENT);
                t = rfl(tk2) - qbase;
                STAMP(17);
            }
            nvalid = t < NTASK;
            nd = helping ? help : home; nu = t;
        }
        STAMP(18);
        return;
    }
    LGKM0();
    __builtin_amdgcn_s_barrier();

    if (wave == 14) {
        const rsrc_t ws = mkr(myws, WS_USED);
        const u32 fx = FLX_OFF + (mt * 64u + lane) * 32u, fw = FLW_OFF + (nt * 64u + lane) * 16u;
        u64 pending = ~0ull;
        u32 n = 0;
        const u64 t0 = rt();
        bool first = true;
        while (pending) {
            const bool mine = (pending >> lane) & 1ull;
            u32x4 a = {0, 0, 0, 0}, bb = {0, 0, 0, 0}, cc = {0, 0, 0, 0};
            if (mine) {
                a = __builtin_amdgcn_raw_buffer_load_b128(ws, fx, 0, AUX_SYS);
                bb = __builtin_amdgcn_raw_buffer_load_b128(ws, fx + 16, 0, AUX_SYS);
                cc = __builtin_amdgcn_raw_buffer_load_b128(ws, fw, 0, AUX_SYS);
            }
            const bool ready = mine && a.y >= call && a.w >= call && bb.y >= call && bb.w >= call && cc.y >= call && cc.w >= call;
            u64 mask = __ballot(ready);
            if (!mask) {
                if (rfl(__hip_atomic_load(tmo, __ATOMIC_RELAXED, __HIP_MEMORY_SCOPE_AGENT)) == call || rt() - t0 > DEADLINE) {
                    __hip_atomic_store(tmo, call, __ATOMIC_RELAXED, __HIP_MEMORY_SCOPE_AGENT);
                    mask = pending;
                } else {
                    __builtin_amdgcn_s_sleep(4);
                    continue;
                }
            }
            if ((mask >> lane) & 1ull) {
                const u32 pos = n + __builtin_amdgcn_mbcnt_hi((u32)(mask >> 32), __builtin_amdgcn_mbcnt_lo((u32)mask, 0u));
                *(LAS u32*)(lds + L_ORD + pos * 4) = lane;
            }
            LGKM0();
            n += __builtin_popcountll(mask);
            pending &= ~mask;
            if (lane == 0) lds_st(lds, L_OCNT, n);
            if (first) { STAMP(1); first = false; }
        }
        STAMP(2);
        if (b == 0) {
            const u32 off = LMX_OFF + lane * 32u;
            const u64 t1 = rt();
            u32 gm = 0;
            for (;;) {
                const u32x4 a = __builtin_amdgcn_raw_buffer_load_b128(ws, off, 0, AUX_SYS);
                const u32x4 c = __builtin_amdgcn_raw_buffer_load_b128(ws, off + 16, 0, AUX_SYS);
                if (__all(a.y >= call && a.w >= call && c.y >= call && c.w >= call)) {
                    const u32 m1 = a.x > a.z ? a.x : a.z, m2 = c.x > c.z ? c.x : c.z;
                    gm = m1 > m2 ? m1 : m2;
                    break;
                }
                if (rfl(__hip_atomic_load(tmo, __ATOMIC_RELAXED, __HIP_MEMORY_SCOPE_AGENT)) == call || rt() - t1 > DEADLINE) {
                    __hip_atomic_store(tmo, call, __ATOMIC_RELAXED, __HIP_MEMORY_SCOPE_AGENT);
                    gm = 0x7f800000u;
                    break;
                }
                __builtin_amdgcn_s_sleep(1);
            }
#pragma unroll
            for (int o = 32; o > 0; o >>= 1) { const u32 other = __shfl_xor(gm, o); gm = gm > other ? gm : other; }
            const u64 g = ((u64)call << 32) | gm;
            __hip_atomic_store((GAS u64*)(P.ws[lane >> 3] + AMX_OFF + (lane & 7) * 128 + rank * 8), g, __ATOMIC_RELAXED, __HIP_MEMORY_SCOPE_SYSTEM);
            STAMP(8);
        }
        return;
    }
    if (wave >= 8) {
        const u32 grp = (wave - 8) / 3u, li = (wave - 8) - grp * 3u;
        const rsrc_t ws = mkr(myws, WS_USED);
        const u32 sv = lane * 16u;
        const u32 tile_off = li < 2 ? XST_OFF + (8u * mt + 4u * li) * BLK : WST_OFF + (4u * nt) * BLK;
        const u32 step_shift = li < 2 ? 19u : 18u;
        u32 slot = grp, use = 0;
        for (u32 n = grp; n < 64; n += 2) {
            lds_wait_ge(lds, L_OCNT, n + 1);
            const u32 s_ = rfl(lds_ld(lds, L_ORD + n * 4));
            const u32 so = tile_off + (s_ << step_shift);
            u32x4 B[16];
#pragma unroll
            for (int i = 0; i < 16; ++i) B[i] = __builtin_amdgcn_raw_buffer_load_b128(ws, sv + (i & 3) * 1024, so + (i >> 2) * BLK, LOAD_AUX);
            lds_wait_ge(lds, L_FREE + slot * 4, 8u * use);
            LAS char* sb = lds + slot * SLOT + li * (4 * BLK) + lane * 16;
#pragma unroll
            for (int i = 0; i < 16; ++i) *(LAS u32x4*)(sb + i * 1024) = B[i];
            LGKM0();
            if (lane == 0) lds_add(lds, L_FULL + slot * 4, 1u);
            slot += 2; if (slot >= NSLOT) { slot -= NSLOT; ++use; }
        }
        return;
    }

    const u32 wm = wave >> 1, wn = wave & 1;
    f32x16 acc[2][2];
#pragma unroll
    for (int i = 0; i < 2; ++i)
#pragma unroll
        for (int f = 0; f < 2; ++f)
#pragma unroll
            for (int r = 0; r < 16; ++r) acc[i][f][r] = 0.f;
    {
        u32 slot = 0, use = 1;
        for (u32 n = 0; n < 64; ++n) {
            lds_wait_ge(lds, L_FULL + slot * 4, 3u * use);
            const LAS char* xb = lds + slot * SLOT + (2 * wm) * BLK + lane * 16;
            const LAS char* wb = lds + slot * SLOT + (8 + 2 * wn) * BLK + lane * 16;
            u32x4 xa[2][2], wa[2][2];
#define RD(set, t) do { xa[set][0] = *(const LAS u32x4*)(xb + (t) * 1024); xa[set][1] = *(const LAS u32x4*)(xb + BLK + (t) * 1024); \
                        wa[set][0] = *(const LAS u32x4*)(wb + (t) * 1024); wa[set][1] = *(const LAS u32x4*)(wb + BLK + (t) * 1024); } while (0)
#define MM4(set) _Pragma("unroll") for (int i = 0; i < 2; ++i) _Pragma("unroll") for (int f = 0; f < 2; ++f) \
            acc[i][f] = __builtin_amdgcn_mfma_f32_32x32x16_bf16(__builtin_bit_cast(bf16x8, wa[set][f]), __builtin_bit_cast(bf16x8, xa[set][i]), acc[i][f], 0, 0, 0)
            RD(0, 0); RD(1, 1);
            MM4(0);
            RD(0, 2);
            MM4(1);
            RD(1, 3);
            MM4(0);
            LGKM0();
            if (lane == 0) lds_add(lds, L_FREE + slot * 4, 1u);
            MM4(1);
            if (++slot == NSLOT) { slot = 0; ++use; }
        }
    }
    if (wave == 0) STAMP(3);
    float mx = 0.f;
#pragma unroll
    for (int i = 0; i < 2; ++i)
#pragma unroll
        for (int f = 0; f < 2; ++f)
#pragma unroll
            for (int r = 0; r < 16; ++r) mx = fmaxf(mx, fabsf(acc[i][f][r]));
    u32 mb = __float_as_uint(mx);
#pragma unroll
    for (int o = 32; o > 0; o >>= 1) { const u32 other = __shfl_xor(mb, o); mb = mb > other ? mb : other; }
    u32 arrived = 0;
    if (lane == 0) {
        (void)__hip_atomic_fetch_max((LAS u32*)(lds + L_WMAX), mb, __ATOMIC_RELAXED, __HIP_MEMORY_SCOPE_WORKGROUP);
        arrived = __hip_atomic_fetch_add((LAS u32*)(lds + L_WCNT), 1u, __ATOMIC_RELAXED, __HIP_MEMORY_SCOPE_WORKGROUP);
    }
    arrived = rfl(arrived);
    if (arrived == 7) {
        const u32 wmax = rfl(lds_ld(lds, L_WMAX));
        if (lane == 0)
            __hip_atomic_store((GAS u64*)(myws + LMX_OFF + b * 8), ((u64)call << 32) | wmax, __ATOMIC_RELAXED, __HIP_MEMORY_SCOPE_AGENT);
    }
    if (wave == 0) {
        GAS u64* g = (GAS u64*)(myws + AMX_OFF + xcd * 128 + (lane & 7) * 8);
        const u64 t0 = rt();
        u32 gm = 0;
        for (;;) {
            const u64 v = __hip_atomic_load(g, __ATOMIC_RELAXED, __HIP_MEMORY_SCOPE_SYSTEM);
            if (__all((u32)(v >> 32) >= call)) { gm = (u32)v; break; }
            if (rfl(__hip_atomic_load(tmo, __ATOMIC_RELAXED, __HIP_MEMORY_SCOPE_AGENT)) == call || rt() - t0 > DEADLINE) {
                __hip_atomic_store(tmo, call, __ATOMIC_RELAXED, __HIP_MEMORY_SCOPE_AGENT);
                break;
            }
            __builtin_amdgcn_s_sleep(1);
        }
#pragma unroll
        for (int o = 4; o > 0; o >>= 1) { const u32 other = __shfl_xor(gm, o); gm = gm > other ? gm : other; }
        if (lane == 0) lds_st(lds, L_GMAX, gm);
    }
    LGKM0();
    __builtin_amdgcn_s_barrier();
    if (wave == 0) STAMP(4);
    const float amax = __uint_as_float(rfl(lds_ld(lds, L_GMAX)));
    const bool poison = rfl(__hip_atomic_load(tmo, __ATOMIC_RELAXED, __HIP_MEMORY_SCOPE_AGENT)) == call;
    if (b == 0 && tid == 0) __hip_atomic_store(ctr, call, __ATOMIC_RELAXED, __HIP_MEMORY_SCOPE_AGENT);
    {
        const float scale = amax / 127.0f;
        const float inv = scale > 0.f ? 1.0f / scale : 0.f;
        const u32 hh = lane >> 5;
#pragma unroll
        for (int i = 0; i < 2; ++i) {
            u32 D[8];
#pragma unroll
            for (int q = 0; q < 4; ++q)
#pragma unroll
                for (int u = 0; u < 2; ++u) {
                    u32 v = 0;
#pragma unroll
                    for (int e = 0; e < 4; ++e) {
                        float y = acc[i][e & 1][4 * q + 2 * u + (e >> 1)] * inv;
                        y = __builtin_amdgcn_fmed3f(rintf(y), -127.f, 127.f);
                        v |= ((u32)(int)y & 0xffu) << (8 * e);
                    }
                    D[2 * q + u] = poison ? 0x80808080u : v;
                }
            const u32 row = 64 * wm + 32 * i + (lane & 31);
            const u32 c0 = 4 * wn + 2 * hh;
            u32x4 v0 = {D[0], D[1], D[2], D[3]}, v1 = {D[4], D[5], D[6], D[7]};
            *(LAS u32x4*)(lds + row * 128 + ((c0 ^ (row & 7)) * 16)) = v0;
            *(LAS u32x4*)(lds + row * 128 + (((c0 + 1) ^ (row & 7)) * 16)) = v1;
        }
    }
    LGKM0();
    __builtin_amdgcn_s_barrier();
    const u32 dest = (u32)mt >> 1;
    if (wave < OUTW) {
        const rsrc_t ob = mkr(P.out[dest], OUT_BYTES);
#pragma unroll
        for (int it = 0; it < 32 / OUTW; ++it) {
            const u32 row = 8 * (it * OUTW + wave) + (lane >> 3), ch = lane & 7;
            const u32x4 v = *(const LAS u32x4*)(lds + row * 128 + ((ch ^ (row & 7)) * 16));
            const u32 lrow = (mt & 1) * 256 + row;
            __builtin_amdgcn_raw_buffer_store_b128(v, ob, lrow * NN + rank * NL + nt * 128 + ch * 16, 0, AUX_SYS);
            if (((it + 1) % OUTT) == 0) VM0();
        }
    }
    if (wave == 0) STAMP(5);
    VM0();
    __builtin_amdgcn_s_barrier();
    if (wave == 0) {
        STAMP(6);
        if (lane == 0)
            __hip_atomic_store((GAS u32*)(P.ws[dest] + DON_OFF + (rank * 32 + (mt & 1) * 16 + nt) * 128), call, __ATOMIC_RELAXED, __HIP_MEMORY_SCOPE_SYSTEM);
        GAS u32* f = (GAS u32*)(myws + DON_OFF + b * 128);
        const u64 t0 = rt();
        while (rfl(__hip_atomic_load(f, __ATOMIC_RELAXED, __HIP_MEMORY_SCOPE_SYSTEM)) < call) {
            if (rfl(__hip_atomic_load(tmo, __ATOMIC_RELAXED, __HIP_MEMORY_SCOPE_AGENT)) == call || rt() - t0 > DEADLINE) {
                __hip_atomic_store(tmo, call, __ATOMIC_RELAXED, __HIP_MEMORY_SCOPE_AGENT);
                if (lane == 0) __hip_atomic_store((GAS unsigned char*)(P.out[rank] + rank * NL), (unsigned char)0x80, __ATOMIC_RELAXED, __HIP_MEMORY_SCOPE_SYSTEM);
                break;
            }
            __builtin_amdgcn_s_sleep(2);
        }
        STAMP(7);
    }
}

extern "C" void kernel_launch(int rank, int nranks, ncclComm_t comm, hipStream_t stream,
                              void* const* d_in, const int* in_sizes, int n_in,
                              void* d_out, int out_size, const int* in_dtypes, int out_dtype,
                              void* const* d_out_all, void* const* d_ws_all, size_t ws_bytes) {
    (void)comm; (void)in_dtypes; (void)out_dtype; (void)d_out;
    if (nranks != NR || (unsigned)rank >= (unsigned)NR || n_in != 2 || in_sizes[0] != MM * KSH || in_sizes[1] != KSH * NN ||
        out_size != (int)OUT_BYTES || ws_bytes < WS_USED) {
        fprintf(stderr, "kernel_launch: unexpected shapes\n");
        return;
    }
    Params P;
    P.x = d_in[0];
    P.w = d_in[1];
    for (int p = 0; p < NR; ++p) { P.ws[p] = (char*)d_ws_all[p]; P.out[p] = (char*)d_out_all[p]; }
    P.rank = rank;
    P.pad = 0;
    hipLaunchKernelGGL(gemm_ag_q8, dim3(256), dim3(1024), 0, stream, P);
}
```

```cpp
#include <hip/hip_runtime.h>
#include <rccl/rccl.h>
#include <cstdio>
#include <cstdint>

typedef unsigned u32;
typedef unsigned long long u64;
typedef u32 u32x4 __attribute__((ext_vector_type(4)));
typedef u32 u32x2 __attribute__((ext_vector_type(2)));
typedef float f32x16 __attribute__((ext_vector_type(16)));
typedef __bf16 bf16x8 __attribute__((ext_vector_type(8)));
typedef __amdgpu_buffer_rsrc_t rsrc_t;
#define LAS __attribute__((address_space(3)))
#define GAS __attribute__((address_space(1)))

constexpr int NR = 8;
constexpr int MM = 4096;
constexpr int KSH = 512;
constexpr int NN = 16384;
constexpr int NL = NN / NR;
constexpr u32 X_BYTES = MM * KSH * 2;
constexpr u32 W_BYTES = KSH * NN * 2;
constexpr u32 OUT_BYTES = (MM / NR) * NN;
constexpr u32 BLK = 4096;
constexpr u32 XS_REL = 0;
constexpr u32 WS_REL = 64u * 128u * BLK;
constexpr u32 STG_BYTES = WS_REL + 64u * 64u * BLK;
constexpr u32 PST_OFF = 0;
constexpr u32 RAW_OFF = STG_BYTES;
constexpr u32 NTASK = 768;
constexpr u32 PCK_OFF = 2u * STG_BYTES;
constexpr u32 PCK_W = 512u * 8192u;
constexpr u32 PRD_OFF = PCK_OFF + 2560u * 8192u;
constexpr u32 ARR_OFF = PRD_OFF + 2560u * 64u;
constexpr u32 MT = 13;
constexpr u32 NGEMM = MT * 16;
constexpr u32 NCOMM = 256 - NGEMM;
constexpr u32 RDX_OFF = ARR_OFF + 8u * NTASK * 64u;
constexpr u32 RDW_OFF = RDX_OFF + MT * 64u * 32u;
constexpr u32 LMX_OFF = RDW_OFF + 16u * 64u * 8u;
constexpr u32 AMX_OFF = LMX_OFF + 256u * 8u;
constexpr u32 DON_OFF = AMX_OFF + 8u * 128u;
constexpr u32 QUE_OFF = DON_OFF + 384u * 128u;
constexpr u32 CTR_OFF = QUE_OFF + 8u * 128u;
constexpr u32 TMO_OFF = CTR_OFF + 128u;
constexpr u32 DBG_OFF = CTR_OFF + 4096u;
constexpr u32 DBG2_OFF = DBG_OFF + 256u * 64u * 8u;
constexpr u32 WS_USED = DBG2_OFF + 16u * 64u * 8u;
constexpr int AUX_SYS = 17;
constexpr u64 DEADLINE = 100000000ull;
constexpr u32 OOB = 0x40000000u;

#define OUTW 1
#define OUTT 2
#define NPUSH 2
#define KPUSH 12
#define STAGGER 20
static_assert(7 * KPUSH <= 48 * NPUSH, "pushers");

constexpr u32 SLOT = 14 * 2048;
constexpr u32 NSLOT = 5;
constexpr u32 L_CTL = NSLOT * SLOT;
constexpr u32 L_FULL = L_CTL;
constexpr u32 L_FREE = L_CTL + 32;
constexpr u32 L_OCNT = L_CTL + 64;
constexpr u32 L_WMAX = L_CTL + 68;
constexpr u32 L_WCNT = L_CTL + 72;
constexpr u32 L_GMAX = L_CTL + 76;
constexpr u32 L_ORD = L_CTL + 128;
constexpr u32 LDS_BYTES = L_ORD + 256;

struct Params {
    const void* x;
    const void* w;
    char* ws[NR];
    char* out[NR];
    int rank;
    int pad;
};

__device__ __forceinline__ rsrc_t mkr(const void* p, u32 bytes) {
    return __builtin_amdgcn_make_buffer_rsrc((void*)p, 0, bytes, 0x00020000);
}
__device__ __forceinline__ u64 rt() { return __builtin_amdgcn_s_memrealtime(); }
__device__ __forceinline__ u32 rfl(u32 v) { return __builtin_amdgcn_readfirstlane(v); }
__device__ __forceinline__ void st128(u32x4 v, rsrc_t r, u32 off) { __builtin_amdgcn_raw_buffer_store_b128(v, r, off, 0, 17); }
#define LGKM0() asm volatile("s_waitcnt lgkmcnt(0)" ::: "memory")
#define VM0() asm volatile("s_waitcnt vmcnt(0)" ::: "memory")

__device__ __forceinline__ u32 lds_ld(const LAS char* l, u32 off) {
    return __hip_atomic_load((const LAS u32*)(l + off), __ATOMIC_RELAXED, __HIP_MEMORY_SCOPE_WORKGROUP);
}
__device__ __forceinline__ void lds_st(LAS char* l, u32 off, u32 v) {
    __hip_atomic_store((LAS u32*)(l + off), v, __ATOMIC_RELAXED, __HIP_MEMORY_SCOPE_WORKGROUP);
}
__device__ __forceinline__ void lds_add(LAS char* l, u32 off, u32 v) {
    (void)__hip_atomic_fetch_add((LAS u32*)(l + off), v, __ATOMIC_RELAXED, __HIP_MEMORY_SCOPE_WORKGROUP);
}
__device__ __forceinline__ void lds_wait_ge(const LAS char* l, u32 off, u32 want) {
    while (rfl(lds_ld(l, off)) < want) __builtin_amdgcn_s_sleep(1);
    asm volatile("" ::: "memory");
}
__device__ __forceinline__ bool give_up(GAS u32* tmo, u32 call, u64 t0) {
    if (rfl(__hip_atomic_load(tmo, __ATOMIC_RELAXED, __HIP_MEMORY_SCOPE_AGENT)) == call || rt() - t0 > DEADLINE) {
        __hip_atomic_store(tmo, call, __ATOMIC_RELAXED, __HIP_MEMORY_SCOPE_AGENT);
        return true;
    }
    return false;
}
__device__ __forceinline__ u32 wave_max_u32(u32 v) {
    u32 o;
    o = __builtin_amdgcn_update_dpp(0u, v, 0x111, 0xf, 0xf, false); v = v > o ? v : o;
    o = __builtin_amdgcn_update_dpp(0u, v, 0x112, 0xf, 0xf, false); v = v > o ? v : o;
    o = __builtin_amdgcn_update_dpp(0u, v, 0x114, 0xf, 0xf, false); v = v > o ? v : o;
    o = __builtin_amdgcn_update_dpp(0u, v, 0x118, 0xf, 0xf, false); v = v > o ? v : o;
    o = __builtin_amdgcn_update_dpp(0u, v, 0x142, 0xa, 0xf, false); v = v > o ? v : o;
    o = __builtin_amdgcn_update_dpp(0u, v, 0x143, 0xc, 0xf, false); v = v > o ? v : o;
    return __builtin_amdgcn_readlane(v, 63);
}

#define EXCCAP 8
constexpr u32 EXC_CAP = EXCCAP;
constexpr u32 META_RAW = 0x800u;
constexpr u32 EXC_REL = 6144;
__device__ __forceinline__ u32 pkmax_u16(u32 a, u32 b) {
    typedef unsigned short u16x2 __attribute__((ext_vector_type(2)));
    return __builtin_bit_cast(u32, __builtin_elementwise_max(__builtin_bit_cast(u16x2, a), __builtin_bit_cast(u16x2, b)));
}
__device__ __forceinline__ u32 encode_task(const u32* V, u32x4* O, rsrc_t dst, u32 slot_off, u32 lane) {
    u32 m = V[0] & 0x7fff7fffu;
#pragma unroll
    for (int i = 1; i < 32; ++i) m = pkmax_u16(m, V[i] & 0x7fff7fffu);
    m = pkmax_u16(m, m >> 16) & 0xffffu;
    const u32 top = wave_max_u32(m) >> 8;
    const u32 base = top > 7u ? top - 7u : 0u;
    const u32 b4 = base * 0x01010101u;
    u32 bad = 0;
    u32 Lp[16], X[8];
#pragma unroll
    for (int t = 0; t < 8; ++t) {
        const u32 h01 = __builtin_amdgcn_perm(V[4 * t + 1], V[4 * t], 0x07050301u), h23 = __builtin_amdgcn_perm(V[4 * t + 3], V[4 * t + 2], 0x07050301u);
        Lp[2 * t] = __builtin_amdgcn_perm(V[4 * t + 1], V[4 * t], 0x06040200u);
        Lp[2 * t + 1] = __builtin_amdgcn_perm(V[4 * t + 3], V[4 * t + 2], 0x06040200u);
        const u32 u01 = (h01 | 0x80808080u) - b4, u23 = (h23 | 0x80808080u) - b4;
        bad |= (u01 ^ 0x80808080u) | (u23 ^ 0x80808080u);
        X[t] = (u01 & 0x07070707u) | ((h01 >> 4) & 0x08080808u) | ((u23 & 0x07070707u) << 4) | (h23 & 0x80808080u);
    }
    u32 meta = base;
    if (__any((bad & 0xf8f8f8f8u) != 0u)) {
        u64 bm = 0;
#pragma unroll
        for (int i = 0; i < 64; ++i) {
            const u32 h7 = (V[i >> 1] >> (8 + 16 * (i & 1))) & 0x7fu;
            bm |= (u64)(h7 - base > 7u ? 1u : 0u) << i;
        }
        u32 n = 0;
        for (;;) {
            const u64 act = __ballot(bm != 0ull);
            if (!act) break;
            const u32 cnt = __builtin_popcountll(act);
            if (n + cnt > EXC_CAP) { n = EXC_CAP + 1u; break; }
            const u32 pos = n + __builtin_amdgcn_mbcnt_hi((u32)(act >> 32), __builtin_amdgcn_mbcnt_lo((u32)act, 0u));
            const u32 i = bm ? __builtin_ctzll(bm) : 0u;
            u32 dw = V[0];
#pragma unroll
            for (int q = 1; q < 32; ++q) dw = (i >> 1) == (u32)q ? V[q] : dw;
            const u32 val = (dw >> (16u * (i & 1u))) & 0xffffu;
            const u32 ent = (lane << 22) | (i << 16) | val;
            __builtin_amdgcn_raw_buffer_store_b32(ent, dst, bm ? EXC_REL + pos * 4u : OOB, slot_off, AUX_SYS);
            bm &= bm - 1ull;
            n += cnt;
        }
        meta = n > EXC_CAP ? META_RAW : (base | (n << 7));
    }
    if (meta & META_RAW) {
#pragma unroll
        for (int t = 0; t < 8; ++t) { O[t].x = V[4 * t]; O[t].y = V[4 * t + 1]; O[t].z = V[4 * t + 2]; O[t].w = V[4 * t + 3]; }
    } else {
#pragma unroll
        for (int t = 0; t < 4; ++t) { O[t].x = Lp[4 * t]; O[t].y = Lp[4 * t + 1]; O[t].z = Lp[4 * t + 2]; O[t].w = Lp[4 * t + 3]; }
#pragma unroll
        for (int t = 0; t < 2; ++t) { O[4 + t].x = X[4 * t]; O[4 + t].y = X[4 * t + 1]; O[4 + t].z = X[4 * t + 2]; O[4 + t].w = X[4 * t + 3]; }
    }
    return meta;
}
__device__ __forceinline__ void decode_task(u32x4* B, u32 meta, u32 exc, u32 lane) {
    const u32 b4 = (meta & 0x7fu) * 0x01010101u;
    u32 V[32];
#pragma unroll
    for (int t = 0; t < 8; ++t) {
        const u32 x = B[4 + (t >> 2)][t & 3], l01 = B[t >> 1][2 * (t & 1)], l23 = B[t >> 1][2 * (t & 1) + 1];
        const u32 he = ((x & 0x07070707u) + b4) | ((x << 4) & 0x80808080u);
        const u32 ho = (((x >> 4) & 0x07070707u) + b4) | (x & 0x80808080u);
        V[4 * t + 0] = __builtin_amdgcn_perm(he, l01, 0x05010400u);
        V[4 * t + 1] = __builtin_amdgcn_perm(he, l01, 0x07030602u);
        V[4 * t + 2] = __builtin_amdgcn_perm(ho, l23, 0x05010400u);
        V[4 * t + 3] = __builtin_amdgcn_perm(ho, l23, 0x07030602u);
    }
    const u32 cnt = (meta >> 7) & 15u;
    for (u32 e = 0; e < cnt; ++e) {
        const u32 ent = __builtin_amdgcn_readlane(exc, e);
        const u32 i = (ent >> 16) & 63u, sh = 16u * (i & 1u);
        const bool me = lane == (ent >> 22);
        const u32 keep = ~(0xffffu << sh), put = (ent & 0xffffu) << sh;
#pragma unroll
        for (int q = 0; q < 32; ++q) V[q] = (me && (i >> 1) == (u32)q) ? ((V[q] & keep) | put) : V[q];
    }
#pragma unroll
    for (int t = 0; t < 8; ++t) { B[t].x = V[4 * t]; B[t].y = V[4 * t + 1]; B[t].z = V[4 * t + 2]; B[t].w = V[4 * t + 3]; }
}

__device__ __forceinline__ bool task_is_w(u32 u) { return (u % 96u) < 32u; }
__device__ __forceinline__ u32 task_xb(u32 src, u32 dst, u32 u) { return (u % 96u - 32u + ((src - dst) & 7u) * 8u) & 63u; }
__device__ __forceinline__ u32 task_slot(u32 src, u32 dst, u32 u) {
    const u32 st = src * 8u + u / 96u;
    return task_is_w(u) ? WS_REL + (st * 64u + 2u * (u % 96u)) * BLK : XS_REL + (st * 128u + 2u * task_xb(src, dst, u)) * BLK;
}
__device__ __forceinline__ u32 task_rdy(u32 src, u32 dst, u32 u) {
    const u32 st = src * 8u + u / 96u;
    if (task_is_w(u)) { const u32 wb = u % 96u; return RDW_OFF + ((wb >> 1) * 64u + st) * 8u + (wb & 1u) * 4u; }
    const u32 xb = task_xb(src, dst, u);
    return RDX_OFF + ((xb / 5u) * 64u + st) * 32u + (xb % 5u) * 4u;
}
__device__ __forceinline__ u32 task_pck(u32 src, u32 dst, u32 u) {
    const u32 j = u / 96u;
    return task_is_w(u) ? 512u + (dst * 8u + j) * 32u + u % 96u : j * 64u + task_xb(src, dst, u);
}
__device__ __forceinline__ u32 raw_lane_off(bool is_w, u32 lane) {
    return is_w ? ((lane >> 4) & 1u) * 4096u + (lane >> 5) * 512u + (lane & 15u) * 16u : (lane & 7u) * 512u + (lane >> 3) * 16u;
}
__device__ __forceinline__ u32 raw_vec_off(bool is_w, int v) { return is_w ? (v >> 1) * 1024u + (v & 1) * 256u : (v >> 2) * 4096u + (v & 3) * 128u; }

#define STAMP(i) do {} while (0)
#define STAMP2(row, n) do {} while (0)

__global__ void __launch_bounds__(768) gemm_ag_q8(Params P) {
    __shared__ __attribute__((aligned(16))) char smem_[LDS_BYTES];
    LAS char* lds = (LAS char*)smem_;
    const int tid = threadIdx.x;
    const u32 lane = tid & 63;
    const u32 wave = rfl(tid >> 6);
    const u32 b = blockIdx.x;
    const u32 rank = P.rank;
    const u32 xcd = b & 7, idx = b >> 3;
    const u32 ngx = 26u;
    char* const myws = P.ws[rank];
    GAS u32* const ctr = (GAS u32*)(myws + CTR_OFF);
    GAS u32* const tmo = (GAS u32*)(myws + TMO_OFF);

    if (idx >= ngx) {
        const u32 ci = (idx - 26u) * 8u + xcd;
        if (wave == NPUSH) {
            if (ci != 0) return;
            const u32 call = rfl(__hip_atomic_load(ctr, __ATOMIC_RELAXED, __HIP_MEMORY_SCOPE_AGENT)) + 1u;
            const rsrc_t ws = mkr(myws, WS_USED);
            const u32 off = LMX_OFF + lane * 32u;
            const bool has = lane * 4u < NGEMM;
            const u64 t1 = rt();
            u32 gm = 0;
            for (;;) {
                const u32x4 a = __builtin_amdgcn_raw_buffer_load_b128(ws, off, 0, AUX_SYS);
                const u32x4 c = __builtin_amdgcn_raw_buffer_load_b128(ws, off + 16, 0, AUX_SYS);
                if (__all(!has || (a.y >= call && a.w >= call && c.y >= call && c.w >= call))) {
                    const u32 m1 = a.x > a.z ? a.x : a.z, m2 = c.x > c.z ? c.x : c.z;
                    gm = has ? (m1 > m2 ? m1 : m2) : 0u;
                    break;
                }
                if (give_up(tmo, call, t1)) break;
                __builtin_amdgcn_s_sleep(2);
            }
            gm = wave_max_u32(gm);
            const u64 g = ((u64)call << 32) | gm;
            __hip_atomic_store((GAS u64*)(P.ws[lane >> 3] + AMX_OFF + (lane & 7) * 128 + rank * 8), g, __ATOMIC_RELAXED, __HIP_MEMORY_SCOPE_SYSTEM);
            STAMP(8);
            return;
        }
        if (wave > NPUSH) return;
        const u32 sid = ci * NPUSH + wave;
        if (sid >= 7u * KPUSH) return;
        const u32 d = (rank + 1u + sid % 7u) & 7u, k = sid / 7u;
        const u32 call = rfl(__hip_atomic_load(ctr, __ATOMIC_RELAXED, __HIP_MEMORY_SCOPE_AGENT)) + 1u;
        const rsrc_t ws = mkr(myws, WS_USED), dst = mkr(P.ws[d], WS_USED);
        const u32 sv = lane * 16u;
        const u32 fv = lane == 0 ? 0u : OOB;
        if (wave == 0) STAMP(16);
        GAS u32* const que = (GAS u32*)(myws + QUE_OFF + d * 128u);
        const u32 qbase = (call - 1u) * (NTASK - 2u * KPUSH) - 3u * KPUSH;
        auto prd_of = [&](u32 u) { return (GAS u64*)(myws + PRD_OFF + task_pck(rank, d, u) * 64u); };
        auto settle = [&](u32 u, u32 lo, u32 hi) {
            if (hi >= call) return lo;
            const u64 t0 = rt();
            for (;;) {
                const u64 f = __hip_atomic_load(prd_of(u), __ATOMIC_RELAXED, __HIP_MEMORY_SCOPE_SYSTEM);
                if (rfl((u32)(f >> 32)) >= call || give_up(tmo, call, t0)) return rfl((u32)f);
                __builtin_amdgcn_s_sleep(1);
            }
        };
        auto load_task = [&](u32x4* B, u32 u) {
            const u32 so = PCK_OFF + task_pck(rank, d, u) * 8192u;
#pragma unroll
            for (int i = 0; i < 8; ++i) B[i] = __builtin_amdgcn_raw_buffer_load_b128(ws, sv + (i & 3) * 1024, so + (i >> 2) * 4096, AUX_SYS);
        };
        u32x4 A[8], B[8];
        u32 c = k, n1 = k + KPUSH, n2 = k + 2u * KPUSH;
        u32 metac = settle(c, 0u, 0u);
        load_task(A, c);
        u32 meta1 = settle(n1, 0u, 0u);
#define PH(i) do {} while (0)
#define PUSH(CUR, NXT) do { \
            const u32 slot = PST_OFF + task_slot(rank, d, c); \
            _Pragma("unroll") for (int i = 0; i < 6; ++i) st128(CUR[i], dst, sv + slot + i * 1024u); \
            if (metac & META_RAW) { st128(CUR[6], dst, sv + slot + 6144u); st128(CUR[7], dst, sv + slot + 7168u); } \
            else if (metac & 0x780u) st128(CUR[6], dst, lane < 2u ? sv + slot + 6144u : OOB); \
            PH(0); \
            asm volatile("" ::: "memory"); \
            u64 pk = 0; u32 tk = 1u; \
            const bool v1 = n1 < NTASK, v2 = n2 < NTASK; \
            if (v1) { \
                load_task(NXT, n1); \
                if (v2) { \
                    u64 sv_; \
                    asm volatile("global_load_dwordx2 %0, %3, off sc0 sc1\n\ts_mov_b64 %2, exec\n\ts_mov_b64 exec, 1\n\t" \
                                 "global_atomic_add %1, %4, %1, off sc0\n\ts_mov_b64 exec, %2" \
                                 : "=&v"(pk), "+v"(tk), "=&s"(sv_) : "v"(prd_of(n2)), "v"(que) : "memory"); \
                    PH(2); \
                    asm volatile("s_waitcnt vmcnt(10)" ::: "memory"); \
                } else { \
                    asm volatile("s_waitcnt vmcnt(8)" ::: "memory"); \
                } \
            } else { \
                VM0(); \
            } \
            PH(3); \
            u32x2 f; f.x = metac; f.y = call; \
            __builtin_amdgcn_raw_buffer_store_b64(f, dst, fv, ARR_OFF + (rank * NTASK + c) * 64u, AUX_SYS); \
            asm volatile("s_waitcnt vmcnt(1)" : "+v"(pk), "+v"(tk) :: "memory"); \
            c = n1; metac = meta1; \
            n1 = n2; \
            if (v2) { meta1 = settle(n1, rfl((u32)pk), rfl((u32)(pk >> 32))); n2 = rfl(tk) - qbase; } \
            PH(4); } while (0)
        for (;;) {
            PUSH(A, B);
            if (c >= NTASK) break;
            PUSH(B, A);
            if (c >= NTASK) break;
        }
        if (wave == 0) STAMP(18);
        return;
    }

    const u32 ti4 = (xcd & 1u) * 26u + idx;
    const u32 mt = ti4 >> 2;
    const u32 nt = (xcd >> 1) * 4u + (ti4 & 3u);
    const u32 gi = mt * 16u + nt;
    if (wave == 0) STAMP(0);
    if (tid < (int)((LDS_BYTES - L_CTL) / 4)) *(LAS u32*)(lds + L_CTL + tid * 4) = 0u;
    const u32 call = rfl(__hip_atomic_load(ctr, __ATOMIC_RELAXED, __HIP_MEMORY_SCOPE_AGENT)) + 1u;
    LGKM0();
    __builtin_amdgcn_s_barrier();

    if (wave == 11) {
        const rsrc_t ws = mkr(myws, WS_USED);
        const u32 sv = lane * 16u;
        const u32 fv = lane == 0 ? 0u : OOB;
        for (u32 q = gi; q < 7u * NTASK; q += NGEMM) {
            const u32 u = q / 7u, src = (rank + 1u + q % 7u) & 7u;
            GAS u64* arr = (GAS u64*)(myws + ARR_OFF + (src * NTASK + u) * 64u);
            const u64 t0 = rt();
            u64 f;
            for (;;) {
                f = __hip_atomic_load(arr, __ATOMIC_RELAXED, __HIP_MEMORY_SCOPE_SYSTEM);
                if (rfl((u32)(f >> 32)) >= call) break;
                if (give_up(tmo, call, t0)) break;
                __builtin_amdgcn_s_sleep(2);
            }
            const u32 meta = rfl((u32)f);
            const u32 so = task_slot(src, rank, u);
            u32x4 B[8];
#pragma unroll
            for (int i = 0; i < 6; ++i) B[i] = __builtin_amdgcn_raw_buffer_load_b128(ws, sv + i * 1024, PST_OFF + so, AUX_SYS);
            if (meta & META_RAW) {
                B[6] = __builtin_amdgcn_raw_buffer_load_b128(ws, sv + 6144, PST_OFF + so, AUX_SYS);
                B[7] = __builtin_amdgcn_raw_buffer_load_b128(ws, sv + 7168, PST_OFF + so, AUX_SYS);
            } else {
                u32 exc = 0;
                if (meta & 0x780u) exc = __builtin_amdgcn_raw_buffer_load_b32(ws, lane * 4u + EXC_REL, PST_OFF + so, AUX_SYS);
                decode_task(B, meta, exc, lane);
            }
            const bool isw = task_is_w(u);
            const u32 lo = raw_lane_off(isw, lane) + RAW_OFF + so;
#pragma unroll
            for (int i = 0; i < 8; ++i) st128(B[i], ws, lo + raw_vec_off(isw, i));
            VM0();
            __builtin_amdgcn_raw_buffer_store_b32(call, ws, fv, task_rdy(src, rank, u), AUX_SYS);
        }
        STAMP(9);
        return;
    }
    if (wave == 10) {
        const rsrc_t ws = mkr(myws, WS_USED);
        const u32 fx = RDX_OFF + (mt * 64u + lane) * 32u, fw = RDW_OFF + (nt * 64u + lane) * 8u;
        const bool five = mt < MT - 1;
        u64 pending = ~0ull;
        u32 n = 0;
        const u64 t0 = rt();
        bool first = true;
        while (pending) {
            const bool mine = (pending >> lane) & 1ull;
            u32x4 a = {0, 0, 0, 0};
            u32 a4 = 0;
            u32x2 cc = {0, 0};
            if (mine) {
                a = __builtin_amdgcn_raw_buffer_load_b128(ws, fx, 0, AUX_SYS);
                a4 = __builtin_amdgcn_raw_buffer_load_b32(ws, fx + 16, 0, AUX_SYS);
                cc = __builtin_amdgcn_raw_buffer_load_b64(ws, fw, 0, AUX_SYS);
            }
            const bool ready = mine && a.x >= call && a.y >= call && a.z >= call && a.w >= call && (a4 >= call || !five) && cc.x >= call && cc.y >= call;
            u64 mask = __ballot(ready);
            if (!mask) {
                if (give_up(tmo, call, t0)) mask = pending;
                else { __builtin_amdgcn_s_sleep(2); continue; }
            }
            if ((mask >> lane) & 1ull) {
                const u32 pos = n + __builtin_amdgcn_mbcnt_hi((u32)(mask >> 32), __builtin_amdgcn_mbcnt_lo((u32)mask, 0u));
                *(LAS u32*)(lds + L_ORD + pos * 4) = lane;
            }
            LGKM0();
            n += __builtin_popcountll(mask);
            pending &= ~mask;
            if (lane == 0) lds_st(lds, L_OCNT, n);
            if (first) { STAMP(1); first = false; }
        }
        STAMP(2);
        return;
    }
    if (wave >= 8) {
        const u32 half = wave - 8;
        const char* xsrc = myws + RAW_OFF + XS_REL + (10u * mt) * BLK + half * 2048u + lane * 16u;
        const char* wsrc = myws + RAW_OFF + WS_REL + (4u * nt) * BLK + half * 2048u + lane * 16u;
        const u32 lbase = (u32)(uintptr_t)lds;
        auto rd = [&](u32 off) {
            u32 v;
            asm volatile("ds_read_b32 %0, %1\n\ts_waitcnt lgkmcnt(0)" : "=v"(v) : "v"(lbase + off) : "memory");
            return rfl(v);
        };
        u32 ni = 0, islot = half, iuse = 0;
        u32 np = 0, pslot = half;
        while (np < 64) {
            bool can = ni < 64 && ni - np < 2 && rd(L_OCNT) > ni;
            if (can) can = rd(L_FREE + islot * 4) >= 8u * iuse;
            if (can) {
                if (half == 0) STAMP2(0, ni);
                const u32 s_ = rd(L_ORD + ni * 4) & 63u;
                const char* gx = xsrc + (size_t)s_ * (128u * BLK);
                const char* gw = wsrc + (size_t)s_ * (64u * BLK);
                LAS char* sb = lds + islot * SLOT;
#define DMA2(g, l) do { \
                __builtin_amdgcn_global_load_lds((const GAS u32*)(g), (LAS u32*)(l), 16, 0, AUX_SYS); \
                __builtin_amdgcn_global_load_lds((const GAS u32*)(g), (LAS u32*)(l), 16, 1024, AUX_SYS); } while (0)
#pragma unroll
                for (int i = 0; i < 10; ++i) DMA2(gx + i * BLK, sb + i * 2048);
#pragma unroll
                for (int i = 0; i < 4; ++i) DMA2(gw + i * BLK, sb + (10 + i) * 2048);
                ++ni;
                islot += 2; if (islot >= NSLOT) { islot -= NSLOT; ++iuse; }
            } else if (np < ni) {
                if (ni - np == 2) asm volatile("s_waitcnt vmcnt(28)" ::: "memory"); else VM0();
                if (half == 0) STAMP2(3, np);
                if (lane == 0) asm volatile("ds_add_u32 %0, %1" :: "v"(lbase + L_FULL + pslot * 4), "v"(1u) : "memory");
                ++np;
                pslot += 2; if (pslot >= NSLOT) pslot -= NSLOT;
            } else {
                __builtin_amdgcn_s_sleep(1);
            }
        }
        return;
    }

    {
        const rsrc_t xin = mkr(P.x, X_BYTES), win = mkr(P.w, W_BYTES), ws = mkr(myws, WS_USED);
        const u32 xv = (lane >> 3) * 1024u + (lane & 7u) * 16u;
        const u32 c16 = lane & 15u;
        const u32 wv = (lane >> 5) * 262144u + ((lane & 16u) + 8u * ((c16 >> 2) & 1u) + 4u * (c16 >> 3) + (c16 & 3u)) * 4u;
        const u32 sv = lane * 16u;
        const u32 fv = lane == 0 ? 0u : OOB;
        for (u32 e = wave * NGEMM + gi; e < 2560u; e += 8u * NGEMM) {
            const u32 j = e / 320u, r = e % 320u;
            if (e < 8u * NGEMM) for (u32 i = 0; i < j; ++i) __builtin_amdgcn_s_sleep(STAGGER);
            const bool isw = r < 256u;
            const u32 d = isw ? (r & 7u) : rank, ti = isw ? (r >> 3) : r - 256u;
            u32 V[32];
            if (isw) {
                u32 R[32];
                const u32 base = (64u * j * 16384u + 2048u * d + 64u * ti) * 2u;
#pragma unroll
                for (int i = 0; i < 32; ++i)
                    R[i] = __builtin_amdgcn_raw_buffer_load_b32(win, wv, base + ((i >> 3) * 16u + (i & 7)) * 32768u, 0);
#pragma unroll
                for (int g = 0; g < 4; ++g)
#pragma unroll
                    for (int q = 0; q < 4; ++q) {
                        V[8 * g + q] = __builtin_amdgcn_perm(R[8 * g + 2 * q + 1], R[8 * g + 2 * q], 0x05040100u);
                        V[8 * g + 4 + q] = __builtin_amdgcn_perm(R[8 * g + 2 * q + 1], R[8 * g + 2 * q], 0x07060302u);
                    }
            } else {
                const u32 base = ti * 65536u + j * 128u;
#pragma unroll
                for (int i = 0; i < 8; ++i) {
                    const u32x4 v = __builtin_amdgcn_raw_buffer_load_b128(xin, xv, base + i * 8192u, 0);
                    V[4 * i] = v.x; V[4 * i + 1] = v.y; V[4 * i + 2] = v.z; V[4 * i + 3] = v.w;
                }
            }
            const u32 st = rank * 8u + j;
            if (d == rank) {
                const u32 rel = isw ? WS_REL + (st * 64u + 2u * ti) * BLK : XS_REL + (st * 128u + 2u * ti) * BLK;
                const u32 lo = raw_lane_off(isw, lane) + RAW_OFF + rel;
#pragma unroll
                for (int i = 0; i < 8; ++i) { const u32x4 v = {V[4 * i], V[4 * i + 1], V[4 * i + 2], V[4 * i + 3]}; st128(v, ws, lo + raw_vec_off(isw, i)); }
            }
            u32 meta = 0;
            const u32 pk = isw ? 512u + (d * 8u + j) * 32u + ti : j * 64u + ti;
            const bool pack = !isw || d != rank;
            if (pack) {
                u32x4 O[8];
                meta = encode_task(V, O, ws, PCK_OFF + pk * 8192u, lane);
                const u32 so = sv + PCK_OFF + pk * 8192u;
#pragma unroll
                for (int i = 0; i < 6; ++i) st128(O[i], ws, so + i * 1024u);
                if (meta & META_RAW) { st128(O[6], ws, so + 6144u); st128(O[7], ws, so + 7168u); }
            }
            VM0();
            if (pack) { u32x2 f; f.x = meta; f.y = call; __builtin_amdgcn_raw_buffer_store_b64(f, ws, fv, PRD_OFF + pk * 64u, AUX_SYS); }
            if (d == rank) {
                const u32 rdy = isw ? RDW_OFF + ((ti >> 1) * 64u + st) * 8u + (ti & 1u) * 4u : RDX_OFF + ((ti / 5u) * 64u + st) * 32u + (ti % 5u) * 4u;
                __builtin_amdgcn_raw_buffer_store_b32(call, ws, fv, rdy, AUX_SYS);
            }
        }
        if (wave == 0) STAMP(10);
    }
    const u32 wm = wave >> 2, wn = wave & 3;
    f32x16 acc[5];
#pragma unroll
    for (int i = 0; i < 5; ++i)
#pragma unroll
        for (int r = 0; r < 16; ++r) acc[i][r] = 0.f;
    {
        const LAS char* xb0 = lds + (5 * wm) * 2048 + lane * 16;
        const LAS char* wb0 = lds + (10 + wn) * 2048 + lane * 16;
        u32x4 F0[6], F1[6];
#define RD(F, slot, t) do { F[5] = *(const LAS u32x4*)(wb0 + (slot) * SLOT + (t) * 1024); \
            _Pragma("unroll") for (int i = 0; i < 5; ++i) F[i] = *(const LAS u32x4*)(xb0 + (slot) * SLOT + i * 2048 + (t) * 1024); } while (0)
#define MM(F) _Pragma("unroll") for (int i = 0; i < 5; ++i) \
            acc[i] = __builtin_amdgcn_mfma_f32_32x32x16_bf16(__builtin_bit_cast(bf16x8, F[5]), __builtin_bit_cast(bf16x8, F[i]), acc[i], 0, 0, 0)
        u32 slot = 0, use = 1;
        lds_wait_ge(lds, L_FULL, 1u);
        RD(F0, 0u, 0);
        for (u32 h = 0; h < 127; ++h) {
            if (wave == 0 && !(h & 1)) STAMP2(4, h >> 1);
            RD(F1, slot, 1);
            __builtin_amdgcn_sched_barrier(0);
            MM(F0);
            __builtin_amdgcn_sched_barrier(0);
            LGKM0();
            if (lane == 0) lds_add(lds, L_FREE + slot * 4, 1u);
            if (++slot == NSLOT) { slot = 0; ++use; }
            lds_wait_ge(lds, L_FULL + slot * 4, use);
            RD(F0, slot, 0);
            __builtin_amdgcn_sched_barrier(0);
            MM(F1);
            __builtin_amdgcn_sched_barrier(0);
            if (wave == 0 && (h & 1)) STAMP2(5, h >> 1);
        }
        RD(F1, slot, 1);
        MM(F0);
        MM(F1);
    }
    if (wave == 0) STAMP(3);
    const u32 ngrp = (mt == MT - 1 && wm == 1) ? 3u : 5u;
    float mx = 0.f;
#pragma unroll
    for (int i = 0; i < 5; ++i) {
        float m = 0.f;
#pragma unroll
        for (int r = 0; r < 16; ++r) m = fmaxf(m, fabsf(acc[i][r]));
        mx = (u32)i < ngrp ? fmaxf(mx, m) : mx;
    }
    const u32 mb = wave_max_u32(__float_as_uint(mx));
    u32 arrived = 0;
    if (lane == 0) {
        (void)__hip_atomic_fetch_max((LAS u32*)(lds + L_WMAX), mb, __ATOMIC_RELAXED, __HIP_MEMORY_SCOPE_WORKGROUP);
        arrived = __hip_atomic_fetch_add((LAS u32*)(lds + L_WCNT), 1u, __ATOMIC_RELAXED, __HIP_MEMORY_SCOPE_WORKGROUP);
    }
    arrived = rfl(arrived);
    if (arrived == 7) {
        const u32 wmax = rfl(lds_ld(lds, L_WMAX));
        if (lane == 0)
            __hip_atomic_store((GAS u64*)(myws + LMX_OFF + gi * 8), ((u64)call << 32) | wmax, __ATOMIC_RELAXED, __HIP_MEMORY_SCOPE_AGENT);
    }
    if (wave == 0) {
        GAS u64* g = (GAS u64*)(myws + AMX_OFF + xcd * 128 + (lane & 7) * 8);
        const u64 t0 = rt();
        u32 gm = 0;
        for (;;) {
            const u64 v = __hip_atomic_load(g, __ATOMIC_RELAXED, __HIP_MEMORY_SCOPE_SYSTEM);
            if (__all((u32)(v >> 32) >= call)) { gm = (u32)v; break; }
            if (give_up(tmo, call, t0)) break;
            __builtin_amdgcn_s_sleep(1);
        }
        gm = wave_max_u32(gm);
        if (lane == 0) lds_st(lds, L_GMAX, gm);
    }
    LGKM0();
    __builtin_amdgcn_s_barrier();
    if (wave == 0) STAMP(4);
    const float amax = __uint_as_float(rfl(lds_ld(lds, L_GMAX)));
    const bool poison = rfl(__hip_atomic_load(tmo, __ATOMIC_RELAXED, __HIP_MEMORY_SCOPE_AGENT)) == call;
    if (gi == 0 && tid == 0) __hip_atomic_store(ctr, call, __ATOMIC_RELAXED, __HIP_MEMORY_SCOPE_AGENT);
    {
        const float scale = amax / 127.0f;
        const float inv = scale > 0.f ? 1.0f / scale : 0.f;
        const u32 ch = 2 * wn + (lane >> 5);
#pragma unroll
        for (int i = 0; i < 5; ++i) {
            u32x4 v;
#pragma unroll
            for (int q = 0; q < 4; ++q) {
                u32 d = 0;
#pragma unroll
                for (int e = 0; e < 4; ++e) {
                    float y = acc[i][2 * q + (e >> 1) + 8 * (e & 1)] * inv;
                    y = __builtin_amdgcn_fmed3f(rintf(y), -127.f, 127.f);
                    d |= ((u32)(int)y & 0xffu) << (8 * e);
                }
                v[q] = poison ? 0x80808080u : d;
            }
            const u32 row = 160 * wm + 32 * i + (lane & 31);
            *(LAS u32x4*)(lds + row * 128 + ((ch ^ (row & 7)) * 16)) = v;
        }
    }
    LGKM0();
    __builtin_amdgcn_s_barrier();
    const u32 row0 = 320u * mt;
    const u32 nrow8 = mt == MT - 1 ? 32u : 40u;
    if (wave < OUTW) {
        for (u32 g8 = wave; g8 < nrow8; g8 += OUTW * OUTT) {
#pragma unroll
            for (int it = 0; it < OUTT; ++it) {
                const u32 gq = g8 + it * OUTW;
                if (gq < nrow8) {
                    const u32 g = (gq * 23u + nt) % nrow8;
                    const u32 row = 8 * g + (lane >> 3), ch = lane & 7;
                    const u32x4 v = *(const LAS u32x4*)(lds + row * 128 + ((ch ^ (row & 7)) * 16));
                    const u32 grow = row0 + 8 * g;
                    const rsrc_t ob = mkr(P.out[(grow >> 9) & 7u], OUT_BYTES);
                    st128(v, ob, ((grow & 511u) + (lane >> 3)) * NN + rank * NL + nt * 128 + ch * 16);
                }
            }
            VM0();
        }
    }
    if (wave == 0) STAMP(5);
    VM0();
    __builtin_amdgcn_s_barrier();
    if (wave == 0) {
        STAMP(6);
        const u32 d0 = row0 >> 9, d1 = (row0 + 8u * nrow8 - 1u) >> 9;
        if (lane <= d1 - d0) {
            const u32 d = (d0 + lane) & 7u;
            __hip_atomic_store((GAS u32*)(P.ws[d] + DON_OFF + ((rank * 3u + (mt - (512u * d) / 320u)) * 16u + nt) * 128u), call, __ATOMIC_RELAXED, __HIP_MEMORY_SCOPE_SYSTEM);
        }
        const u32 cnt = (512u * rank + 511u) / 320u - (512u * rank) / 320u + 1u;
        for (u32 e = gi; e < 8u * cnt * 16u; e += NGEMM) {
            GAS u32* f = (GAS u32*)(myws + DON_OFF + (((e / (cnt * 16u)) * 3u + (e / 16u) % cnt) * 16u + e % 16u) * 128u);
            const u64 t0 = rt();
            while (rfl(__hip_atomic_load(f, __ATOMIC_RELAXED, __HIP_MEMORY_SCOPE_SYSTEM)) < call) {
                if (give_up(tmo, call, t0)) {
                    if (lane == 0) __hip_atomic_store((GAS unsigned char*)(P.out[rank] + rank * NL), (unsigned char)0x80, __ATOMIC_RELAXED, __HIP_MEMORY_SCOPE_SYSTEM);
                    break;
                }
                __builtin_amdgcn_s_sleep(2);
            }
        }
        STAMP(7);
    }
}

extern "C" void kernel_launch(int rank, int nranks, ncclComm_t comm, hipStream_t stream,
                              void* const* d_in, const int* in_sizes, int n_in,
                              void* d_out, int out_size, const int* in_dtypes, int out_dtype,
                              void* const* d_out_all, void* const* d_ws_all, size_t ws_bytes) {
    (void)comm; (void)in_dtypes; (void)out_dtype; (void)d_out;
    if (nranks != NR || (unsigned)rank >= (unsigned)NR || n_in != 2 || in_sizes[0] != MM * KSH || in_sizes[1] != KSH * NN ||
        out_size != (int)OUT_BYTES || ws_bytes < WS_USED) {
        fprintf(stderr, "kernel_launch: unexpected shapes\n");
        return;
    }
    Params P;
    P.x = d_in[0];
    P.w = d_in[1];
    for (int p = 0; p < NR; ++p) { P.ws[p] = (char*)d_ws_all[p]; P.out[p] = (char*)d_out_all[p]; }
    P.rank = rank;
    P.pad = 0;
    hipLaunchKernelGGL(gemm_ag_q8, dim3(256), dim3(768), 0, stream, P);
}
```

```cpp
#include <hip/hip_runtime.h>
#include <rccl/rccl.h>
#include <cstdio>
#include <cstdint>

typedef unsigned u32;
typedef unsigned long long u64;
typedef u32 u32x4 __attribute__((ext_vector_type(4)));
typedef u32 u32x2 __attribute__((ext_vector_type(2)));
typedef float f32x16 __attribute__((ext_vector_type(16)));
typedef float f32x2 __attribute__((ext_vector_type(2)));
typedef __bf16 bf16x8 __attribute__((ext_vector_type(8)));
typedef __amdgpu_buffer_rsrc_t rsrc_t;
#define LAS __attribute__((address_space(3)))
#define GAS __attribute__((address_space(1)))

constexpr int NR = 8;
constexpr int MM = 4096;
constexpr int KSH = 512;
constexpr int NN = 16384;
constexpr int NL = NN / NR;
constexpr u32 X_BYTES = MM * KSH * 2;
constexpr u32 W_BYTES = KSH * NN * 2;
constexpr u32 OUT_BYTES = (MM / NR) * NN;
constexpr u32 BLK = 4096;
constexpr u32 XS_REL = 0;
constexpr u32 WS_REL = 64u * 128u * BLK;
constexpr u32 STG_BYTES = WS_REL + 64u * 64u * BLK;
constexpr u32 PST_OFF = 0;
constexpr u32 RAW_OFF = STG_BYTES;
constexpr u32 NTASK = 768;
constexpr u32 PCK_OFF = 2u * STG_BYTES;
constexpr u32 PCK_W = 512u * 8192u;
constexpr u32 PRD_OFF = PCK_OFF + 2560u * 8192u;
constexpr u32 ARR_OFF = PRD_OFF + 2560u * 64u;
constexpr u32 MT = 13;
constexpr u32 NGEMM = MT * 16;
constexpr u32 NCOMM = 256 - NGEMM;
constexpr u32 RDX_OFF = ARR_OFF + 8u * NTASK * 64u;
constexpr u32 RDW_OFF = RDX_OFF + MT * 64u * 32u;
constexpr u32 LMX_OFF = RDW_OFF + 16u * 64u * 8u;
constexpr u32 AMX_OFF = LMX_OFF + 256u * 8u;
constexpr u32 DON_OFF = AMX_OFF + 8u * 128u;
constexpr u32 QUE_OFF = DON_OFF + 384u * 128u;
constexpr u32 CTR_OFF = QUE_OFF + 8u * 128u;
constexpr u32 TMO_OFF = CTR_OFF + 128u;
constexpr u32 DBG_OFF = CTR_OFF + 4096u;
constexpr u32 DBG2_OFF = DBG_OFF + 256u * 64u * 8u;
constexpr u32 WS_USED = DBG2_OFF + 16u * 64u * 8u;
constexpr int AUX_SYS = 17;
constexpr u64 DEADLINE = 100000000ull;
constexpr u32 OOB = 0x40000000u;

#define OUTW 1
#define OUTT 2
#define DECWG 16
#define ULATE 704
#define LDEPTH 2
#define OUT_GOAL 1950u
#define OUT_MIN 2u
#define OUT_MAX 4u
#define OUT_RTT0 95u
#define TIGHT 1
#define PB 4
#define PB2 6
#define TFAST 26u
#define NPUSH 6
#define KPUSH 26
#define NRAW 48
#define STAGGER 40
static_assert(7 * KPUSH <= (48 - DECWG) * NPUSH && NPUSH < 8, "pushers");
static_assert(NRAW <= 64, "the unpacked tail consists of x tasks");

constexpr u32 SLOT = 14 * 2048;
constexpr u32 NSLOT = 5;
constexpr u32 L_CTL = NSLOT * SLOT;
constexpr u32 L_FULL = L_CTL;
constexpr u32 L_FREE = L_CTL + 32;
constexpr u32 L_OCNT = L_CTL + 64;
constexpr u32 L_WMAX = L_CTL + 68;
constexpr u32 L_WCNT = L_CTL + 72;
constexpr u32 L_GMAX = L_CTL + 76;
constexpr u32 L_BAD = L_CTL + 80;
constexpr u32 L_ORD = L_CTL + 128;
constexpr u32 L_SCR = L_ORD + 256;
constexpr u32 LDS_BYTES = L_SCR + 4 * 4352;

struct Params {
    const void* x;
    const void* w;
    char* ws[NR];
    char* out[NR];
    int rank;
    int pad;
};

__device__ __forceinline__ rsrc_t mkr(const void* p, u32 bytes) {
    return __builtin_amdgcn_make_buffer_rsrc((void*)p, 0, bytes, 0x00020000);
}
__device__ __forceinline__ u64 rt() { return __builtin_amdgcn_s_memrealtime(); }
__device__ __forceinline__ u32 rfl(u32 v) { return __builtin_amdgcn_readfirstlane(v); }
__device__ __forceinline__ void st128(u32x4 v, rsrc_t r, u32 off) { __builtin_amdgcn_raw_buffer_store_b128(v, r, off, 0, 17); }
#define LGKM0() asm volatile("s_waitcnt lgkmcnt(0)" ::: "memory")
#define VM0() asm volatile("s_waitcnt vmcnt(0)" ::: "memory")

__device__ __forceinline__ u32 lds_ld(const LAS char* l, u32 off) {
    return __hip_atomic_load((const LAS u32*)(l + off), __ATOMIC_RELAXED, __HIP_MEMORY_SCOPE_WORKGROUP);
}
__device__ __forceinline__ void lds_st(LAS char* l, u32 off, u32 v) {
    __hip_atomic_store((LAS u32*)(l + off), v, __ATOMIC_RELAXED, __HIP_MEMORY_SCOPE_WORKGROUP);
}
__device__ __forceinline__ void lds_add(LAS char* l, u32 off, u32 v) {
    (void)__hip_atomic_fetch_add((LAS u32*)(l + off), v, __ATOMIC_RELAXED, __HIP_MEMORY_SCOPE_WORKGROUP);
}
__device__ __forceinline__ void lds_wait_ge(const LAS char* l, u32 off, u32 want) {
    while (rfl(lds_ld(l, off)) < want) __builtin_amdgcn_s_sleep(1);
    asm volatile("" ::: "memory");
}
__device__ __forceinline__ bool give_up(GAS u32* tmo, u32 call, u64 t0) {
    if (rfl(__hip_atomic_load(tmo, __ATOMIC_RELAXED, __HIP_MEMORY_SCOPE_AGENT)) == call || rt() - t0 > DEADLINE) {
        __hip_atomic_store(tmo, call, __ATOMIC_RELAXED, __HIP_MEMORY_SCOPE_AGENT);
        return true;
    }
    return false;
}
__device__ __forceinline__ u32 wave_max_u32(u32 v) {
    u32 o;
    o = __builtin_amdgcn_update_dpp(0u, v, 0x111, 0xf, 0xf, false); v = v > o ? v : o;
    o = __builtin_amdgcn_update_dpp(0u, v, 0x112, 0xf, 0xf, false); v = v > o ? v : o;
    o = __builtin_amdgcn_update_dpp(0u, v, 0x114, 0xf, 0xf, false); v = v > o ? v : o;
    o = __builtin_amdgcn_update_dpp(0u, v, 0x118, 0xf, 0xf, false); v = v > o ? v : o;
    o = __builtin_amdgcn_update_dpp(0u, v, 0x142, 0xa, 0xf, false); v = v > o ? v : o;
    o = __builtin_amdgcn_update_dpp(0u, v, 0x143, 0xc, 0xf, false); v = v > o ? v : o;
    return __builtin_amdgcn_readlane(v, 63);
}

#define EXCCAP 8
constexpr u32 EXC_CAP = EXCCAP;
constexpr u32 META_RAW = 0x800u;
constexpr u32 EXC_REL = 6144;
__device__ __forceinline__ u32 pkmax_u16(u32 a, u32 b) {
    typedef unsigned short u16x2 __attribute__((ext_vector_type(2)));
    return __builtin_bit_cast(u32, __builtin_elementwise_max(__builtin_bit_cast(u16x2, a), __builtin_bit_cast(u16x2, b)));
}
__device__ __forceinline__ u32 encode_task(const u32* V, u32x4* O, rsrc_t dst, u32 slot_off, u32 lane) {
    u32 m = V[0] & 0x7fff7fffu;
#pragma unroll
    for (int i = 1; i < 32; ++i) m = pkmax_u16(m, V[i] & 0x7fff7fffu);
    m = pkmax_u16(m, m >> 16) & 0xffffu;
    const u32 top = wave_max_u32(m) >> 8;
    const u32 base = top > 7u ? top - 7u : 0u;
    const u32 b4 = base * 0x01010101u;
    u32 bad = 0;
    u32 Lp[16], X[8];
#pragma unroll
    for (int t = 0; t < 8; ++t) {
        const u32 h01 = __builtin_amdgcn_perm(V[4 * t + 1], V[4 * t], 0x07050301u), h23 = __builtin_amdgcn_perm(V[4 * t + 3], V[4 * t + 2], 0x07050301u);
        Lp[2 * t] = __builtin_amdgcn_perm(V[4 * t + 1], V[4 * t], 0x06040200u);
        Lp[2 * t + 1] = __builtin_amdgcn_perm(V[4 * t + 3], V[4 * t + 2], 0x06040200u);
        const u32 u01 = (h01 | 0x80808080u) - b4, u23 = (h23 | 0x80808080u) - b4;
        bad |= (u01 ^ 0x80808080u) | (u23 ^ 0x80808080u);
        X[t] = (u01 & 0x07070707u) | ((h01 >> 4) & 0x08080808u) | ((u23 & 0x07070707u) << 4) | (h23 & 0x80808080u);
    }
    u32 meta = base;
    if (__any((bad & 0xf8f8f8f8u) != 0u)) {
        u64 bm = 0;
#pragma unroll
        for (int i = 0; i < 64; ++i) {
            const u32 h7 = (V[i >> 1] >> (8 + 16 * (i & 1))) & 0x7fu;
            bm |= (u64)(h7 - base > 7u ? 1u : 0u) << i;
        }
        u32 n = 0;
        for (;;) {
            const u64 act = __ballot(bm != 0ull);
            if (!act) break;
            const u32 cnt = __builtin_popcountll(act);
            if (n + cnt > EXC_CAP) { n = EXC_CAP + 1u; break; }
            const u32 pos = n + __builtin_amdgcn_mbcnt_hi((u32)(act >> 32), __builtin_amdgcn_mbcnt_lo((u32)act, 0u));
            const u32 i = bm ? __builtin_ctzll(bm) : 0u;
            u32 dw = V[0];
#pragma unroll
            for (int q = 1; q < 32; ++q) dw = (i >> 1) == (u32)q ? V[q] : dw;
            const u32 val = (dw >> (16u * (i & 1u))) & 0xffffu;
            const u32 ent = (lane << 22) | (i << 16) | val;
            __builtin_amdgcn_raw_buffer_store_b32(ent, dst, bm ? EXC_REL + pos * 4u : OOB, slot_off, AUX_SYS);
            bm &= bm - 1ull;
            n += cnt;
        }
        meta = n > EXC_CAP ? META_RAW : (base | (n << 7));
    }
    if (meta & META_RAW) {
#pragma unroll
        for (int t = 0; t < 8; ++t) { O[t].x = V[4 * t]; O[t].y = V[4 * t + 1]; O[t].z = V[4 * t + 2]; O[t].w = V[4 * t + 3]; }
    } else {
#pragma unroll
        for (int t = 0; t < 4; ++t) { O[t].x = Lp[4 * t]; O[t].y = Lp[4 * t + 1]; O[t].z = Lp[4 * t + 2]; O[t].w = Lp[4 * t + 3]; }
#pragma unroll
        for (int t = 0; t < 2; ++t) { O[4 + t].x = X[4 * t]; O[4 + t].y = X[4 * t + 1]; O[4 + t].z = X[4 * t + 2]; O[4 + t].w = X[4 * t + 3]; }
    }
    return meta;
}
__device__ __forceinline__ void decode_task(u32x4* B, u32 meta, u32 exc, u32 lane) {
    const u32 b4 = (meta & 0x7fu) * 0x01010101u;
    u32 V[32];
#pragma unroll
    for (int t = 0; t < 8; ++t) {
        const u32 x = B[4 + (t >> 2)][t & 3], l01 = B[t >> 1][2 * (t & 1)], l23 = B[t >> 1][2 * (t & 1) + 1];
        const u32 he = ((x & 0x07070707u) + b4) | ((x << 4) & 0x80808080u);
        const u32 ho = (((x >> 4) & 0x07070707u) + b4) | (x & 0x80808080u);
        V[4 * t + 0] = __builtin_amdgcn_perm(he, l01, 0x05010400u);
        V[4 * t + 1] = __builtin_amdgcn_perm(he, l01, 0x07030602u);
        V[4 * t + 2] = __builtin_amdgcn_perm(ho, l23, 0x05010400u);
        V[4 * t + 3] = __builtin_amdgcn_perm(ho, l23, 0x07030602u);
    }
    const u32 cnt = (meta >> 7) & 15u;
    for (u32 e = 0; e < cnt; ++e) {
        const u32 ent = __builtin_amdgcn_readlane(exc, e);
        const u32 i = (ent >> 16) & 63u, sh = 16u * (i & 1u);
        const bool me = lane == (ent >> 22);
        const u32 keep = ~(0xffffu << sh), put = (ent & 0xffffu) << sh;
#pragma unroll
        for (int q = 0; q < 32; ++q) V[q] = (me && (i >> 1) == (u32)q) ? ((V[q] & keep) | put) : V[q];
    }
#pragma unroll
    for (int t = 0; t < 8; ++t) { B[t].x = V[4 * t]; B[t].y = V[4 * t + 1]; B[t].z = V[4 * t + 2]; B[t].w = V[4 * t + 3]; }
}

constexpr u32 META_TIGHT = 0x10000u;
constexpr u32 SGN_REL = 5120, LST_REL = 5632;
constexpr u32 SCR_BYTES = 4352;
__device__ __forceinline__ u32 wave_scan_incl(u32 v) {
    v += __builtin_amdgcn_update_dpp(0u, v, 0x111, 0xf, 0xf, false);
    v += __builtin_amdgcn_update_dpp(0u, v, 0x112, 0xf, 0xf, false);
    v += __builtin_amdgcn_update_dpp(0u, v, 0x114, 0xf, 0xf, false);
    v += __builtin_amdgcn_update_dpp(0u, v, 0x118, 0xf, 0xf, false);
    v += __builtin_amdgcn_update_dpp(0u, v, 0x142, 0xa, 0xf, false);
    v += __builtin_amdgcn_update_dpp(0u, v, 0x143, 0xc, 0xf, false);
    return v;
}
__device__ __forceinline__ u32 encode_tight(const u32* V, u32x4* O, u32x2& S, u32& list, LAS char* sc, u32 lane) {
    u32 m = V[0] & 0x7fff7fffu;
#pragma unroll
    for (int i = 1; i < 32; ++i) m = pkmax_u16(m, V[i] & 0x7fff7fffu);
    m = pkmax_u16(m, m >> 16) & 0xffffu;
    const u32 top = wave_max_u32(m) >> 8;
    const u32 base = top > 3u ? top - 3u : 0u;
    const u32 b4 = base * 0x01010101u;
    u32 U[16], C[4] = {0, 0, 0, 0}, Sg[2] = {0, 0}, E[2] = {0, 0};
#pragma unroll
    for (int k = 0; k < 16; ++k) {
        const int t = k >> 1, o = 2 * (k & 1);
        const u32 h = __builtin_amdgcn_perm(V[4 * t + o + 1], V[4 * t + o], 0x07050301u);
        O[k >> 2][k & 3] = __builtin_amdgcn_perm(V[4 * t + o + 1], V[4 * t + o], 0x06040200u);
        const u32 u = (h | 0x80808080u) - b4;
        const u32 e = (u ^ 0x80808080u) & 0xfcfcfcfcu;
        const u32 nz = (((e & 0x7c7c7c7cu) + 0x7c7c7c7cu) | e) & 0x80808080u;
        U[k] = u;
        C[k >> 2] |= (u & 0x03030303u) << (2 * (k & 3));
        Sg[k >> 3] |= (h & 0x80808080u) >> (7 - (k & 7));
        E[k >> 3] |= nz >> (7 - (k & 7));
    }
    const u32 n = __builtin_popcount(E[0]) + __builtin_popcount(E[1]);
    const u32 incl = wave_scan_incl(n);
    const u32 total = __builtin_amdgcn_readlane(incl, 63);
    if (total > 128u) return 0u;
    list = 0;
    if (total) {
#pragma unroll
        for (int j = 0; j < 4; ++j) { const u32x4 q = {U[4 * j], U[4 * j + 1], U[4 * j + 2], U[4 * j + 3]}; *(LAS u32x4*)(sc + j * 1024 + lane * 16) = q; }
        u32 pos = incl - n, e0 = E[0], e1 = E[1];
        bool far = false;
        while (__any((e0 | e1) != 0u)) {
            const bool has = (e0 | e1) != 0u;
            const u32 w = e0 ? 0u : 1u, mk = e0 ? e0 : e1;
            const u32 p = has ? __builtin_ctz(mk) : 0u;
            const u32 k = 8u * w + (p & 7u);
            const u32 ub = *(const LAS unsigned char*)(sc + (k >> 2) * 1024u + lane * 16u + (k & 3u) * 4u + (p >> 3));
            const u32 code = (0x7fu - ub) >> 2;
            if (has) {
                far |= code > 15u;
                *(LAS unsigned short*)(sc + 4096u + pos * 2u) = (unsigned short)((lane << 10) | ((w * 32u + p) << 4) | (code & 15u));
                ++pos;
            }
            if (e0) e0 &= e0 - 1u; else e1 &= e1 - 1u;
        }
        if (__any(far)) return 0u;
        list = *(const LAS u32*)(sc + 4096u + lane * 4u);
    }
    O[4].x = C[0]; O[4].y = C[1]; O[4].z = C[2]; O[4].w = C[3];
    S.x = Sg[0]; S.y = Sg[1];
    return base | META_TIGHT | (total << 20);
}
__device__ __forceinline__ void decode_tight(u32x4* B, u32x2 S, u32 list, u32 meta, LAS char* sc, u32 lane) {
    const u32 b4 = (meta & 0x7fu) * 0x01010101u;
    u32 H[16];
#pragma unroll
    for (int k = 0; k < 16; ++k)
        H[k] = (((B[4][k >> 2] >> (2 * (k & 3))) & 0x03030303u) + b4) | ((S[k >> 3] << (7 - (k & 7))) & 0x80808080u);
    const u32 total = (meta >> 20) & 0xffu;
    if (total) {
#pragma unroll
        for (int j = 0; j < 4; ++j) { const u32x4 q = {H[4 * j], H[4 * j + 1], H[4 * j + 2], H[4 * j + 3]}; *(LAS u32x4*)(sc + j * 1024 + lane * 16) = q; }
#pragma unroll
        for (int e = 0; e < 2; ++e) {
            const u32 ent = e ? list >> 16 : list & 0xffffu;
            const u32 p = (ent >> 4) & 31u, k = ((ent >> 9) & 1u) * 8u + (p & 7u);
            if (2u * lane + e < total)
                (void)__hip_atomic_fetch_sub((LAS u32*)(sc + (k >> 2) * 1024u + (ent >> 10) * 16u + (k & 3u) * 4u), (4u * ((ent & 15u) + 1u)) << (8u * (p >> 3)),
                                             __ATOMIC_RELAXED, __HIP_MEMORY_SCOPE_WORKGROUP);
        }
        asm volatile("" ::: "memory");
#pragma unroll
        for (int j = 0; j < 4; ++j) { const u32x4 q = *(const LAS u32x4*)(sc + j * 1024 + lane * 16); H[4 * j] = q.x; H[4 * j + 1] = q.y; H[4 * j + 2] = q.z; H[4 * j + 3] = q.w; }
    }
    u32 V[32];
#pragma unroll
    for (int t = 0; t < 8; ++t) {
        const u32 l01 = B[t >> 1][2 * (t & 1)], l23 = B[t >> 1][2 * (t & 1) + 1];
        V[4 * t + 0] = __builtin_amdgcn_perm(H[2 * t], l01, 0x05010400u);
        V[4 * t + 1] = __builtin_amdgcn_perm(H[2 * t], l01, 0x07030602u);
        V[4 * t + 2] = __builtin_amdgcn_perm(H[2 * t + 1], l23, 0x05010400u);
        V[4 * t + 3] = __builtin_amdgcn_perm(H[2 * t + 1], l23, 0x07030602u);
    }
#pragma unroll
    for (int t = 0; t < 8; ++t) { B[t].x = V[4 * t]; B[t].y = V[4 * t + 1]; B[t].z = V[4 * t + 2]; B[t].w = V[4 * t + 3]; }
}

__device__ __forceinline__ bool task_is_w(u32 u) { return (u % 96u) < 32u; }
__device__ __forceinline__ u32 task_xb(u32 src, u32 dst, u32 u) { return (u % 96u - 32u + ((src - dst) & 7u) * 8u) & 63u; }
__device__ __forceinline__ u32 task_slot(u32 src, u32 dst, u32 u) {
    const u32 st = src * 8u + u / 96u;
    return task_is_w(u) ? WS_REL + (st * 64u + 2u * (u % 96u)) * BLK : XS_REL + (st * 128u + 2u * task_xb(src, dst, u)) * BLK;
}
__device__ __forceinline__ u32 task_rdy(u32 src, u32 dst, u32 u) {
    const u32 st = src * 8u + u / 96u;
    if (task_is_w(u)) { const u32 wb = u % 96u; return RDW_OFF + ((wb >> 1) * 64u + st) * 8u + (wb & 1u) * 4u; }
    const u32 xb = task_xb(src, dst, u);
    return RDX_OFF + ((xb / 5u) * 64u + st) * 32u + (xb % 5u) * 4u;
}
__device__ __forceinline__ u32 task_pck(u32 src, u32 dst, u32 u) {
    const u32 j = u / 96u;
    return task_is_w(u) ? 512u + (dst * 8u + j) * 32u + u % 96u : j * 64u + task_xb(src, dst, u);
}
__device__ __forceinline__ u32 raw_lane_off(bool is_w, u32 lane) {
    return is_w ? ((lane >> 4) & 1u) * 4096u + (lane >> 5) * 512u + (lane & 15u) * 16u : (lane & 7u) * 512u + (lane >> 3) * 16u;
}
__device__ __forceinline__ u32 raw_vec_off(bool is_w, int v) { return is_w ? (v >> 1) * 1024u + (v & 1) * 256u : (v >> 2) * 4096u + (v & 3) * 128u; }

constexpr u32 NDECWG = DECWG;
constexpr u32 NDEC = NGEMM + NDECWG * 8u, NDECL = NDECWG * 8u;
__device__ __forceinline__ void decoder(char* myws, u32 rank, u32 call, u32 di, u32 lane, LAS char* sc) {
    const rsrc_t ws = mkr(myws, WS_USED);
    GAS u32* const tmo = (GAS u32*)(myws + TMO_OFF);
    const u32 sv = lane * 16u;
    const u32 fv = lane == 0 ? 0u : OOB;
    u32 owed = 0;
    constexpr u32 QL = 7u * ULATE, QE = 7u * (NTASK - NRAW);
    for (u32 q = di; q < QE;) {
        u32 qn = q + (q < QL ? NDEC : NDECL);
        if (q < QL && qn >= QL) qn = di >= NGEMM ? QL + (di - NGEMM) : QE;
        const u32 u = q / 7u, src = (rank + 1u + q % 7u) & 7u;
        GAS u64* arr = (GAS u64*)(myws + ARR_OFF + (src * NTASK + u) * 64u);
        const u64 t0 = rt();
        u64 f;
        for (u32 spins = 0;; ++spins) {
            f = __hip_atomic_load(arr, __ATOMIC_RELAXED, __HIP_MEMORY_SCOPE_SYSTEM);
            const u32 seen = rfl((u32)(f >> 32));
            if (owed) { VM0(); __builtin_amdgcn_raw_buffer_store_b32(call, ws, fv, owed, AUX_SYS); owed = 0; }
            if (seen >= call) break;
            if ((spins & 15u) == 15u && give_up(tmo, call, t0)) break;
            __builtin_amdgcn_s_sleep(1);
        }
        const u32 meta = rfl((u32)f);
        const u32 so = task_slot(src, rank, u);
        u32x4 B[8];
#pragma unroll
        for (int i = 0; i < 5; ++i) B[i] = __builtin_amdgcn_raw_buffer_load_b128(ws, sv + i * 1024, PST_OFF + so, AUX_SYS);
        if (meta & META_TIGHT) {
            const u32x2 S = __builtin_amdgcn_raw_buffer_load_b64(ws, lane * 8u + SGN_REL, PST_OFF + so, AUX_SYS);
            const u32 list = __builtin_amdgcn_raw_buffer_load_b32(ws, lane * 4u + LST_REL, PST_OFF + so, AUX_SYS);
            decode_tight(B, S, list, meta, sc, lane);
        } else {
            B[5] = __builtin_amdgcn_raw_buffer_load_b128(ws, sv + 5120, PST_OFF + so, AUX_SYS);
            if (meta & META_RAW) {
                B[6] = __builtin_amdgcn_raw_buffer_load_b128(ws, sv + 6144, PST_OFF + so, AUX_SYS);
                B[7] = __builtin_amdgcn_raw_buffer_load_b128(ws, sv + 7168, PST_OFF + so, AUX_SYS);
            } else {
                u32 exc = 0;
                if (meta & 0x780u) exc = __builtin_amdgcn_raw_buffer_load_b32(ws, lane * 4u + EXC_REL, PST_OFF + so, AUX_SYS);
                decode_task(B, meta, exc, lane);
            }
        }
        const bool isw = task_is_w(u);
        const u32 lo = raw_lane_off(isw, lane) + RAW_OFF + so;
#pragma unroll
        for (int i = 0; i < 8; ++i) st128(B[i], ws, lo + raw_vec_off(isw, i));
        owed = task_rdy(src, rank, u);
        if (qn >= QE || u >= NTASK - 96u) {
            VM0(); __builtin_amdgcn_raw_buffer_store_b32(call, ws, fv, owed, AUX_SYS); owed = 0;
        }
        q = qn;
    }
    if (owed) { VM0(); __builtin_amdgcn_raw_buffer_store_b32(call, ws, fv, owed, AUX_SYS); }
}

#define STAMP(i) do {} while (0)
#define STAMP2(row, n) do {} while (0)

__global__ void __launch_bounds__(512) gemm_ag_q8(Params P) {
    __shared__ __attribute__((aligned(16))) char smem_[LDS_BYTES];
    LAS char* lds = (LAS char*)smem_;
    const int tid = threadIdx.x;
    const u32 lane = tid & 63;
    const u32 wave = rfl(tid >> 6);
    const u32 b = blockIdx.x;
    const u32 rank = P.rank;
    const u32 xcd = b & 7, idx = b >> 3;
    const u32 ngx = 26u;
    char* const myws = P.ws[rank];
    GAS u32* const ctr = (GAS u32*)(myws + CTR_OFF);
    GAS u32* const tmo = (GAS u32*)(myws + TMO_OFF);

    if (idx >= ngx) {
        const u32 ci = (idx - 26u) * 8u + xcd;
        if (ci >= NCOMM - NDECWG) {
            const u32 call = rfl(__hip_atomic_load(ctr, __ATOMIC_RELAXED, __HIP_MEMORY_SCOPE_AGENT)) + 1u;
            decoder(myws, rank, call, NGEMM + (ci - (NCOMM - NDECWG)) * 8u + wave, lane, lds + wave * SCR_BYTES);
            STAMP(40 + wave);
            return;
        }
        if (wave == NPUSH) {
            if (ci != 0) return;
            const u32 call = rfl(__hip_atomic_load(ctr, __ATOMIC_RELAXED, __HIP_MEMORY_SCOPE_AGENT)) + 1u;
            const rsrc_t ws = mkr(myws, WS_USED);
            const u32 off = LMX_OFF + lane * 32u;
            const bool has = lane * 4u < NGEMM;
            const u64 t1 = rt();
            u32 gm = 0;
            for (u32 spins = 1;; ++spins) {
                const u32x4 a = __builtin_amdgcn_raw_buffer_load_b128(ws, off, 0, AUX_SYS);
                const u32x4 c = __builtin_amdgcn_raw_buffer_load_b128(ws, off + 16, 0, AUX_SYS);
                if (__all(!has || (a.y >= call && a.w >= call && c.y >= call && c.w >= call))) {
                    const u32 m1 = a.x > a.z ? a.x : a.z, m2 = c.x > c.z ? c.x : c.z;
                    gm = has ? (m1 > m2 ? m1 : m2) : 0u;
                    break;
                }
                if ((spins & 15u) == 0u && give_up(tmo, call, t1)) break;
                __builtin_amdgcn_s_sleep(1);
            }
            gm = wave_max_u32(gm);
            const u64 g = ((u64)call << 32) | gm;
            __hip_atomic_store((GAS u64*)(P.ws[lane >> 3] + AMX_OFF + (lane & 7) * 128 + rank * 8), g, __ATOMIC_RELAXED, __HIP_MEMORY_SCOPE_SYSTEM);
            STAMP(8);
            return;
        }
        if (wave > NPUSH) return;
        const u32 sid = ci * NPUSH + wave;
        if (sid >= 7u * KPUSH) return;
        const u32 d = (rank + 1u + sid % 7u) & 7u, k = sid / 7u;
        const u32 call = rfl(__hip_atomic_load(ctr, __ATOMIC_RELAXED, __HIP_MEMORY_SCOPE_AGENT)) + 1u;
        const rsrc_t ws = mkr(myws, WS_USED), dst = mkr(P.ws[d], WS_USED);
        const u32 sv = lane * 16u;
        const u32 fv = lane == 0 ? 0u : OOB;
        if (wave == 0) STAMP(16);
        GAS u32* const que = (GAS u32*)(myws + QUE_OFF + d * 128u);
        const u32 qbase = (call - 1u) * (NTASK - 4u * KPUSH) - 5u * KPUSH;
        auto prd_of = [&](u32 u) { return (GAS u64*)(myws + PRD_OFF + task_pck(rank, d, u) * 64u); };
        auto settle = [&](u32 u, u32 lo, u32 hi) {
            if (hi >= call) return lo;
            const u64 t0 = rt();
            for (u32 spins = 1;; ++spins) {
                const u64 f = __hip_atomic_load(prd_of(u), __ATOMIC_RELAXED, __HIP_MEMORY_SCOPE_SYSTEM);
                if (rfl((u32)(f >> 32)) >= call || ((spins & 15u) == 0u && give_up(tmo, call, t0))) return rfl((u32)f);
                __builtin_amdgcn_s_sleep(1);
            }
        };
        auto load_task = [&](u32x4* B, u32 u) {
            const u32 so = u >= NTASK - NRAW ? RAW_OFF + task_slot(rank, d, u) : PCK_OFF + task_pck(rank, d, u) * 8192u;
#pragma unroll
            for (int i = 0; i < 8; ++i) B[i] = __builtin_amdgcn_raw_buffer_load_b128(ws, sv + (i & 3) * 1024, so + (i >> 2) * 4096, AUX_SYS);
        };
        u32x4 A[8], B[8];
        static_assert(3 * KPUSH <= 96, "the tasks packed by the pushers lie in k slice 0");
        u32 c, n1, metac;
        {
            const rsrc_t xin = mkr(P.x, X_BYTES), win = mkr(P.w, W_BYTES);
            const u32 xv = (lane >> 3) * 1024u + (lane & 7u) * 16u;
            const u32 c16 = lane & 15u;
            const u32 wv = (lane >> 5) * 262144u + ((lane & 16u) + 8u * ((c16 >> 2) & 1u) + 4u * (c16 >> 3) + (c16 & 3u)) * 4u;
            u32 R[32], V[32];
            auto fetch = [&](u32 u) {
                if (u < 32u) {
                    const u32 base = (2048u * d + 64u * u) * 2u;
#pragma unroll
                    for (int i = 0; i < 32; ++i) R[i] = __builtin_amdgcn_raw_buffer_load_b32(win, wv, base + ((i >> 3) * 16u + (i & 7)) * 32768u, 0);
                } else {
                    const u32 base = task_xb(rank, d, u) * 65536u;
#pragma unroll
                    for (int i = 0; i < 8; ++i) {
                        const u32x4 v = __builtin_amdgcn_raw_buffer_load_b128(xin, xv, base + i * 8192u, 0);
                        R[4 * i] = v.x; R[4 * i + 1] = v.y; R[4 * i + 2] = v.z; R[4 * i + 3] = v.w;
                    }
                }
            };
            auto arrange = [&](u32 u) {
                if (u < 32u) {
#pragma unroll
                    for (int g = 0; g < 4; ++g)
#pragma unroll
                        for (int q = 0; q < 4; ++q) {
                            V[8 * g + q] = __builtin_amdgcn_perm(R[8 * g + 2 * q + 1], R[8 * g + 2 * q], 0x05040100u);
                            V[8 * g + 4 + q] = __builtin_amdgcn_perm(R[8 * g + 2 * q + 1], R[8 * g + 2 * q], 0x07060302u);
                        }
                } else {
#pragma unroll
                    for (int i = 0; i < 32; ++i) V[i] = R[i];
                }
            };
            auto send = [&](u32x4* O, u32 u, u32 meta) {
                const u32 slot = PST_OFF + task_slot(rank, d, u);
#pragma unroll
                for (int i = 0; i < 6; ++i) st128(O[i], dst, sv + slot + i * 1024u);
                if (meta & META_RAW) { st128(O[6], dst, sv + slot + 6144u); st128(O[7], dst, sv + slot + 7168u); }
            };
            auto flag = [&](u32 u, u32 meta) {
                VM0();
                u32x2 f; f.x = meta; f.y = call;
                __builtin_amdgcn_raw_buffer_store_b64(f, dst, fv, ARR_OFF + (rank * NTASK + u) * 64u, AUX_SYS);
            };
            const u32 u0 = k, u1 = k + KPUSH, u2 = k + 2u * KPUSH;
            fetch(u0);
            arrange(u0); fetch(u1);
            const u32 m0 = encode_task(V, A, dst, PST_OFF + task_slot(rank, d, u0), lane);
            send(A, u0, m0);
            if (wave == 0) STAMP(17);
            arrange(u1); fetch(u2);
            const u32 m1 = encode_task(V, B, dst, PST_OFF + task_slot(rank, d, u1), lane);
            flag(u0, m0);
            send(B, u1, m1);
            arrange(u2);
            const u32 m2 = encode_task(V, A, dst, PST_OFF + task_slot(rank, d, u2), lane);
            flag(u1, m1);
            send(A, u2, m2);
            c = k + 3u * KPUSH; n1 = k + 4u * KPUSH;
            metac = settle(c, 0u, 0u);
            load_task(B, c);
            flag(u2, m2);
            asm volatile("s_waitcnt vmcnt(0)" : "+v"(B[0]), "+v"(B[1]), "+v"(B[2]), "+v"(B[3]), "+v"(B[4]), "+v"(B[5]), "+v"(B[6]), "+v"(B[7]) :: "memory");
        }

#define PH(i) do {} while (0)
        u32 fa = OOB, fb = OOB, oa = 0, ob = 0;
        u32x2 fo; fo.x = 0; fo.y = call;
#define ST1(CUR, i) do { \
            if ((i) < 5) st128(CUR[i], dst, so + (i) * 1024u); \
            else if ((i) == 5) st128(CUR[5], dst, (!full && (metac & META_TIGHT) && lane >= 48u) ? OOB : so + 5120u); \
            else if ((i) == 6) st128(CUR[6], dst, (full || ((metac & 0x780u) && lane < 2u)) ? so + 6144u : OOB); \
            else st128(CUR[7], dst, full ? so + 7168u : OOB); } while (0)
#define STR2(x) #x
#define STR(x) STR2(x)
#define PUSH(CUR, NXT) do { \
            const bool tail = c >= NTASK - NRAW; \
            const bool full = tail || (metac & META_RAW); \
            const u32 so = sv + (tail ? RAW_OFF : PST_OFF) + task_slot(rank, d, c); \
            const bool v1 = n1 < NTASK; \
            GAS u64* const pa = prd_of(v1 ? n1 : 0u); \
            u32 tk = 1u, meta1 = 0; \
            u64 pk; \
            asm volatile("global_load_dwordx2 %0, %1, off sc0 sc1" : "=v"(pk) : "v"(pa) : "memory"); \
            __builtin_amdgcn_raw_buffer_store_b64(fo, dst, fa, oa, AUX_SYS); \
            __builtin_amdgcn_raw_buffer_store_b32(call, dst, fb, ob, AUX_SYS); \
            _Pragma("unroll") for (int i = 0; i < PB; ++i) ST1(CUR, i); \
            asm volatile("s_waitcnt vmcnt(" STR(PB2) ")" : "+v"(pk) :: "memory"); \
            if (v1) { \
                meta1 = settle(n1, rfl((u32)pk), rfl((u32)(pk >> 32))); \
                asm volatile("" ::: "memory"); \
                load_task(NXT, n1); \
                if (PB < 8 && n1 < NTASK - TFAST) asm volatile("s_waitcnt vmcnt(8)" ::: "memory");        \
                _Pragma("unroll") for (int i = PB; i < 8; ++i) ST1(CUR, i); \
                u64 sv_; \
                asm volatile("s_mov_b64 %1, exec\n\ts_mov_b64 exec, 1\n\tglobal_atomic_add %0, %2, %0, off sc0\n\ts_mov_b64 exec, %1\n\t" \
                             "s_waitcnt vmcnt(0)" : "+v"(tk), "=&s"(sv_) : "v"(que) : "memory"); \
                asm volatile("" : "+v"(NXT[0]), "+v"(NXT[1]), "+v"(NXT[2]), "+v"(NXT[3]), "+v"(NXT[4]), "+v"(NXT[5]), "+v"(NXT[6]), \
                             "+v"(NXT[7]) :: "memory");            \
            } else { \
                _Pragma("unroll") for (int i = PB; i < 8; ++i) ST1(CUR, i); \
                VM0(); \
            } \
            fo.x = metac; \
            fa = tail ? OOB : fv; fb = tail ? fv : OOB; \
            oa = ARR_OFF + (rank * NTASK + c) * 64u; ob = task_rdy(rank, d, c); \
            if (wave == 0 && c < 96u && n1 >= 96u) STAMP(19); \
            if (wave == 0 && c < 384u && n1 >= 384u) STAMP(20); \
            if (wave == 0 && c < 672u && n1 >= 672u) STAMP(21); \
            c = n1; metac = meta1; \
            if (v1) n1 = rfl(tk) - qbase; \
            } while (0)
        for (;;) {
            PUSH(B, A);
            if (c >= NTASK) break;
            PUSH(A, B);
            if (c >= NTASK) break;
        }
        __builtin_amdgcn_raw_buffer_store_b64(fo, dst, fa, oa, AUX_SYS);
        __builtin_amdgcn_raw_buffer_store_b32(call, dst, fb, ob, AUX_SYS);
        VM0();
        STAMP(32 + wave);
        return;
    }

    const u32 ti4 = (xcd & 1u) * 26u + idx;
    const u32 mt = ti4 >> 2;
    const u32 nt = (xcd >> 1) * 4u + (ti4 & 3u);
    const u32 gi = mt * 16u + nt;
    if (wave == 0) STAMP(0);
    if (tid < (int)((L_SCR - L_CTL) / 4)) *(LAS u32*)(lds + L_CTL + tid * 4) = 0u;
    const u32 call = rfl(__hip_atomic_load(ctr, __ATOMIC_RELAXED, __HIP_MEMORY_SCOPE_AGENT)) + 1u;
    LGKM0();
    __builtin_amdgcn_s_barrier();

    if (wave == 7) {
        decoder(myws, rank, call, gi, lane, lds + L_SCR + 3 * SCR_BYTES);
        STAMP(9);
        return;
    }
    if (wave == 6) {
        const rsrc_t ws = mkr(myws, WS_USED);
        const u32 fx = RDX_OFF + (mt * 64u + lane) * 32u, fw = RDW_OFF + (nt * 64u + lane) * 8u;
        const bool five = mt < MT - 1;
        u64 pending = ~0ull;
        u32 n = 0;
        const u64 t0 = rt();
        bool first = true;
        u32 spins = 0;
        while (pending) {
            const bool mine = (pending >> lane) & 1ull;
            u32x4 a = {0, 0, 0, 0};
            u32 a4 = 0;
            u32x2 cc = {0, 0};
            if (mine) {
                a = __builtin_amdgcn_raw_buffer_load_b128(ws, fx, 0, AUX_SYS);
                a4 = __builtin_amdgcn_raw_buffer_load_b32(ws, fx + 16, 0, AUX_SYS);
                cc = __builtin_amdgcn_raw_buffer_load_b64(ws, fw, 0, AUX_SYS);
            }
            const bool ready = mine && a.x >= call && a.y >= call && a.z >= call && a.w >= call && (a4 >= call || !five) && cc.x >= call && cc.y >= call;
            u64 mask = __ballot(ready);
            if (!mask) {
                if ((++spins & 15u) == 0u && give_up(tmo, call, t0)) mask = pending;
                else { __builtin_amdgcn_s_sleep(1); continue; }
            }
            if ((mask >> lane) & 1ull) {
                const u32 pos = n + __builtin_amdgcn_mbcnt_hi((u32)(mask >> 32), __builtin_amdgcn_mbcnt_lo((u32)mask, 0u));
                *(LAS u32*)(lds + L_ORD + pos * 4) = lane;
            }
            LGKM0();
            n += __builtin_popcountll(mask);
            pending &= ~mask;
            if (lane == 0) lds_st(lds, L_OCNT, n);
            if (first) { STAMP(1); first = false; }
        }
        STAMP(2);
        return;
    }
    if (wave >= 4) {
        const u32 half = wave - 4;
        const char* xsrc = myws + RAW_OFF + XS_REL + (10u * mt) * BLK + half * 2048u + lane * 16u;
        const char* wsrc = myws + RAW_OFF + WS_REL + (4u * nt) * BLK + half * 2048u + lane * 16u;
        const u32 lbase = (u32)(uintptr_t)lds;
        auto rd = [&](u32 off) {
            u32 v;
            asm volatile("ds_read_b32 %0, %1\n\ts_waitcnt lgkmcnt(0)" : "=v"(v) : "v"(lbase + off) : "memory");
            return rfl(v);
        };
        u32 ni = 0, islot = half, iuse = 0;
        u32 np = 0, pslot = half;
        while (np < 64) {
            bool can = ni < 64 && ni - np < LDEPTH && rd(L_OCNT) > ni;
            if (can) can = rd(L_FREE + islot * 4) >= 4u * iuse;
            if (can) {
                if (half == 0) STAMP2(0, ni);
                const u32 s_ = rd(L_ORD + ni * 4) & 63u;
                const char* gx = xsrc + (size_t)s_ * (128u * BLK);
                const char* gw = wsrc + (size_t)s_ * (64u * BLK);
                LAS char* sb = lds + islot * SLOT;
#define DMA2(g, l) do { \
                __builtin_amdgcn_global_load_lds((const GAS u32*)(g), (LAS u32*)(l), 16, 0, AUX_SYS); \
                __builtin_amdgcn_global_load_lds((const GAS u32*)(g), (LAS u32*)(l), 16, 1024, AUX_SYS); } while (0)
#pragma unroll
                for (int i = 0; i < 10; ++i) DMA2(gx + i * BLK, sb + i * 2048);
#pragma unroll
                for (int i = 0; i < 4; ++i) DMA2(gw + i * BLK, sb + (10 + i) * 2048);
                ++ni;
                islot += 2; if (islot >= NSLOT) { islot -= NSLOT; ++iuse; }
            } else if (np < ni) {
                if (ni - np == 2) asm volatile("s_waitcnt vmcnt(28)" ::: "memory"); else VM0();
                if (half == 0) STAMP2(3, np);
                if (lane == 0) asm volatile("ds_add_u32 %0, %1" :: "v"(lbase + L_FULL + pslot * 4), "v"(1u) : "memory");
                ++np;
                pslot += 2; if (pslot >= NSLOT) pslot -= NSLOT;
            } else {
                __builtin_amdgcn_s_sleep(1);
            }
        }
        return;
    }

    {
        const rsrc_t xin = mkr(P.x, X_BYTES), win = mkr(P.w, W_BYTES), ws = mkr(myws, WS_USED);
        const u32 xv = (lane >> 3) * 1024u + (lane & 7u) * 16u;
        const u32 c16 = lane & 15u;
        const u32 wv = (lane >> 5) * 262144u + ((lane & 16u) + 8u * ((c16 >> 2) & 1u) + 4u * (c16 >> 3) + (c16 & 3u)) * 4u;
        const u32 sv = lane * 16u;
        const u32 fv = lane == 0 ? 0u : OOB;
        for (u32 e = wave * NGEMM + gi; wave < 3u && e < 2560u; e += 3u * NGEMM) {
            const u32 j = e / 320u, r = e % 320u;
            if (e < 3u * NGEMM) for (u32 i = 0; i < j; ++i) __builtin_amdgcn_s_sleep(STAGGER);
            const bool isw = r < 256u;
            const u32 d = isw ? (r & 7u) : rank, ti = isw ? (r >> 3) : r - 256u;
            if (isw && j == 0 && ti < 3u * KPUSH && d != rank) continue;
            u32 V[32];
            if (isw) {
                u32 R[32];
                const u32 base = (64u * j * 16384u + 2048u * d + 64u * ti) * 2u;
#pragma unroll
                for (int i = 0; i < 32; ++i)
                    R[i] = __builtin_amdgcn_raw_buffer_load_b32(win, wv, base + ((i >> 3) * 16u + (i & 7)) * 32768u, 0);
#pragma unroll
                for (int g = 0; g < 4; ++g)
#pragma unroll
                    for (int q = 0; q < 4; ++q) {
                        V[8 * g + q] = __builtin_amdgcn_perm(R[8 * g + 2 * q + 1], R[8 * g + 2 * q], 0x05040100u);
                        V[8 * g + 4 + q] = __builtin_amdgcn_perm(R[8 * g + 2 * q + 1], R[8 * g + 2 * q], 0x07060302u);
                    }
            } else {
                const u32 base = ti * 65536u + j * 128u;
#pragma unroll
                for (int i = 0; i < 8; ++i) {
                    const u32x4 v = __builtin_amdgcn_raw_buffer_load_b128(xin, xv, base + i * 8192u, 0);
                    V[4 * i] = v.x; V[4 * i + 1] = v.y; V[4 * i + 2] = v.z; V[4 * i + 3] = v.w;
                }
            }
            const u32 st = rank * 8u + j;
            if (d == rank) {
                const u32 rel = isw ? WS_REL + (st * 64u + 2u * ti) * BLK : XS_REL + (st * 128u + 2u * ti) * BLK;
                const u32 lo = raw_lane_off(isw, lane) + RAW_OFF + rel;
#pragma unroll
                for (int i = 0; i < 8; ++i) { const u32x4 v = {V[4 * i], V[4 * i + 1], V[4 * i + 2], V[4 * i + 3]}; st128(v, ws, lo + raw_vec_off(isw, i)); }
            }
            u32 meta = 0;
            const u32 pk = isw ? 512u + (d * 8u + j) * 32u + ti : j * 64u + ti;
            const bool pack = !isw || d != rank;
            if (pack) {
                u32x4 O[8];
                const u32 slot = PCK_OFF + pk * 8192u, so = sv + slot;
                u32x2 S; u32 list;
                meta = TIGHT ? encode_tight(V, O, S, list, lds + L_SCR + wave * SCR_BYTES, lane) : 0u;
                if (meta) {
#pragma unroll
                    for (int i = 0; i < 5; ++i) st128(O[i], ws, so + i * 1024u);
                    __builtin_amdgcn_raw_buffer_store_b64(S, ws, lane * 8u + SGN_REL, slot, AUX_SYS);
                    __builtin_amdgcn_raw_buffer_store_b32(list, ws, lane * 4u + LST_REL, slot, AUX_SYS);
                } else {
                    meta = encode_task(V, O, ws, slot, lane);
#pragma unroll
                    for (int i = 0; i < 6; ++i) st128(O[i], ws, so + i * 1024u);
                    if (meta & META_RAW) { st128(O[6], ws, so + 6144u); st128(O[7], ws, so + 7168u); }
                }
            }
            VM0();
            if (pack) { u32x2 f; f.x = meta; f.y = call; __builtin_amdgcn_raw_buffer_store_b64(f, ws, fv, PRD_OFF + pk * 64u, AUX_SYS); }
            if (d == rank) {
                const u32 rdy = isw ? RDW_OFF + ((ti >> 1) * 64u + st) * 8u + (ti & 1u) * 4u : RDX_OFF + ((ti / 5u) * 64u + st) * 32u + (ti % 5u) * 4u;
                __builtin_amdgcn_raw_buffer_store_b32(call, ws, fv, rdy, AUX_SYS);
            }
        }
        if (wave == 0) STAMP(10);
    }
    const u32 wm = wave >> 1, wn = wave & 1;
    f32x16 acc[2][5];
#pragma unroll
    for (int f = 0; f < 2; ++f)
#pragma unroll
        for (int i = 0; i < 5; ++i)
#pragma unroll
            for (int r = 0; r < 16; ++r) acc[f][i][r] = 0.f;
    {
        const LAS char* xb0 = lds + (5 * wm) * 2048 + lane * 16;
        const LAS char* wb0 = lds + (10 + 2 * wn) * 2048 + lane * 16;
        u32x4 F0[7], F1[7];
#define RD(F, slot, t) do { F[5] = *(const LAS u32x4*)(wb0 + (slot) * SLOT + (t) * 1024); \
            F[0] = *(const LAS u32x4*)(xb0 + (slot) * SLOT + (t) * 1024); \
            F[6] = *(const LAS u32x4*)(wb0 + (slot) * SLOT + 2048 + (t) * 1024); \
            _Pragma("unroll") for (int i = 1; i < 5; ++i) F[i] = *(const LAS u32x4*)(xb0 + (slot) * SLOT + i * 2048 + (t) * 1024); } while (0)
#define MM(F) _Pragma("unroll") for (int i = 0; i < 5; ++i) _Pragma("unroll") for (int f = 0; f < 2; ++f) \
            acc[f][i] = __builtin_amdgcn_mfma_f32_32x32x16_bf16(__builtin_bit_cast(bf16x8, F[5 + f]), __builtin_bit_cast(bf16x8, F[i]), acc[f][i], 0, 0, 0)
#define SB() __builtin_amdgcn_sched_barrier(0)
#define MF(F, i, f) acc[f][i] = __builtin_amdgcn_mfma_f32_32x32x16_bf16(__builtin_bit_cast(bf16x8, F[5 + f]), __builtin_bit_cast(bf16x8, F[i]), acc[f][i], 0, 0, 0)
#define RX(F, i, sl, t) F[i] = *(const LAS u32x4*)(xb0 + (sl) + (i) * 2048 + (t) * 1024)
#define RW(F, f, sl, t) F[5 + f] = *(const LAS u32x4*)(wb0 + (sl) + (f) * 2048 + (t) * 1024)
        const u32 lbase = (u32)(uintptr_t)lds;
        u32 slot = 0, use = 1;
        lds_wait_ge(lds, L_FULL, 1u);
        SB();
        RW(F0, 0, 0u, 0); SB(); RX(F0, 0, 0u, 0); SB(); RW(F0, 1, 0u, 0); SB(); RX(F0, 1, 0u, 0); SB();
        RX(F0, 2, 0u, 0); SB(); RX(F0, 3, 0u, 0); SB(); RX(F0, 4, 0u, 0); SB();
        for (u32 h = 0; h < 127; ++h) {
            const u32 cur = slot, co = cur * SLOT;
            if (++slot == NSLOT) { slot = 0; ++use; }
            const u32 no = slot * SLOT;
            SB();
            MF(F0, 0, 0); const u32 fl = lds_ld(lds, L_FULL + slot * 4); SB();
            MF(F0, 0, 1); RW(F1, 0, co, 1); SB();
            MF(F0, 1, 0); RX(F1, 0, co, 1); SB();
            MF(F0, 1, 1); RW(F1, 1, co, 1); SB();
            MF(F0, 2, 0); RX(F1, 1, co, 1); SB();
            MF(F0, 2, 1); RX(F1, 2, co, 1); SB();
            MF(F0, 3, 0); RX(F1, 3, co, 1); SB();
            MF(F0, 3, 1); RX(F1, 4, co, 1); SB();
            MF(F0, 4, 0); SB();
            MF(F0, 4, 1); SB();
            MF(F1, 0, 0); SB();
            MF(F1, 0, 1); SB();
            LGKM0();
            {
                u64 sv_;
                asm volatile("s_mov_b64 %0, exec\n\ts_mov_b64 exec, 1\n\tds_add_u32 %1, %2\n\ts_mov_b64 exec, %0"
                             : "=&s"(sv_) : "v"(lbase + L_FREE + cur * 4), "v"(1u) : "memory");
            }
            if (rfl(fl) < use) lds_wait_ge(lds, L_FULL + slot * 4, use);
            asm volatile("" ::: "memory");
            SB();
            MF(F1, 1, 0); RW(F0, 0, no, 0); SB();
            MF(F1, 1, 1); RX(F0, 0, no, 0); SB();
            MF(F1, 2, 0); RW(F0, 1, no, 0); SB();
            MF(F1, 2, 1); RX(F0, 1, no, 0); SB();
            MF(F1, 3, 0); RX(F0, 2, no, 0); SB();
            MF(F1, 3, 1); RX(F0, 3, no, 0); SB();
            MF(F1, 4, 0); RX(F0, 4, no, 0); SB();
            MF(F1, 4, 1); SB();
        }
        const u32 slot_last = slot * SLOT;
        RW(F1, 0, slot_last, 1); RW(F1, 1, slot_last, 1);
#pragma unroll
        for (int i = 0; i < 5; ++i) RX(F1, i, slot_last, 1);
        MM(F0);
        MM(F1);
    }
    if (wave == 0) STAMP(3);
    const u32 ngrp = (mt == MT - 1 && wm == 1) ? 3u : 5u;
    float mx = 0.f;
#pragma unroll
    for (int i = 0; i < 5; ++i) {
        float m = 0.f;
#pragma unroll
        for (int f = 0; f < 2; ++f)
#pragma unroll
            for (int r = 0; r < 16; ++r) m = fmaxf(m, fabsf(acc[f][i][r]));
        mx = (u32)i < ngrp ? fmaxf(mx, m) : mx;
    }
    const u32 mb = wave_max_u32(__float_as_uint(mx));
    u32 arrived = 0;
    if (lane == 0) {
        (void)__hip_atomic_fetch_max((LAS u32*)(lds + L_WMAX), mb, __ATOMIC_RELAXED, __HIP_MEMORY_SCOPE_WORKGROUP);
        arrived = __hip_atomic_fetch_add((LAS u32*)(lds + L_WCNT), 1u, __ATOMIC_RELAXED, __HIP_MEMORY_SCOPE_WORKGROUP);
    }
    arrived = rfl(arrived);
    if (arrived == 3) {
        const u32 wmax = rfl(lds_ld(lds, L_WMAX));
        if (lane == 0)
            __hip_atomic_store((GAS u64*)(myws + LMX_OFF + gi * 8), ((u64)call << 32) | wmax, __ATOMIC_RELAXED, __HIP_MEMORY_SCOPE_AGENT);
    }
    if (wave == 0) {
        GAS u64* g = (GAS u64*)(myws + AMX_OFF + xcd * 128 + (lane & 7) * 8);
        const u64 t0 = rt();
        u32 gm = 0, bad = 0;
        for (u32 spins = 1;; ++spins) {
            const u64 v = __hip_atomic_load(g, __ATOMIC_RELAXED, __HIP_MEMORY_SCOPE_SYSTEM);
            const u32 tm = __hip_atomic_load(tmo, __ATOMIC_RELAXED, __HIP_MEMORY_SCOPE_AGENT);
            bad = rfl(tm) == call;
            if (bad || __all((u32)(v >> 32) >= call)) { gm = (u32)v; break; }
            if ((spins & 15u) == 0u && give_up(tmo, call, t0)) { bad = 1; break; }
            __builtin_amdgcn_s_sleep(1);
        }
        gm = wave_max_u32(gm);
        if (lane == 0) { lds_st(lds, L_GMAX, gm); lds_st(lds, L_BAD, bad); }
    }
    LGKM0();
    __builtin_amdgcn_s_barrier();
    if (wave == 0) STAMP(4);
    const float amax = __uint_as_float(rfl(lds_ld(lds, L_GMAX)));
    const bool poison = rfl(lds_ld(lds, L_BAD)) != 0u;
    if (gi == 0 && tid == 0) __hip_atomic_store(ctr, call, __ATOMIC_RELAXED, __HIP_MEMORY_SCOPE_AGENT);
    {
        const float scale = amax / 127.0f;
        const float inv = scale > 0.f ? 1.0f / scale : 0.f;
        const f32x2 inv2 = {inv, inv}, magic2 = {12582912.f + 128.f, 12582912.f + 128.f};
#pragma unroll
        for (int f = 0; f < 2; ++f) {
            const u32 ch = 4 * wn + 2 * f + (lane >> 5);
#pragma unroll
            for (int i = 0; i < 5; ++i) {
                u32x4 v;
#pragma unroll
                for (int q = 0; q < 4; ++q) {
                    constexpr float MAGIC = 12582912.f + 128.f;
                    const f32x2 lo2 = {acc[f][i][2 * q], acc[f][i][2 * q + 1]}, hi2 = {acc[f][i][2 * q + 8], acc[f][i][2 * q + 9]};
                    const f32x2 tl = __builtin_elementwise_fma(lo2, inv2, magic2), th = __builtin_elementwise_fma(hi2, inv2, magic2);
                    const u32 b0 = __float_as_uint(__builtin_amdgcn_fmed3f(tl.x, MAGIC - 127.f, MAGIC + 127.f));
                    const u32 b1 = __float_as_uint(__builtin_amdgcn_fmed3f(th.x, MAGIC - 127.f, MAGIC + 127.f));
                    const u32 b2 = __float_as_uint(__builtin_amdgcn_fmed3f(tl.y, MAGIC - 127.f, MAGIC + 127.f));
                    const u32 b3 = __float_as_uint(__builtin_amdgcn_fmed3f(th.y, MAGIC - 127.f, MAGIC + 127.f));
                    const u32 d = __builtin_amdgcn_perm(__builtin_amdgcn_perm(b3, b2, 0x0c0c0400u), __builtin_amdgcn_perm(b1, b0, 0x0c0c0400u), 0x05040100u) ^ 0x80808080u;
                    v[q] = poison ? 0x80808080u : d;
                }
                const u32 row = 160 * wm + 32 * i + (lane & 31);
                *(LAS u32x4*)(lds + row * 128 + ((ch ^ (row & 7)) * 16)) = v;
            }
        }
    }
    LGKM0();
    __builtin_amdgcn_s_barrier();
    if (wave == 0) STAMP(12);
    const u32 row0 = 320u * mt;
    const u32 nrow8 = mt == MT - 1 ? 32u : 40u;
    if (wave == 0) {
        for (u32 gq = 0; gq < nrow8; gq += OUTT) {
#pragma unroll
            for (u32 it = 0; it < OUTT; ++it) {
                const u32 g = ((gq + it) * 23u + nt) % nrow8;
                const u32 row = 8 * g + (lane >> 3), ch = lane & 7;
                const u32x4 v = *(const LAS u32x4*)(lds + row * 128 + ((ch ^ (row & 7)) * 16));
                const u32 grow = row0 + 8 * g;
                const rsrc_t ob = mkr(P.out[(grow >> 9) & 7u], OUT_BYTES);
                st128(v, ob, ((grow & 511u) + (lane >> 3)) * NN + rank * NL + nt * 128 + ch * 16);
            }
            VM0();
        }
    }
    if (wave == 0) STAMP(5);
    VM0();
    __builtin_amdgcn_s_barrier();
    if (wave == 0) {
        STAMP(6);
        const u32 d0 = row0 >> 9, d1 = (row0 + 8u * nrow8 - 1u) >> 9;
        if (lane <= d1 - d0) {
            const u32 d = (d0 + lane) & 7u;
            __hip_atomic_store((GAS u32*)(P.ws[d] + DON_OFF + ((rank * 3u + (mt - (512u * d) / 320u)) * 16u + nt) * 128u), call, __ATOMIC_RELAXED, __HIP_MEMORY_SCOPE_SYSTEM);
        }
        const u32 cnt = (512u * rank + 511u) / 320u - (512u * rank) / 320u + 1u;
        for (u32 e = gi; e < 8u * cnt * 16u; e += NGEMM) {
            GAS u32* f = (GAS u32*)(myws + DON_OFF + (((e / (cnt * 16u)) * 3u + (e / 16u) % cnt) * 16u + e % 16u) * 128u);
            const u64 t0 = rt();
            for (u32 spins = 1; rfl(__hip_atomic_load(f, __ATOMIC_RELAXED, __HIP_MEMORY_SCOPE_SYSTEM)) < call; ++spins) {
                if ((spins & 15u) == 0u && give_up(tmo, call, t0)) {
                    if (lane == 0) __hip_atomic_store((GAS unsigned char*)(P.out[rank] + rank * NL), (unsigned char)0x80, __ATOMIC_RELAXED, __HIP_MEMORY_SCOPE_SYSTEM);
                    break;
                }
                __builtin_amdgcn_s_sleep(1);
            }
        }
        STAMP(7);
    }
}

extern "C" void kernel_launch(int rank, int nranks, ncclComm_t comm, hipStream_t stream,
                              void* const* d_in, const int* in_sizes, int n_in,
                              void* d_out, int out_size, const int* in_dtypes, int out_dtype,
                              void* const* d_out_all, void* const* d_ws_all, size_t ws_bytes) {
    (void)comm; (void)in_dtypes; (void)out_dtype; (void)d_out;
    if (nranks != NR || (unsigned)rank >= (unsigned)NR || n_in != 2 || in_sizes[0] != MM * KSH || in_sizes[1] != KSH * NN ||
        out_size != (int)OUT_BYTES || ws_bytes < WS_USED) {
        fprintf(stderr, "kernel_launch: unexpected shapes\n");
        return;
    }
    Params P;
    P.x = d_in[0];
    P.w = d_in[1];
    for (int p = 0; p < NR; ++p) { P.ws[p] = (char*)d_ws_all[p]; P.out[p] = (char*)d_out_all[p]; }
    P.rank = rank;
    P.pad = 0;
    hipLaunchKernelGGL(gemm_ag_q8, dim3(256), dim3(512), 0, stream, P);
}
```

```cpp
#include <hip/hip_runtime.h>
#include <rccl/rccl.h>
#include <cstdio>
#include <cstdint>

typedef unsigned u32;
typedef unsigned long long u64;
typedef u32 u32x4 __attribute__((ext_vector_type(4)));
typedef u32 u32x2 __attribute__((ext_vector_type(2)));
typedef float f32x16 __attribute__((ext_vector_type(16)));
typedef float f32x2 __attribute__((ext_vector_type(2)));
typedef __bf16 bf16x8 __attribute__((ext_vector_type(8)));
typedef __amdgpu_buffer_rsrc_t rsrc_t;
#define LAS __attribute__((address_space(3)))
#define GAS __attribute__((address_space(1)))

constexpr int NR = 8;
constexpr int MM = 4096;
constexpr int KSH = 512;
constexpr int NN = 16384;
constexpr int NL = NN / NR;
constexpr u32 X_BYTES = MM * KSH * 2;
constexpr u32 W_BYTES = KSH * NN * 2;
constexpr u32 OUT_BYTES = (MM / NR) * NN;
constexpr u32 BLK = 4096;
constexpr u32 XS_REL = 0;
constexpr u32 WS_REL = 64u * 128u * BLK;
constexpr u32 STG_BYTES = WS_REL + 64u * 64u * BLK;
constexpr u32 PST_OFF = 0;
constexpr u32 RAW_OFF = STG_BYTES;
constexpr u32 NTASK = 768;
constexpr u32 PCK_OFF = 2u * STG_BYTES;
constexpr u32 PRD_OFF = PCK_OFF + 2560u * 8192u;
constexpr u32 ARR_OFF = PRD_OFF + 2560u * 64u;
constexpr u32 MT = 13;
constexpr u32 NGEMM = MT * 16;
constexpr u32 NCOMM = 256 - NGEMM;
constexpr u32 RDX_OFF = ARR_OFF + 8u * NTASK * 64u;
constexpr u32 RDW_OFF = RDX_OFF + MT * 64u * 32u;
constexpr u32 LMX_OFF = RDW_OFF + 16u * 64u * 8u;
constexpr u32 AMX_OFF = LMX_OFF + 256u * 8u;
constexpr u32 DON_OFF = AMX_OFF + 8u * 128u;
constexpr u32 QUE_OFF = DON_OFF + 384u * 128u;
constexpr u32 CTR_OFF = QUE_OFF + 8u * 128u;
constexpr u32 TMO_OFF = CTR_OFF + 128u;
constexpr u32 WS_USED = CTR_OFF + 4096u;
constexpr int AUX_SYS = 17;
constexpr u64 DEADLINE = 100000000ull;
constexpr u32 OOB = 0x40000000u;

constexpr u32 NDECWG = 16;
constexpr u32 NPUSH = 6;
constexpr u32 KPUSH = 27;
#define PB 3
#define PB2 5
constexpr u32 NRAW = 48;
#define STAGGER 40
#define OUTWIN1 1
static_assert(7 * KPUSH <= (NCOMM - NDECWG) * NPUSH && NPUSH < 8, "pushers");
static_assert(NRAW <= 64, "the unpacked tail consists of x tasks");

constexpr u32 SLOT = 14 * 2048;
constexpr u32 NSLOT = 5;
constexpr u32 L_CTL = NSLOT * SLOT;
constexpr u32 L_FULL = L_CTL;
constexpr u32 L_FREE = L_CTL + 32;
constexpr u32 L_OCNT = L_CTL + 64;
constexpr u32 L_WMAX = L_CTL + 68;
constexpr u32 L_WCNT = L_CTL + 72;
constexpr u32 L_GMAX = L_CTL + 76;
constexpr u32 L_BAD = L_CTL + 80;
constexpr u32 L_GOK = L_CTL + 84;
constexpr u32 L_OUT = L_CTL + 96;
constexpr u32 L_ORD = L_CTL + 128;
constexpr u32 L_SCR = L_ORD + 256;
constexpr u32 LDS_BYTES = L_SCR + 4 * 4352;

struct Params {
    const void* x;
    const void* w;
    char* ws[NR];
    char* out[NR];
    int rank;
    int pad;
};

__device__ __forceinline__ rsrc_t mkr(const void* p, u32 bytes) {
    return __builtin_amdgcn_make_buffer_rsrc((void*)p, 0, bytes, 0x00020000);
}
__device__ __forceinline__ u64 rt() { return __builtin_amdgcn_s_memrealtime(); }
__device__ __forceinline__ u32 rfl(u32 v) { return __builtin_amdgcn_readfirstlane(v); }
__device__ __forceinline__ void st128(u32x4 v, rsrc_t r, u32 off) { __builtin_amdgcn_raw_buffer_store_b128(v, r, off, 0, 17); }
#define STR2(x) #x
#define STR(x) STR2(x)
#define LGKM0() asm volatile("s_waitcnt lgkmcnt(0)" ::: "memory")
#define VM0() asm volatile("s_waitcnt vmcnt(0)" ::: "memory")

__device__ __forceinline__ u32 lds_ld(const LAS char* l, u32 off) {
    return __hip_atomic_load((const LAS u32*)(l + off), __ATOMIC_RELAXED, __HIP_MEMORY_SCOPE_WORKGROUP);
}
__device__ __forceinline__ void lds_st(LAS char* l, u32 off, u32 v) {
    __hip_atomic_store((LAS u32*)(l + off), v, __ATOMIC_RELAXED, __HIP_MEMORY_SCOPE_WORKGROUP);
}
__device__ __forceinline__ void lds_add(LAS char* l, u32 off, u32 v) {
    (void)__hip_atomic_fetch_add((LAS u32*)(l + off), v, __ATOMIC_RELAXED, __HIP_MEMORY_SCOPE_WORKGROUP);
}
__device__ __forceinline__ void lds_wait_ge(const LAS char* l, u32 off, u32 want) {
    while (rfl(lds_ld(l, off)) < want) __builtin_amdgcn_s_sleep(1);
    asm volatile("" ::: "memory");
}
__device__ __forceinline__ bool give_up(GAS u32* tmo, u32 call, u64 t0) {
    if (rfl(__hip_atomic_load(tmo, __ATOMIC_RELAXED, __HIP_MEMORY_SCOPE_AGENT)) == call || rt() - t0 > DEADLINE) {
        __hip_atomic_store(tmo, call, __ATOMIC_RELAXED, __HIP_MEMORY_SCOPE_AGENT);
        return true;
    }
    return false;
}
__device__ __forceinline__ u32 wave_max_u32(u32 v) {
    u32 o;
    o = __builtin_amdgcn_update_dpp(0u, v, 0x111, 0xf, 0xf, false); v = v > o ? v : o;
    o = __builtin_amdgcn_update_dpp(0u, v, 0x112, 0xf, 0xf, false); v = v > o ? v : o;
    o = __builtin_amdgcn_update_dpp(0u, v, 0x114, 0xf, 0xf, false); v = v > o ? v : o;
    o = __builtin_amdgcn_update_dpp(0u, v, 0x118, 0xf, 0xf, false); v = v > o ? v : o;
    o = __builtin_amdgcn_update_dpp(0u, v, 0x142, 0xa, 0xf, false); v = v > o ? v : o;
    o = __builtin_amdgcn_update_dpp(0u, v, 0x143, 0xc, 0xf, false); v = v > o ? v : o;
    return __builtin_amdgcn_readlane(v, 63);
}

constexpr u32 EXC_CAP = 8;
constexpr u32 META_RAW = 0x800u;
constexpr u32 EXC_REL = 6144;
__device__ __forceinline__ u32 pkmax_u16(u32 a, u32 b) {
    typedef unsigned short u16x2 __attribute__((ext_vector_type(2)));
    return __builtin_bit_cast(u32, __builtin_elementwise_max(__builtin_bit_cast(u16x2, a), __builtin_bit_cast(u16x2, b)));
}
__device__ __forceinline__ u32 encode_task(const u32* V, u32x4* O, rsrc_t dst, u32 slot_off, u32 lane) {
    u32 m = V[0] & 0x7fff7fffu;
#pragma unroll
    for (int i = 1; i < 32; ++i) m = pkmax_u16(m, V[i] & 0x7fff7fffu);
    m = pkmax_u16(m, m >> 16) & 0xffffu;
    const u32 top = wave_max_u32(m) >> 8;
    const u32 base = top > 7u ? top - 7u : 0u;
    const u32 b4 = base * 0x01010101u;
    u32 bad = 0;
    u32 Lp[16], X[8];
#pragma unroll
    for (int t = 0; t < 8; ++t) {
        const u32 h01 = __builtin_amdgcn_perm(V[4 * t + 1], V[4 * t], 0x07050301u), h23 = __builtin_amdgcn_perm(V[4 * t + 3], V[4 * t + 2], 0x07050301u);
        Lp[2 * t] = __builtin_amdgcn_perm(V[4 * t + 1], V[4 * t], 0x06040200u);
        Lp[2 * t + 1] = __builtin_amdgcn_perm(V[4 * t + 3], V[4 * t + 2], 0x06040200u);
        const u32 u01 = (h01 | 0x80808080u) - b4, u23 = (h23 | 0x80808080u) - b4;
        bad |= (u01 ^ 0x80808080u) | (u23 ^ 0x80808080u);
        X[t] = (u01 & 0x07070707u) | ((h01 >> 4) & 0x08080808u) | ((u23 & 0x07070707u) << 4) | (h23 & 0x80808080u);
    }
    u32 meta = base;
    if (__any((bad & 0xf8f8f8f8u) != 0u)) {
        u64 bm = 0;
#pragma unroll
        for (int i = 0; i < 64; ++i) {
            const u32 h7 = (V[i >> 1] >> (8 + 16 * (i & 1))) & 0x7fu;
            bm |= (u64)(h7 - base > 7u ? 1u : 0u) << i;
        }
        u32 n = 0;
        for (;;) {
            const u64 act = __ballot(bm != 0ull);
            if (!act) break;
            const u32 cnt = __builtin_popcountll(act);
            if (n + cnt > EXC_CAP) { n = EXC_CAP + 1u; break; }
            const u32 pos = n + __builtin_amdgcn_mbcnt_hi((u32)(act >> 32), __builtin_amdgcn_mbcnt_lo((u32)act, 0u));
            const u32 i = bm ? __builtin_ctzll(bm) : 0u;
            u32 dw = V[0];
#pragma unroll
            for (int q = 1; q < 32; ++q) dw = (i >> 1) == (u32)q ? V[q] : dw;
            const u32 val = (dw >> (16u * (i & 1u))) & 0xffffu;
            const u32 ent = (lane << 22) | (i << 16) | val;
            __builtin_amdgcn_raw_buffer_store_b32(ent, dst, bm ? EXC_REL + pos * 4u : OOB, slot_off, AUX_SYS);
            bm &= bm - 1ull;
            n += cnt;
        }
        meta = n > EXC_CAP ? META_RAW : (base | (n << 7));
    }
    if (meta & META_RAW) {
#pragma unroll
        for (int t = 0; t < 8; ++t) { O[t].x = V[4 * t]; O[t].y = V[4 * t + 1]; O[t].z = V[4 * t + 2]; O[t].w = V[4 * t + 3]; }
    } else {
#pragma unroll
        for (int t = 0; t < 4; ++t) { O[t].x = Lp[4 * t]; O[t].y = Lp[4 * t + 1]; O[t].z = Lp[4 * t + 2]; O[t].w = Lp[4 * t + 3]; }
#pragma unroll
        for (int t = 0; t < 2; ++t) { O[4 + t].x = X[4 * t]; O[4 + t].y = X[4 * t + 1]; O[4 + t].z = X[4 * t + 2]; O[4 + t].w = X[4 * t + 3]; }
    }
    return meta;
}
__device__ __forceinline__ void decode_task(u32x4* B, u32 meta, u32 exc, u32 lane) {
    const u32 b4 = (meta & 0x7fu) * 0x01010101u;
    u32 V[32];
#pragma unroll
    for (int t = 0; t < 8; ++t) {
        const u32 x = B[4 + (t >> 2)][t & 3], l01 = B[t >> 1][2 * (t & 1)], l23 = B[t >> 1][2 * (t & 1) + 1];
        const u32 he = ((x & 0x07070707u) + b4) | ((x << 4) & 0x80808080u);
        const u32 ho = (((x >> 4) & 0x07070707u) + b4) | (x & 0x80808080u);
        V[4 * t + 0] = __builtin_amdgcn_perm(he, l01, 0x05010400u);
        V[4 * t + 1] = __builtin_amdgcn_perm(he, l01, 0x07030602u);
        V[4 * t + 2] = __builtin_amdgcn_perm(ho, l23, 0x05010400u);
        V[4 * t + 3] = __builtin_amdgcn_perm(ho, l23, 0x07030602u);
    }
    const u32 cnt = (meta >> 7) & 15u;
    for (u32 e = 0; e < cnt; ++e) {
        const u32 ent = __builtin_amdgcn_readlane(exc, e);
        const u32 i = (ent >> 16) & 63u, sh = 16u * (i & 1u);
        const bool me = lane == (ent >> 22);
        const u32 keep = ~(0xffffu << sh), put = (ent & 0xffffu) << sh;
#pragma unroll
        for (int q = 0; q < 32; ++q) V[q] = (me && (i >> 1) == (u32)q) ? ((V[q] & keep) | put) : V[q];
    }
#pragma unroll
    for (int t = 0; t < 8; ++t) { B[t].x = V[4 * t]; B[t].y = V[4 * t + 1]; B[t].z = V[4 * t + 2]; B[t].w = V[4 * t + 3]; }
}

constexpr u32 META_TIGHT = 0x10000u;
constexpr u32 SGN_REL = 5120, LST_REL = 5632;
constexpr u32 SCR_BYTES = 4352;
__device__ __forceinline__ u32 wave_scan_incl(u32 v) {
    v += __builtin_amdgcn_update_dpp(0u, v, 0x111, 0xf, 0xf, false);
    v += __builtin_amdgcn_update_dpp(0u, v, 0x112, 0xf, 0xf, false);
    v += __builtin_amdgcn_update_dpp(0u, v, 0x114, 0xf, 0xf, false);
    v += __builtin_amdgcn_update_dpp(0u, v, 0x118, 0xf, 0xf, false);
    v += __builtin_amdgcn_update_dpp(0u, v, 0x142, 0xa, 0xf, false);
    v += __builtin_amdgcn_update_dpp(0u, v, 0x143, 0xc, 0xf, false);
    return v;
}
__device__ __forceinline__ u32 encode_tight(const u32* V, u32x4* O, u32x2& S, u32& list, LAS char* sc, u32 lane) {
    u32 m = V[0] & 0x7fff7fffu;
#pragma unroll
    for (int i = 1; i < 32; ++i) m = pkmax_u16(m, V[i] & 0x7fff7fffu);
    m = pkmax_u16(m, m >> 16) & 0xffffu;
    const u32 top = wave_max_u32(m) >> 8;
    const u32 base = top > 3u ? top - 3u : 0u;
    const u32 b4 = base * 0x01010101u;
    u32 U[16], C[4] = {0, 0, 0, 0}, Sg[2] = {0, 0}, E[2] = {0, 0};
#pragma unroll
    for (int k = 0; k < 16; ++k) {
        const int t = k >> 1, o = 2 * (k & 1);
        const u32 h = __builtin_amdgcn_perm(V[4 * t + o + 1], V[4 * t + o], 0x07050301u);
        O[k >> 2][k & 3] = __builtin_amdgcn_perm(V[4 * t + o + 1], V[4 * t + o], 0x06040200u);
        const u32 u = (h | 0x80808080u) - b4;
        const u32 e = (u ^ 0x80808080u) & 0xfcfcfcfcu;
        const u32 nz = (((e & 0x7c7c7c7cu) + 0x7c7c7c7cu) | e) & 0x80808080u;
        U[k] = u;
        C[k >> 2] |= (u & 0x03030303u) << (2 * (k & 3));
        Sg[k >> 3] |= (h & 0x80808080u) >> (7 - (k & 7));
        E[k >> 3] |= nz >> (7 - (k & 7));
    }
    const u32 n = __builtin_popcount(E[0]) + __builtin_popcount(E[1]);
    const u32 incl = wave_scan_incl(n);
    const u32 total = __builtin_amdgcn_readlane(incl, 63);
    if (total > 128u) return 0u;
    list = 0;
    if (total) {
#pragma unroll
        for (int j = 0; j < 4; ++j) { const u32x4 q = {U[4 * j], U[4 * j + 1], U[4 * j + 2], U[4 * j + 3]}; *(LAS u32x4*)(sc + j * 1024 + lane * 16) = q; }
        u32 pos = incl - n, e0 = E[0], e1 = E[1];
        bool far = false;
        while (__any((e0 | e1) != 0u)) {
            const bool has = (e0 | e1) != 0u;
            const u32 w = e0 ? 0u : 1u, mk = e0 ? e0 : e1;
            const u32 p = has ? __builtin_ctz(mk) : 0u;
            const u32 k = 8u * w + (p & 7u);
            const u32 ub = *(const LAS unsigned char*)(sc + (k >> 2) * 1024u + lane * 16u + (k & 3u) * 4u + (p >> 3));
            const u32 code = (0x7fu - ub) >> 2;
            if (has) {
                far |= code > 15u;
                *(LAS unsigned short*)(sc + 4096u + pos * 2u) = (unsigned short)((lane << 10) | ((w * 32u + p) << 4) | (code & 15u));
                ++pos;
            }
            if (e0) e0 &= e0 - 1u; else e1 &= e1 - 1u;
        }
        if (__any(far)) return 0u;
        list = *(const LAS u32*)(sc + 4096u + lane * 4u);
    }
    O[4].x = C[0]; O[4].y = C[1]; O[4].z = C[2]; O[4].w = C[3];
    S.x = Sg[0]; S.y = Sg[1];
    return base | META_TIGHT | (total << 20);
}
__device__ __forceinline__ void decode_tight(u32x4* B, u32x2 S, u32 list, u32 meta, LAS char* sc, u32 lane) {
    const u32 b4 = (meta & 0x7fu) * 0x01010101u;
    u32 H[16];
#pragma unroll
    for (int k = 0; k < 16; ++k)
        H[k] = (((B[4][k >> 2] >> (2 * (k & 3))) & 0x03030303u) + b4) | ((S[k >> 3] << (7 - (k & 7))) & 0x80808080u);
    const u32 total = (meta >> 20) & 0xffu;
    if (total) {
#pragma unroll
        for (int j = 0; j < 4; ++j) { const u32x4 q = {H[4 * j], H[4 * j + 1], H[4 * j + 2], H[4 * j + 3]}; *(LAS u32x4*)(sc + j * 1024 + lane * 16) = q; }
#pragma unroll
        for (int e = 0; e < 2; ++e) {
            const u32 ent = e ? list >> 16 : list & 0xffffu;
            const u32 p = (ent >> 4) & 31u, k = ((ent >> 9) & 1u) * 8u + (p & 7u);
            if (2u * lane + e < total)
                (void)__hip_atomic_fetch_sub((LAS u32*)(sc + (k >> 2) * 1024u + (ent >> 10) * 16u + (k & 3u) * 4u), (4u * ((ent & 15u) + 1u)) << (8u * (p >> 3)),
                                             __ATOMIC_RELAXED, __HIP_MEMORY_SCOPE_WORKGROUP);
        }
        asm volatile("" ::: "memory");
#pragma unroll
        for (int j = 0; j < 4; ++j) { const u32x4 q = *(const LAS u32x4*)(sc + j * 1024 + lane * 16); H[4 * j] = q.x; H[4 * j + 1] = q.y; H[4 * j + 2] = q.z; H[4 * j + 3] = q.w; }
    }
    u32 V[32];
#pragma unroll
    for (int t = 0; t < 8; ++t) {
        const u32 l01 = B[t >> 1][2 * (t & 1)], l23 = B[t >> 1][2 * (t & 1) + 1];
        V[4 * t + 0] = __builtin_amdgcn_perm(H[2 * t], l01, 0x05010400u);
        V[4 * t + 1] = __builtin_amdgcn_perm(H[2 * t], l01, 0x07030602u);
        V[4 * t + 2] = __builtin_amdgcn_perm(H[2 * t + 1], l23, 0x05010400u);
        V[4 * t + 3] = __builtin_amdgcn_perm(H[2 * t + 1], l23, 0x07030602u);
    }
#pragma unroll
    for (int t = 0; t < 8; ++t) { B[t].x = V[4 * t]; B[t].y = V[4 * t + 1]; B[t].z = V[4 * t + 2]; B[t].w = V[4 * t + 3]; }
}

__device__ __forceinline__ bool task_is_w(u32 u) { return (u % 96u) < 32u; }
__device__ __forceinline__ u32 task_xb(u32 src, u32 dst, u32 u) { return (u % 96u - 32u + ((src - dst) & 7u) * 8u) & 63u; }
__device__ __forceinline__ u32 task_slot(u32 src, u32 dst, u32 u) {
    const u32 st = src * 8u + u / 96u;
    return task_is_w(u) ? WS_REL + (st * 64u + 2u * (u % 96u)) * BLK : XS_REL + (st * 128u + 2u * task_xb(src, dst, u)) * BLK;
}
__device__ __forceinline__ u32 task_rdy(u32 src, u32 dst, u32 u) {
    const u32 st = src * 8u + u / 96u;
    if (task_is_w(u)) { const u32 wb = u % 96u; return RDW_OFF + ((wb >> 1) * 64u + st) * 8u + (wb & 1u) * 4u; }
    const u32 xb = task_xb(src, dst, u);
    return RDX_OFF + ((xb / 5u) * 64u + st) * 32u + (xb % 5u) * 4u;
}
__device__ __forceinline__ u32 task_pck(u32 src, u32 dst, u32 u) {
    const u32 j = u / 96u;
    return task_is_w(u) ? 512u + (dst * 8u + j) * 32u + u % 96u : j * 64u + task_xb(src, dst, u);
}
__device__ __forceinline__ u32 raw_lane_off(bool is_w, u32 lane) {
    return is_w ? ((lane >> 4) & 1u) * 4096u + (lane >> 5) * 512u + (lane & 15u) * 16u : (lane & 7u) * 512u + (lane >> 3) * 16u;
}
__device__ __forceinline__ u32 raw_vec_off(bool is_w, int v) { return is_w ? (v >> 1) * 1024u + (v & 1) * 256u : (v >> 2) * 4096u + (v & 3) * 128u; }

constexpr u32 NDEC = NGEMM + NDECWG * 8u;
__device__ __forceinline__ void decoder(char* myws, u32 rank, u32 call, u32 di, u32 lane, LAS char* sc) {
    const rsrc_t ws = mkr(myws, WS_USED);
    GAS u32* const tmo = (GAS u32*)(myws + TMO_OFF);
    const u32 sv = lane * 16u;
    const u32 fv = lane == 0 ? 0u : OOB;
    u32 owed = 0;
    constexpr u32 QE = 7u * (NTASK - NRAW);
    for (u32 q = di; q < QE;) {
        const u32 qn = q + NDEC;
        const u32 u = q / 7u, src = (rank + 1u + q % 7u) & 7u;
        GAS u64* arr = (GAS u64*)(myws + ARR_OFF + (src * NTASK + u) * 64u);
        const u64 t0 = rt();
        u64 f;
        for (u32 spins = 0;; ++spins) {
            f = __hip_atomic_load(arr, __ATOMIC_RELAXED, __HIP_MEMORY_SCOPE_SYSTEM);
            const u32 seen = rfl((u32)(f >> 32));
            if (owed) { VM0(); __builtin_amdgcn_raw_buffer_store_b32(call, ws, fv, owed, AUX_SYS); owed = 0; }
            if (seen >= call) break;
            if ((spins & 15u) == 15u && give_up(tmo, call, t0)) break;
            __builtin_amdgcn_s_sleep(1);
        }
        const u32 meta = rfl((u32)f);
        const u32 so = task_slot(src, rank, u);
        u32x4 B[8];
#pragma unroll
        for (int i = 0; i < 5; ++i) B[i] = __builtin_amdgcn_raw_buffer_load_b128(ws, sv + i * 1024, PST_OFF + so, AUX_SYS);
        if (meta & META_TIGHT) {
            const u32x2 S = __builtin_amdgcn_raw_buffer_load_b64(ws, lane * 8u + SGN_REL, PST_OFF + so, AUX_SYS);
            const u32 list = __builtin_amdgcn_raw_buffer_load_b32(ws, lane * 4u + LST_REL, PST_OFF + so, AUX_SYS);
            decode_tight(B, S, list, meta, sc, lane);
        } else {
            B[5] = __builtin_amdgcn_raw_buffer_load_b128(ws, sv + 5120, PST_OFF + so, AUX_SYS);
            if (meta & META_RAW) {
                B[6] = __builtin_amdgcn_raw_buffer_load_b128(ws, sv + 6144, PST_OFF + so, AUX_SYS);
                B[7] = __builtin_amdgcn_raw_buffer_load_b128(ws, sv + 7168, PST_OFF + so, AUX_SYS);
            } else {
                u32 exc = 0;
                if (meta & 0x780u) exc = __builtin_amdgcn_raw_buffer_load_b32(ws, lane * 4u + EXC_REL, PST_OFF + so, AUX_SYS);
                decode_task(B, meta, exc, lane);
            }
        }
        const bool isw = task_is_w(u);
        const u32 lo = raw_lane_off(isw, lane) + RAW_OFF + so;
#pragma unroll
        for (int i = 0; i < 8; ++i) st128(B[i], ws, lo + raw_vec_off(isw, i));
        owed = task_rdy(src, rank, u);
        if (qn >= QE || u >= NTASK - 96u) {
            VM0(); __builtin_amdgcn_raw_buffer_store_b32(call, ws, fv, owed, AUX_SYS); owed = 0;
        }
        q = qn;
    }
    if (owed) { VM0(); __builtin_amdgcn_raw_buffer_store_b32(call, ws, fv, owed, AUX_SYS); }
}


__global__ void __launch_bounds__(512) gemm_ag_q8(Params P) {
    __shared__ __attribute__((aligned(16))) char smem_[LDS_BYTES];
    LAS char* lds = (LAS char*)smem_;
    const int tid = threadIdx.x;
    const u32 lane = tid & 63;
    const u32 wave = rfl(tid >> 6);
    const u32 b = blockIdx.x;
    const u32 rank = P.rank;
    const u32 xcd = b & 7, idx = b >> 3;
    char* const myws = P.ws[rank];
    GAS u32* const ctr = (GAS u32*)(myws + CTR_OFF);
    GAS u32* const tmo = (GAS u32*)(myws + TMO_OFF);

    if (idx >= 26u) {
        const u32 ci = (idx - 26u) * 8u + xcd;
        if (ci >= NCOMM - NDECWG) {
            const u32 call = rfl(__hip_atomic_load(ctr, __ATOMIC_RELAXED, __HIP_MEMORY_SCOPE_AGENT)) + 1u;
            decoder(myws, rank, call, NGEMM + (ci - (NCOMM - NDECWG)) * 8u + wave, lane, lds + wave * SCR_BYTES);
            return;
        }
        if (wave == NPUSH) {
            if (ci != 0) return;
            const u32 call = rfl(__hip_atomic_load(ctr, __ATOMIC_RELAXED, __HIP_MEMORY_SCOPE_AGENT)) + 1u;
            const rsrc_t ws = mkr(myws, WS_USED);
            const u32 off = LMX_OFF + lane * 32u;
            const bool has = lane * 4u < NGEMM;
            const u64 t1 = rt();
            u32 gm = 0;
            for (u32 spins = 1;; ++spins) {
                const u32x4 a = __builtin_amdgcn_raw_buffer_load_b128(ws, off, 0, AUX_SYS);
                const u32x4 c = __builtin_amdgcn_raw_buffer_load_b128(ws, off + 16, 0, AUX_SYS);
                if (__all(!has || (a.y >= call && a.w >= call && c.y >= call && c.w >= call))) {
                    const u32 m1 = a.x > a.z ? a.x : a.z, m2 = c.x > c.z ? c.x : c.z;
                    gm = has ? (m1 > m2 ? m1 : m2) : 0u;
                    break;
                }
                if ((spins & 15u) == 0u && give_up(tmo, call, t1)) break;
                __builtin_amdgcn_s_sleep(1);
            }
            gm = wave_max_u32(gm);
            const u64 g = ((u64)call << 32) | gm;
            __hip_atomic_store((GAS u64*)(P.ws[lane >> 3] + AMX_OFF + (lane & 7) * 128 + rank * 8), g, __ATOMIC_RELAXED, __HIP_MEMORY_SCOPE_SYSTEM);
            return;
        }
        if (wave > NPUSH) return;
        const u32 sid = ci * NPUSH + wave;
        if (sid >= 7u * KPUSH) return;
        const u32 d = (rank + 1u + sid % 7u) & 7u, k = sid / 7u;
        const u32 call = rfl(__hip_atomic_load(ctr, __ATOMIC_RELAXED, __HIP_MEMORY_SCOPE_AGENT)) + 1u;
        const rsrc_t ws = mkr(myws, WS_USED), dst = mkr(P.ws[d], WS_USED);
        const u32 sv = lane * 16u;
        const u32 fv = lane == 0 ? 0u : OOB;
        GAS u32* const que = (GAS u32*)(myws + QUE_OFF + d * 128u);
        const u32 qbase = (call - 1u) * (NTASK - 4u * KPUSH) - 5u * KPUSH;
        auto prd_of = [&](u32 u) { return (GAS u64*)(myws + PRD_OFF + task_pck(rank, d, u) * 64u); };
        auto settle = [&](u32 u, u32 lo, u32 hi) {
            if (hi >= call) return lo;
            const u64 t0 = rt();
            for (u32 spins = 1;; ++spins) {
                const u64 f = __hip_atomic_load(prd_of(u), __ATOMIC_RELAXED, __HIP_MEMORY_SCOPE_SYSTEM);
                if (rfl((u32)(f >> 32)) >= call || ((spins & 15u) == 0u && give_up(tmo, call, t0))) return rfl((u32)f);
                __builtin_amdgcn_s_sleep(1);
            }
        };
        auto load_task = [&](u32x4* B, u32 u) {
            const u32 so = u >= NTASK - NRAW ? RAW_OFF + task_slot(rank, d, u) : PCK_OFF + task_pck(rank, d, u) * 8192u;
#pragma unroll
            for (int i = 0; i < 8; ++i) B[i] = __builtin_amdgcn_raw_buffer_load_b128(ws, sv + (i & 3) * 1024, so + (i >> 2) * 4096, AUX_SYS);
        };
        u32x4 A[8], B[8];
        static_assert(3 * KPUSH <= 96, "the tasks packed by the pushers lie in k slice 0");
        u32 c, n1, metac;
        {
            const rsrc_t xin = mkr(P.x, X_BYTES), win = mkr(P.w, W_BYTES);
            const u32 xv = (lane >> 3) * 1024u + (lane & 7u) * 16u;
            const u32 c16 = lane & 15u;
            const u32 wv = (lane >> 5) * 262144u + ((lane & 16u) + 8u * ((c16 >> 2) & 1u) + 4u * (c16 >> 3) + (c16 & 3u)) * 4u;
            u32 R[32], V[32];
            auto fetch = [&](u32 u) {
                if (u < 32u) {
                    const u32 base = (2048u * d + 64u * u) * 2u;
#pragma unroll
                    for (int i = 0; i < 32; ++i) R[i] = __builtin_amdgcn_raw_buffer_load_b32(win, wv, base + ((i >> 3) * 16u + (i & 7)) * 32768u, 0);
                } else {
                    const u32 base = task_xb(rank, d, u) * 65536u;
#pragma unroll
                    for (int i = 0; i < 8; ++i) {
                        const u32x4 v = __builtin_amdgcn_raw_buffer_load_b128(xin, xv, base + i * 8192u, 0);
                        R[4 * i] = v.x; R[4 * i + 1] = v.y; R[4 * i + 2] = v.z; R[4 * i + 3] = v.w;
                    }
                }
            };
            auto arrange = [&](u32 u) {
                if (u < 32u) {
#pragma unroll
                    for (int g = 0; g < 4; ++g)
#pragma unroll
                        for (int q = 0; q < 4; ++q) {
                            V[8 * g + q] = __builtin_amdgcn_perm(R[8 * g + 2 * q + 1], R[8 * g + 2 * q], 0x05040100u);
                            V[8 * g + 4 + q] = __builtin_amdgcn_perm(R[8 * g + 2 * q + 1], R[8 * g + 2 * q], 0x07060302u);
                        }
                } else {
#pragma unroll
                    for (int i = 0; i < 32; ++i) V[i] = R[i];
                }
            };
            auto send = [&](u32x4* O, u32 u, u32 meta) {
                const u32 slot = PST_OFF + task_slot(rank, d, u);
#pragma unroll
                for (int i = 0; i < 6; ++i) st128(O[i], dst, sv + slot + i * 1024u);
                if (meta & META_RAW) { st128(O[6], dst, sv + slot + 6144u); st128(O[7], dst, sv + slot + 7168u); }
            };
            auto flag = [&](u32 u, u32 meta) {
                VM0();
                u32x2 f; f.x = meta; f.y = call;
                __builtin_amdgcn_raw_buffer_store_b64(f, dst, fv, ARR_OFF + (rank * NTASK + u) * 64u, AUX_SYS);
            };
            const u32 u0 = k, u1 = k + KPUSH, u2 = k + 2u * KPUSH;
            fetch(u0);
            arrange(u0); fetch(u1);
            const u32 m0 = encode_task(V, A, dst, PST_OFF + task_slot(rank, d, u0), lane);
            send(A, u0, m0);
            arrange(u1); fetch(u2);
            const u32 m1 = encode_task(V, B, dst, PST_OFF + task_slot(rank, d, u1), lane);
            flag(u0, m0);
            send(B, u1, m1);
            arrange(u2);
            const u32 m2 = encode_task(V, A, dst, PST_OFF + task_slot(rank, d, u2), lane);
            flag(u1, m1);
            send(A, u2, m2);
            c = k + 3u * KPUSH; n1 = k + 4u * KPUSH;
            metac = settle(c, 0u, 0u);
            load_task(B, c);
            flag(u2, m2);
            asm volatile("s_waitcnt vmcnt(0)" : "+v"(B[0]), "+v"(B[1]), "+v"(B[2]), "+v"(B[3]), "+v"(B[4]), "+v"(B[5]), "+v"(B[6]), "+v"(B[7]) :: "memory");
        }

        u32 fa = OOB, fb = OOB, oa = 0, ob = 0;
        u32x2 fo; fo.x = 0; fo.y = call;
#define ST1(CUR, i) do { \
            if ((i) < 5) st128(CUR[i], dst, so + (i) * 1024u); \
            else if ((i) == 5) st128(CUR[5], dst, (!full && (metac & META_TIGHT) && lane >= 48u) ? OOB : so + 5120u); \
            else if ((i) == 6) st128(CUR[6], dst, (full || ((metac & 0x780u) && lane < 2u)) ? so + 6144u : OOB); \
            else st128(CUR[7], dst, full ? so + 7168u : OOB); } while (0)
#define PUSH(CUR, NXT) do { \
            const bool tail = c >= NTASK - NRAW; \
            const bool full = tail || (metac & META_RAW); \
            const u32 so = sv + (tail ? RAW_OFF : PST_OFF) + task_slot(rank, d, c); \
            const bool v1 = n1 < NTASK; \
            GAS u64* const pa = prd_of(v1 ? n1 : 0u); \
            u32 tk = 1u, meta1 = 0; \
            u64 pk; \
            asm volatile("global_load_dwordx2 %0, %1, off sc0 sc1" : "=v"(pk) : "v"(pa) : "memory"); \
            __builtin_amdgcn_raw_buffer_store_b64(fo, dst, fa, oa, AUX_SYS); \
            __builtin_amdgcn_raw_buffer_store_b32(call, dst, fb, ob, AUX_SYS); \
            _Pragma("unroll") for (int i = 0; i < PB; ++i) ST1(CUR, i); \
            asm volatile("s_waitcnt vmcnt(" STR(PB2) ")" : "+v"(pk) :: "memory"); \
            if (v1) { \
                meta1 = settle(n1, rfl((u32)pk), rfl((u32)(pk >> 32))); \
                asm volatile("" ::: "memory"); \
                load_task(NXT, n1); \
                if (PB < 8 && n1 < NTASK - KPUSH) asm volatile("s_waitcnt vmcnt(8)" ::: "memory");        \
                _Pragma("unroll") for (int i = PB; i < 8; ++i) ST1(CUR, i); \
                u64 sv_; \
                asm volatile("s_mov_b64 %1, exec\n\ts_mov_b64 exec, 1\n\tglobal_atomic_add %0, %2, %0, off sc0\n\ts_mov_b64 exec, %1\n\t" \
                             "s_waitcnt vmcnt(0)" : "+v"(tk), "=&s"(sv_) : "v"(que) : "memory"); \
                asm volatile("" : "+v"(NXT[0]), "+v"(NXT[1]), "+v"(NXT[2]), "+v"(NXT[3]), "+v"(NXT[4]), "+v"(NXT[5]), "+v"(NXT[6]), \
                             "+v"(NXT[7]) :: "memory");            \
            } else { \
                _Pragma("unroll") for (int i = PB; i < 8; ++i) ST1(CUR, i); \
                VM0(); \
            } \
            fo.x = metac; \
            fa = tail ? OOB : fv; fb = tail ? fv : OOB; \
            oa = ARR_OFF + (rank * NTASK + c) * 64u; ob = task_rdy(rank, d, c); \
            c = n1; metac = meta1; \
            if (v1) n1 = rfl(tk) - qbase; \
            } while (0)
        for (;;) {
            PUSH(B, A);
            if (c >= NTASK) break;
            PUSH(A, B);
            if (c >= NTASK) break;
        }
        __builtin_amdgcn_raw_buffer_store_b64(fo, dst, fa, oa, AUX_SYS);
        __builtin_amdgcn_raw_buffer_store_b32(call, dst, fb, ob, AUX_SYS);
        VM0();
        return;
    }

    const u32 ti4 = (xcd & 1u) * 26u + idx;
    const u32 mt = ti4 >> 2;
    const u32 nt = (xcd >> 1) * 4u + (ti4 & 3u);
    const u32 gi = mt * 16u + nt;
    if (tid < (int)((L_SCR - L_CTL) / 4)) *(LAS u32*)(lds + L_CTL + tid * 4) = 0u;
    const u32 call = rfl(__hip_atomic_load(ctr, __ATOMIC_RELAXED, __HIP_MEMORY_SCOPE_AGENT)) + 1u;
    LGKM0();
    __builtin_amdgcn_s_barrier();

    if (wave == 7) {
        decoder(myws, rank, call, gi, lane, lds + L_SCR + 3 * SCR_BYTES);
        return;
    }
    if (wave == 6) {
        const rsrc_t ws = mkr(myws, WS_USED);
        const u32 fx = RDX_OFF + (mt * 64u + lane) * 32u, fw = RDW_OFF + (nt * 64u + lane) * 8u;
        const bool five = mt < MT - 1;
        u64 pending = ~0ull;
        u32 n = 0;
        const u64 t0 = rt();
        u32 spins = 0;
        while (pending) {
            const bool mine = (pending >> lane) & 1ull;
            u32x4 a = {0, 0, 0, 0};
            u32 a4 = 0;
            u32x2 cc = {0, 0};
            if (mine) {
                a = __builtin_amdgcn_raw_buffer_load_b128(ws, fx, 0, AUX_SYS);
                a4 = __builtin_amdgcn_raw_buffer_load_b32(ws, fx + 16, 0, AUX_SYS);
                cc = __builtin_amdgcn_raw_buffer_load_b64(ws, fw, 0, AUX_SYS);
            }
            const bool ready = mine && a.x >= call && a.y >= call && a.z >= call && a.w >= call && (a4 >= call || !five) && cc.x >= call && cc.y >= call;
            u64 mask = __ballot(ready);
            if (!mask) {
                if ((++spins & 15u) == 0u && give_up(tmo, call, t0)) mask = pending;
                else { __builtin_amdgcn_s_sleep(1); continue; }
            }
            if ((mask >> lane) & 1ull) {
                const u32 pos = n + __builtin_amdgcn_mbcnt_hi((u32)(mask >> 32), __builtin_amdgcn_mbcnt_lo((u32)mask, 0u));
                *(LAS u32*)(lds + L_ORD + pos * 4) = lane;
            }
            LGKM0();
            n += __builtin_popcountll(mask);
            pending &= ~mask;
            if (lane == 0) lds_st(lds, L_OCNT, n);
        }
        const u32 row0 = 320u * mt;
        const u32 nrow8 = mt == MT - 1 ? 32u : 40u;
        for (u32 gq = 0; gq < 40u; ++gq) {
            const u32 i = gq >> 3, g = (gq & 1u) * 20u + 4u * i + ((gq >> 1) & 3u);
            if ((gq & 7u) == 0u) lds_wait_ge(lds, L_OUT + i * 4u, 4u);
            if (g >= nrow8) continue;
            const u32 row = 8 * g + (lane >> 3), ch = lane & 7;
            const u32x4 v = *(const LAS u32x4*)(lds + row * 128 + ((ch ^ (row & 7)) * 16));
            const u32 grow = row0 + 8 * g;
            const rsrc_t ob = mkr(P.out[(grow >> 9) & 7u], OUT_BYTES);
            st128(v, ob, ((grow & 511u) + (lane >> 3)) * NN + rank * NL + nt * 128 + ch * 16);
            asm volatile("s_waitcnt vmcnt(" STR(OUTWIN1) ")" ::: "memory");
        }
        VM0();
        const u32 d0 = row0 >> 9, d1 = (row0 + 8u * nrow8 - 1u) >> 9;
        if (lane <= d1 - d0) {
            const u32 d = (d0 + lane) & 7u;
            __hip_atomic_store((GAS u32*)(P.ws[d] + DON_OFF + ((rank * 3u + (mt - (512u * d) / 320u)) * 16u + nt) * 128u), call, __ATOMIC_RELAXED, __HIP_MEMORY_SCOPE_SYSTEM);
        }
        const u32 cnt = (512u * rank + 511u) / 320u - (512u * rank) / 320u + 1u;
        for (u32 e = gi; e < 8u * cnt * 16u; e += NGEMM) {
            GAS u32* f = (GAS u32*)(myws + DON_OFF + (((e / (cnt * 16u)) * 3u + (e / 16u) % cnt) * 16u + e % 16u) * 128u);
            const u64 t1 = rt();
            for (u32 sp = 1; rfl(__hip_atomic_load(f, __ATOMIC_RELAXED, __HIP_MEMORY_SCOPE_SYSTEM)) < call; ++sp) {
                if ((sp & 15u) == 0u && give_up(tmo, call, t1)) {
                    if (lane == 0) __hip_atomic_store((GAS unsigned char*)(P.out[rank] + rank * NL), (unsigned char)0x80, __ATOMIC_RELAXED, __HIP_MEMORY_SCOPE_SYSTEM);
                    break;
                }
                __builtin_amdgcn_s_sleep(1);
            }
        }
        return;
    }
    if (wave >= 4) {
        const u32 half = wave - 4;
        const char* xsrc = myws + RAW_OFF + XS_REL + (10u * mt) * BLK + half * 2048u + lane * 16u;
        const char* wsrc = myws + RAW_OFF + WS_REL + (4u * nt) * BLK + half * 2048u + lane * 16u;
        const u32 lbase = (u32)(uintptr_t)lds;
        auto rd = [&](u32 off) {
            u32 v;
            asm volatile("ds_read_b32 %0, %1\n\ts_waitcnt lgkmcnt(0)" : "=v"(v) : "v"(lbase + off) : "memory");
            return rfl(v);
        };
        u32 ni = 0, islot = half, iuse = 0;
        u32 np = 0, pslot = half;
        while (np < 64) {
            bool can = ni < 64 && ni - np < 2u && rd(L_OCNT) > ni;
            if (can) can = rd(L_FREE + islot * 4) >= 4u * iuse;
            if (can) {
                const u32 s_ = rd(L_ORD + ni * 4) & 63u;
                const char* gx = xsrc + (size_t)s_ * (128u * BLK);
                const char* gw = wsrc + (size_t)s_ * (64u * BLK);
                LAS char* sb = lds + islot * SLOT;
#define DMA2(g, l) do { \
                __builtin_amdgcn_global_load_lds((const GAS u32*)(g), (LAS u32*)(l), 16, 0, AUX_SYS); \
                __builtin_amdgcn_global_load_lds((const GAS u32*)(g), (LAS u32*)(l), 16, 1024, AUX_SYS); } while (0)
#pragma unroll
                for (int i = 0; i < 10; ++i) DMA2(gx + i * BLK, sb + i * 2048);
#pragma unroll
                for (int i = 0; i < 4; ++i) DMA2(gw + i * BLK, sb + (10 + i) * 2048);
                ++ni;
                islot += 2; if (islot >= NSLOT) { islot -= NSLOT; ++iuse; }
            } else if (np < ni) {
                if (ni - np == 2) asm volatile("s_waitcnt vmcnt(28)" ::: "memory"); else VM0();
                if (lane == 0) asm volatile("ds_add_u32 %0, %1" :: "v"(lbase + L_FULL + pslot * 4), "v"(1u) : "memory");
                ++np;
                pslot += 2; if (pslot >= NSLOT) pslot -= NSLOT;
            } else {
                __builtin_amdgcn_s_sleep(1);
            }
        }
        return;
    }

    {
        const rsrc_t xin = mkr(P.x, X_BYTES), win = mkr(P.w, W_BYTES), ws = mkr(myws, WS_USED);
        const u32 xv = (lane >> 3) * 1024u + (lane & 7u) * 16u;
        const u32 c16 = lane & 15u;
        const u32 wv = (lane >> 5) * 262144u + ((lane & 16u) + 8u * ((c16 >> 2) & 1u) + 4u * (c16 >> 3) + (c16 & 3u)) * 4u;
        const u32 sv = lane * 16u;
        const u32 fv = lane == 0 ? 0u : OOB;
        for (u32 e = wave * NGEMM + gi; wave < 3u && e < 2560u; e += 3u * NGEMM) {
            const u32 j = e / 320u, r = e % 320u;
            if (e < 3u * NGEMM) for (u32 i = 0; i < j; ++i) __builtin_amdgcn_s_sleep(STAGGER);
            const bool isw = r < 256u;
            const u32 d = isw ? (r & 7u) : rank, ti = isw ? (r >> 3) : r - 256u;
            if (isw && j == 0 && ti < 3u * KPUSH && d != rank) continue;
            u32 V[32];
            if (isw) {
                u32 R[32];
                const u32 base = (64u * j * 16384u + 2048u * d + 64u * ti) * 2u;
#pragma unroll
                for (int i = 0; i < 32; ++i)
                    R[i] = __builtin_amdgcn_raw_buffer_load_b32(win, wv, base + ((i >> 3) * 16u + (i & 7)) * 32768u, 0);
#pragma unroll
                for (int g = 0; g < 4; ++g)
#pragma unroll
                    for (int q = 0; q < 4; ++q) {
                        V[8 * g + q] = __builtin_amdgcn_perm(R[8 * g + 2 * q + 1], R[8 * g + 2 * q], 0x05040100u);
                        V[8 * g + 4 + q] = __builtin_amdgcn_perm(R[8 * g + 2 * q + 1], R[8 * g + 2 * q], 0x07060302u);
                    }
            } else {
                const u32 base = ti * 65536u + j * 128u;
#pragma unroll
                for (int i = 0; i < 8; ++i) {
                    const u32x4 v = __builtin_amdgcn_raw_buffer_load_b128(xin, xv, base + i * 8192u, 0);
                    V[4 * i] = v.x; V[4 * i + 1] = v.y; V[4 * i + 2] = v.z; V[4 * i + 3] = v.w;
                }
            }
            const u32 st = rank * 8u + j;
            if (d == rank) {
                const u32 rel = isw ? WS_REL + (st * 64u + 2u * ti) * BLK : XS_REL + (st * 128u + 2u * ti) * BLK;
                const u32 lo = raw_lane_off(isw, lane) + RAW_OFF + rel;
#pragma unroll
                for (int i = 0; i < 8; ++i) { const u32x4 v = {V[4 * i], V[4 * i + 1], V[4 * i + 2], V[4 * i + 3]}; st128(v, ws, lo + raw_vec_off(isw, i)); }
            }
            u32 meta = 0;
            const u32 pk = isw ? 512u + (d * 8u + j) * 32u + ti : j * 64u + ti;
            const bool pack = !isw || d != rank;
            if (pack) {
                u32x4 O[8];
                const u32 slot = PCK_OFF + pk * 8192u, so = sv + slot;
                u32x2 S; u32 list;
                meta = encode_tight(V, O, S, list, lds + L_SCR + wave * SCR_BYTES, lane);
                if (meta) {
#pragma unroll
                    for (int i = 0; i < 5; ++i) st128(O[i], ws, so + i * 1024u);
                    __builtin_amdgcn_raw_buffer_store_b64(S, ws, lane * 8u + SGN_REL, slot, AUX_SYS);
                    __builtin_amdgcn_raw_buffer_store_b32(list, ws, lane * 4u + LST_REL, slot, AUX_SYS);
                } else {
                    meta = encode_task(V, O, ws, slot, lane);
#pragma unroll
                    for (int i = 0; i < 6; ++i) st128(O[i], ws, so + i * 1024u);
                    if (meta & META_RAW) { st128(O[6], ws, so + 6144u); st128(O[7], ws, so + 7168u); }
                }
            }
            VM0();
            if (pack) { u32x2 f; f.x = meta; f.y = call; __builtin_amdgcn_raw_buffer_store_b64(f, ws, fv, PRD_OFF + pk * 64u, AUX_SYS); }
            if (d == rank) {
                const u32 rdy = isw ? RDW_OFF + ((ti >> 1) * 64u + st) * 8u + (ti & 1u) * 4u : RDX_OFF + ((ti / 5u) * 64u + st) * 32u + (ti % 5u) * 4u;
                __builtin_amdgcn_raw_buffer_store_b32(call, ws, fv, rdy, AUX_SYS);
            }
        }
    }
    const u32 wm = wave >> 1, wn = wave & 1;
    f32x16 acc[2][5];
#pragma unroll
    for (int f = 0; f < 2; ++f)
#pragma unroll
        for (int i = 0; i < 5; ++i)
#pragma unroll
            for (int r = 0; r < 16; ++r) acc[f][i][r] = 0.f;
    {
        const LAS char* xb0 = lds + (5 * wm) * 2048 + lane * 16;
        const LAS char* wb0 = lds + (10 + 2 * wn) * 2048 + lane * 16;
        u32x4 F0[7], F1[7];
#define MM(F) _Pragma("unroll") for (int i = 0; i < 5; ++i) _Pragma("unroll") for (int f = 0; f < 2; ++f) \
            acc[f][i] = __builtin_amdgcn_mfma_f32_32x32x16_bf16(__builtin_bit_cast(bf16x8, F[5 + f]), __builtin_bit_cast(bf16x8, F[i]), acc[f][i], 0, 0, 0)
#define SB() __builtin_amdgcn_sched_barrier(0)
#define MF(F, i, f) acc[f][i] = __builtin_amdgcn_mfma_f32_32x32x16_bf16(__builtin_bit_cast(bf16x8, F[5 + f]), __builtin_bit_cast(bf16x8, F[i]), acc[f][i], 0, 0, 0)
#define RX(F, i, sl, t) F[i] = *(const LAS u32x4*)(xb0 + (sl) + (i) * 2048 + (t) * 1024)
#define RW(F, f, sl, t) F[5 + f] = *(const LAS u32x4*)(wb0 + (sl) + (f) * 2048 + (t) * 1024)
        const u32 lbase = (u32)(uintptr_t)lds;
        u32 slot = 0, use = 1;
        lds_wait_ge(lds, L_FULL, 1u);
        SB();
        RW(F0, 0, 0u, 0); SB(); RX(F0, 0, 0u, 0); SB(); RW(F0, 1, 0u, 0); SB(); RX(F0, 1, 0u, 0); SB();
        RX(F0, 2, 0u, 0); SB(); RX(F0, 3, 0u, 0); SB(); RX(F0, 4, 0u, 0); SB();
        u32 cur = 0, nxt = 1;
        for (u32 h = 0; h < 127; ++h) {
            const u32 co = cur * SLOT, no = nxt * SLOT;
            SB();
            MF(F0, 0, 0); const u32 fl = lds_ld(lds, L_FULL + nxt * 4); SB();
            MF(F0, 0, 1); RW(F1, 0, co, 1); SB();
            MF(F0, 1, 0); RX(F1, 0, co, 1); SB();
            MF(F0, 1, 1); RW(F1, 1, co, 1); SB();
            MF(F0, 2, 0); RX(F1, 1, co, 1); SB();
            MF(F0, 2, 1); RX(F1, 2, co, 1); SB();
            MF(F0, 3, 0); RX(F1, 3, co, 1); SB();
            MF(F0, 3, 1); RX(F1, 4, co, 1); SB();
            MF(F0, 4, 0); SB();
            MF(F0, 4, 1); SB();
            MF(F1, 0, 0); SB();
            MF(F1, 0, 1); SB();
            if (rfl(fl) < use) lds_wait_ge(lds, L_FULL + nxt * 4, use);
            asm volatile("" ::: "memory");
            SB();
            MF(F1, 1, 0); RW(F0, 0, no, 0); SB();
            MF(F1, 1, 1); RX(F0, 0, no, 0); SB();
            MF(F1, 2, 0); RW(F0, 1, no, 0); SB();
            {
                u64 sv_;
                asm volatile("s_waitcnt lgkmcnt(3)\n\ts_mov_b64 %0, exec\n\ts_mov_b64 exec, 1\n\tds_add_u32 %1, %2\n\ts_mov_b64 exec, %0"
                             : "=&s"(sv_) : "v"(lbase + L_FREE + cur * 4), "v"(1u) : "memory");
            }
            SB();
            MF(F1, 2, 1); RX(F0, 1, no, 0); SB();
            MF(F1, 3, 0); RX(F0, 2, no, 0); SB();
            MF(F1, 3, 1); RX(F0, 3, no, 0); SB();
            MF(F1, 4, 0); RX(F0, 4, no, 0);
            cur = nxt;
            if (++nxt == NSLOT) { nxt = 0; ++use; }
            SB();
            MF(F1, 4, 1); SB();
        }
        slot = cur;
        const u32 slot_last = slot * SLOT;
        RW(F1, 0, slot_last, 1); RW(F1, 1, slot_last, 1);
#pragma unroll
        for (int i = 0; i < 5; ++i) RX(F1, i, slot_last, 1);
        MM(F0);
        MM(F1);
    }
    const u32 ngrp = (mt == MT - 1 && wm == 1) ? 3u : 5u;
    float mx = 0.f;
#pragma unroll
    for (int i = 0; i < 5; ++i) {
        float m = 0.f;
#pragma unroll
        for (int f = 0; f < 2; ++f)
#pragma unroll
            for (int r = 0; r < 16; ++r) m = fmaxf(m, fabsf(acc[f][i][r]));
        mx = (u32)i < ngrp ? fmaxf(mx, m) : mx;
    }
    const u32 mb = wave_max_u32(__float_as_uint(mx));
    u32 arrived = 0;
    if (lane == 0) {
        (void)__hip_atomic_fetch_max((LAS u32*)(lds + L_WMAX), mb, __ATOMIC_RELAXED, __HIP_MEMORY_SCOPE_WORKGROUP);
        arrived = __hip_atomic_fetch_add((LAS u32*)(lds + L_WCNT), 1u, __ATOMIC_RELAXED, __HIP_MEMORY_SCOPE_WORKGROUP);
    }
    arrived = rfl(arrived);
    if (arrived == 3) {
        const u32 wmax = rfl(lds_ld(lds, L_WMAX));
        if (lane == 0)
            __hip_atomic_store((GAS u64*)(myws + LMX_OFF + gi * 8), ((u64)call << 32) | wmax, __ATOMIC_RELAXED, __HIP_MEMORY_SCOPE_AGENT);
    }
    if (wave == 0) {
        GAS u64* g = (GAS u64*)(myws + AMX_OFF + xcd * 128 + (lane & 7) * 8);
        const u64 t0 = rt();
        u32 gm = 0, bad = 0;
        for (u32 spins = 1;; ++spins) {
            const u64 v = __hip_atomic_load(g, __ATOMIC_RELAXED, __HIP_MEMORY_SCOPE_SYSTEM);
            const u32 tm = __hip_atomic_load(tmo, __ATOMIC_RELAXED, __HIP_MEMORY_SCOPE_AGENT);
            bad = rfl(tm) == call;
            if (bad || __all((u32)(v >> 32) >= call)) { gm = (u32)v; break; }
            if ((spins & 15u) == 0u && give_up(tmo, call, t0)) { bad = 1; break; }
            __builtin_amdgcn_s_sleep(1);
        }
        gm = wave_max_u32(gm);
        if (lane == 0) { lds_st(lds, L_GMAX, gm); lds_st(lds, L_BAD, bad); }
        LGKM0();
        if (lane == 0) lds_st(lds, L_GOK, 1u);
        if (gi == 0 && lane == 0) __hip_atomic_store(ctr, call, __ATOMIC_RELAXED, __HIP_MEMORY_SCOPE_AGENT);
    }
    lds_wait_ge(lds, L_GOK, 1u);
    const float amax = __uint_as_float(rfl(lds_ld(lds, L_GMAX)));
    const bool poison = rfl(lds_ld(lds, L_BAD)) != 0u;
    {
        const float scale = amax / 127.0f;
        const float inv = scale > 0.f ? 1.0f / scale : 0.f;
        const f32x2 inv2 = {inv, inv}, magic2 = {12582912.f + 128.f, 12582912.f + 128.f};
#pragma unroll
        for (int i = 0; i < 5; ++i) {
#pragma unroll
            for (int f = 0; f < 2; ++f) {
                const u32 ch = 4 * wn + 2 * f + (lane >> 5);
                u32x4 v;
#pragma unroll
                for (int q = 0; q < 4; ++q) {
                    constexpr float MAGIC = 12582912.f + 128.f;
                    const f32x2 lo2 = {acc[f][i][2 * q], acc[f][i][2 * q + 1]}, hi2 = {acc[f][i][2 * q + 8], acc[f][i][2 * q + 9]};
                    const f32x2 tl = __builtin_elementwise_fma(lo2, inv2, magic2), th = __builtin_elementwise_fma(hi2, inv2, magic2);
                    const u32 b0 = __float_as_uint(__builtin_amdgcn_fmed3f(tl.x, MAGIC - 127.f, MAGIC + 127.f));
                    const u32 b1 = __float_as_uint(__builtin_amdgcn_fmed3f(th.x, MAGIC - 127.f, MAGIC + 127.f));
                    const u32 b2 = __float_as_uint(__builtin_amdgcn_fmed3f(tl.y, MAGIC - 127.f, MAGIC + 127.f));
                    const u32 b3 = __float_as_uint(__builtin_amdgcn_fmed3f(th.y, MAGIC - 127.f, MAGIC + 127.f));
                    const u32 d = __builtin_amdgcn_perm(__builtin_amdgcn_perm(b3, b2, 0x0c0c0400u), __builtin_amdgcn_perm(b1, b0, 0x0c0c0400u), 0x05040100u) ^ 0x80808080u;
                    v[q] = poison ? 0x80808080u : d;
                }
                const u32 row = 160 * wm + 32 * i + (lane & 31);
                *(LAS u32x4*)(lds + row * 128 + ((ch ^ (row & 7)) * 16)) = v;
            }
            LGKM0();
            if (lane == 0) lds_add(lds, L_OUT + i * 4, 1u);
            __builtin_amdgcn_sched_barrier(0);
        }
    }
}

extern "C" void kernel_launch(int rank, int nranks, ncclComm_t comm, hipStream_t stream,
                              void* const* d_in, const int* in_sizes, int n_in,
                              void* d_out, int out_size, const int* in_dtypes, int out_dtype,
                              void* const* d_out_all, void* const* d_ws_all, size_t ws_bytes) {
    (void)comm; (void)in_dtypes; (void)out_dtype; (void)d_out;
    if (nranks != NR || (unsigned)rank >= (unsigned)NR || n_in != 2 || in_sizes[0] != MM * KSH || in_sizes[1] != KSH * NN ||
        out_size != (int)OUT_BYTES || ws_bytes < WS_USED) {
        fprintf(stderr, "kernel_launch: unexpected shapes\n");
        return;
    }
    Params P;
    P.x = d_in[0];
    P.w = d_in[1];
    for (int p = 0; p < NR; ++p) { P.ws[p] = (char*)d_ws_all[p]; P.out[p] = (char*)d_out_all[p]; }
    P.rank = rank;
    P.pad = 0;
    hipLaunchKernelGGL(gemm_ag_q8, dim3(256), dim3(512), 0, stream, P);
}
```

```cpp
#include <hip/hip_runtime.h>
#include <rccl/rccl.h>
#include <cstdio>
#include <cstdint>

typedef unsigned u32;
typedef unsigned long long u64;
typedef u32 u32x4 __attribute__((ext_vector_type(4)));
typedef u32 u32x2 __attribute__((ext_vector_type(2)));
typedef float f32x16 __attribute__((ext_vector_type(16)));
typedef float f32x2 __attribute__((ext_vector_type(2)));
typedef __bf16 bf16x8 __attribute__((ext_vector_type(8)));
typedef __amdgpu_buffer_rsrc_t rsrc_t;
#define LAS __attribute__((address_space(3)))
#define GAS __attribute__((address_space(1)))

constexpr int NR = 8;
constexpr int MM = 4096;
constexpr int KSH = 512;
constexpr int NN = 16384;
constexpr int NL = NN / NR;
constexpr u32 X_BYTES = MM * KSH * 2;
constexpr u32 W_BYTES = KSH * NN * 2;
constexpr u32 OUT_BYTES = (MM / NR) * NN;
constexpr u32 BLK = 4096;
constexpr u32 XS_REL = 0;
constexpr u32 WS_REL = 64u * 128u * BLK;
constexpr u32 STG_BYTES = WS_REL + 64u * 64u * BLK;
constexpr u32 PST_OFF = 0;
constexpr u32 RAW_OFF = STG_BYTES;
constexpr u32 NTASK = 768;
constexpr u32 PCK_OFF = 2u * STG_BYTES;
constexpr u32 PRD_OFF = PCK_OFF + 2560u * 8192u;
constexpr u32 ARR_OFF = PRD_OFF + 2560u * 64u;
constexpr u32 MT = 13;
constexpr u32 NGEMM = MT * 16;
constexpr u32 NCOMM = 256 - NGEMM;
constexpr u32 RDX_OFF = ARR_OFF + 8u * NTASK * 64u;
constexpr u32 RDW_OFF = RDX_OFF + MT * 64u * 32u;
constexpr u32 LMX_OFF = RDW_OFF + 16u * 64u * 8u;
constexpr u32 AMX_OFF = LMX_OFF + 256u * 8u;
constexpr u32 DON_OFF = AMX_OFF + 8u * 128u;
constexpr u32 QUE_OFF = DON_OFF + 384u * 128u;
constexpr u32 CTR_OFF = QUE_OFF + 8u * 128u;
constexpr u32 TMO_OFF = CTR_OFF + 128u;
constexpr u32 WS_USED = CTR_OFF + 4096u;
constexpr int AUX_SYS = 17;
constexpr u64 DEADLINE = 100000000ull;
constexpr u32 OOB = 0x40000000u;

constexpr u32 NDECWG = 16;
constexpr u32 NPUSH = 6;
constexpr u32 KPUSH = 27;
#define PB 3
#define PB2 5
constexpr u32 NRAW = 48;
#define STAGGER 40
#define OUTWIN1 1
static_assert(7 * KPUSH <= (NCOMM - NDECWG) * NPUSH && NPUSH < 8, "pushers");
static_assert(NRAW <= 64, "the unpacked tail consists of x tasks");

constexpr u32 SLOT = 14 * 2048;
constexpr u32 NSLOT = 5;
constexpr u32 L_CTL = NSLOT * SLOT;
constexpr u32 L_FULL = L_CTL;
constexpr u32 L_FREE = L_CTL + 32;
constexpr u32 L_OCNT = L_CTL + 64;
constexpr u32 L_WMAX = L_CTL + 68;
constexpr u32 L_WCNT = L_CTL + 72;
constexpr u32 L_GMAX = L_CTL + 76;
constexpr u32 L_BAD = L_CTL + 80;
constexpr u32 L_GOK = L_CTL + 84;
constexpr u32 L_OUT = L_CTL + 96;
constexpr u32 L_ORD = L_CTL + 128;
constexpr u32 L_SCR = L_ORD + 256;
constexpr u32 LDS_BYTES = L_SCR + 4 * 4352;

struct Params {
    const void* x;
    const void* w;
    char* ws[NR];
    char* out[NR];
    int rank;
    int pad;
};

__device__ __forceinline__ rsrc_t mkr(const void* p, u32 bytes) {
    return __builtin_amdgcn_make_buffer_rsrc((void*)p, 0, bytes, 0x00020000);
}
__device__ __forceinline__ u64 rt() { return __builtin_amdgcn_s_memrealtime(); }
__device__ __forceinline__ u32 rfl(u32 v) { return __builtin_amdgcn_readfirstlane(v); }
__device__ __forceinline__ void st128(u32x4 v, rsrc_t r, u32 off) { __builtin_amdgcn_raw_buffer_store_b128(v, r, off, 0, 17); }
#define STR2(x) #x
#define STR(x) STR2(x)
#define LGKM0() asm volatile("s_waitcnt lgkmcnt(0)" ::: "memory")
#define VM0() asm volatile("s_waitcnt vmcnt(0)" ::: "memory")

__device__ __forceinline__ u32 lds_ld(const LAS char* l, u32 off) {
    return __hip_atomic_load((const LAS u32*)(l + off), __ATOMIC_RELAXED, __HIP_MEMORY_SCOPE_WORKGROUP);
}
__device__ __forceinline__ void lds_st(LAS char* l, u32 off, u32 v) {
    __hip_atomic_store((LAS u32*)(l + off), v, __ATOMIC_RELAXED, __HIP_MEMORY_SCOPE_WORKGROUP);
}
__device__ __forceinline__ void lds_add(LAS char* l, u32 off, u32 v) {
    (void)__hip_atomic_fetch_add((LAS u32*)(l + off), v, __ATOMIC_RELAXED, __HIP_MEMORY_SCOPE_WORKGROUP);
}
__device__ __forceinline__ void lds_wait_ge(const LAS char* l, u32 off, u32 want) {
    while (rfl(lds_ld(l, off)) < want) __builtin_amdgcn_s_sleep(1);
    asm volatile("" ::: "memory");
}
__device__ __forceinline__ bool give_up(GAS u32* tmo, u32 call, u64 t0) {
    if (rfl(__hip_atomic_load(tmo, __ATOMIC_RELAXED, __HIP_MEMORY_SCOPE_AGENT)) == call || rt() - t0 > DEADLINE) {
        __hip_atomic_store(tmo, call, __ATOMIC_RELAXED, __HIP_MEMORY_SCOPE_AGENT);
        return true;
    }
    return false;
}
__device__ __forceinline__ u32 wave_max_u32(u32 v) {
    u32 o;
    o = __builtin_amdgcn_update_dpp(0u, v, 0x111, 0xf, 0xf, false); v = v > o ? v : o;
    o = __builtin_amdgcn_update_dpp(0u, v, 0x112, 0xf, 0xf, false); v = v > o ? v : o;
    o = __builtin_amdgcn_update_dpp(0u, v, 0x114, 0xf, 0xf, false); v = v > o ? v : o;
    o = __builtin_amdgcn_update_dpp(0u, v, 0x118, 0xf, 0xf, false); v = v > o ? v : o;
    o = __builtin_amdgcn_update_dpp(0u, v, 0x142, 0xa, 0xf, false); v = v > o ? v : o;
    o = __builtin_amdgcn_update_dpp(0u, v, 0x143, 0xc, 0xf, false); v = v > o ? v : o;
    return __builtin_amdgcn_readlane(v, 63);
}

constexpr u32 EXC_CAP = 8;
constexpr u32 META_RAW = 0x800u;
constexpr u32 EXC_REL = 6144;
__device__ __forceinline__ u32 pkmax_u16(u32 a, u32 b) {
    typedef unsigned short u16x2 __attribute__((ext_vector_type(2)));
    return __builtin_bit_cast(u32, __builtin_elementwise_max(__builtin_bit_cast(u16x2, a), __builtin_bit_cast(u16x2, b)));
}
__device__ __forceinline__ u32 encode_task(const u32* V, u32x4* O, rsrc_t dst, u32 slot_off, u32 lane) {
    u32 m = V[0] & 0x7fff7fffu;
#pragma unroll
    for (int i = 1; i < 32; ++i) m = pkmax_u16(m, V[i] & 0x7fff7fffu);
    m = pkmax_u16(m, m >> 16) & 0xffffu;
    const u32 top = wave_max_u32(m) >> 8;
    const u32 base = top > 7u ? top - 7u : 0u;
    const u32 b4 = base * 0x01010101u;
    u32 bad = 0;
    u32 Lp[16], X[8];
#pragma unroll
    for (int t = 0; t < 8; ++t) {
        const u32 h01 = __builtin_amdgcn_perm(V[4 * t + 1], V[4 * t], 0x07050301u), h23 = __builtin_amdgcn_perm(V[4 * t + 3], V[4 * t + 2], 0x07050301u);
        Lp[2 * t] = __builtin_amdgcn_perm(V[4 * t + 1], V[4 * t], 0x06040200u);
        Lp[2 * t + 1] = __builtin_amdgcn_perm(V[4 * t + 3], V[4 * t + 2], 0x06040200u);
        const u32 u01 = (h01 | 0x80808080u) - b4, u23 = (h23 | 0x80808080u) - b4;
        bad |= (u01 ^ 0x80808080u) | (u23 ^ 0x80808080u);
        X[t] = (u01 & 0x07070707u) | ((h01 >> 4) & 0x08080808u) | ((u23 & 0x07070707u) << 4) | (h23 & 0x80808080u);
    }
    u32 meta = base;
    if (__any((bad & 0xf8f8f8f8u) != 0u)) {
        u64 bm = 0;
#pragma unroll
        for (int i = 0; i < 64; ++i) {
            const u32 h7 = (V[i >> 1] >> (8 + 16 * (i & 1))) & 0x7fu;
            bm |= (u64)(h7 - base > 7u ? 1u : 0u) << i;
        }
        u32 n = 0;
        for (;;) {
            const u64 act = __ballot(bm != 0ull);
            if (!act) break;
            const u32 cnt = __builtin_popcountll(act);
            if (n + cnt > EXC_CAP) { n = EXC_CAP + 1u; break; }
            const u32 pos = n + __builtin_amdgcn_mbcnt_hi((u32)(act >> 32), __builtin_amdgcn_mbcnt_lo((u32)act, 0u));
            const u32 i = bm ? __builtin_ctzll(bm) : 0u;
            u32 dw = V[0];
#pragma unroll
            for (int q = 1; q < 32; ++q) dw = (i >> 1) == (u32)q ? V[q] : dw;
            const u32 val = (dw >> (16u * (i & 1u))) & 0xffffu;
            const u32 ent = (lane << 22) | (i << 16) | val;
            __builtin_amdgcn_raw_buffer_store_b32(ent, dst, bm ? EXC_REL + pos * 4u : OOB, slot_off, AUX_SYS);
            bm &= bm - 1ull;
            n += cnt;
        }
        meta = n > EXC_CAP ? META_RAW : (base | (n << 7));
    }
    if (meta & META_RAW) {
#pragma unroll
        for (int t = 0; t < 8; ++t) { O[t].x = V[4 * t]; O[t].y = V[4 * t + 1]; O[t].z = V[4 * t + 2]; O[t].w = V[4 * t + 3]; }
    } else {
#pragma unroll
        for (int t = 0; t < 4; ++t) { O[t].x = Lp[4 * t]; O[t].y = Lp[4 * t + 1]; O[t].z = Lp[4 * t + 2]; O[t].w = Lp[4 * t + 3]; }
#pragma unroll
        for (int t = 0; t < 2; ++t) { O[4 + t].x = X[4 * t]; O[4 + t].y = X[4 * t + 1]; O[4 + t].z = X[4 * t + 2]; O[4 + t].w = X[4 * t + 3]; }
    }
    return meta;
}
__device__ __forceinline__ void decode_task(u32x4* B, u32 meta, u32 exc, u32 lane) {
    const u32 b4 = (meta & 0x7fu) * 0x01010101u;
    u32 V[32];
#pragma unroll
    for (int t = 0; t < 8; ++t) {
        const u32 x = B[4 + (t >> 2)][t & 3], l01 = B[t >> 1][2 * (t & 1)], l23 = B[t >> 1][2 * (t & 1) + 1];
        const u32 he = ((x & 0x07070707u) + b4) | ((x << 4) & 0x80808080u);
        const u32 ho = (((x >> 4) & 0x07070707u) + b4) | (x & 0x80808080u);
        V[4 * t + 0] = __builtin_amdgcn_perm(he, l01, 0x05010400u);
        V[4 * t + 1] = __builtin_amdgcn_perm(he, l01, 0x07030602u);
        V[4 * t + 2] = __builtin_amdgcn_perm(ho, l23, 0x05010400u);
        V[4 * t + 3] = __builtin_amdgcn_perm(ho, l23, 0x07030602u);
    }
    const u32 cnt = (meta >> 7) & 15u;
    for (u32 e = 0; e < cnt; ++e) {
        const u32 ent = __builtin_amdgcn_readlane(exc, e);
        const u32 i = (ent >> 16) & 63u, sh = 16u * (i & 1u);
        const bool me = lane == (ent >> 22);
        const u32 keep = ~(0xffffu << sh), put = (ent & 0xffffu) << sh;
#pragma unroll
        for (int q = 0; q < 32; ++q) V[q] = (me && (i >> 1) == (u32)q) ? ((V[q] & keep) | put) : V[q];
    }
#pragma unroll
    for (int t = 0; t < 8; ++t) { B[t].x = V[4 * t]; B[t].y = V[4 * t + 1]; B[t].z = V[4 * t + 2]; B[t].w = V[4 * t + 3]; }
}

constexpr u32 META_TIGHT = 0x10000u;
constexpr u32 SGN_REL = 5120, LST_REL = 5632;
constexpr u32 SCR_BYTES = 4352;
__device__ __forceinline__ u32 wave_scan_incl(u32 v) {
    v += __builtin_amdgcn_update_dpp(0u, v, 0x111, 0xf, 0xf, false);
    v += __builtin_amdgcn_update_dpp(0u, v, 0x112, 0xf, 0xf, false);
    v += __builtin_amdgcn_update_dpp(0u, v, 0x114, 0xf, 0xf, false);
    v += __builtin_amdgcn_update_dpp(0u, v, 0x118, 0xf, 0xf, false);
    v += __builtin_amdgcn_update_dpp(0u, v, 0x142, 0xa, 0xf, false);
    v += __builtin_amdgcn_update_dpp(0u, v, 0x143, 0xc, 0xf, false);
    return v;
}
__device__ __forceinline__ u32 encode_tight(const u32* V, u32x4* O, u32x2& S, u32& list, LAS char* sc, u32 lane) {
    u32 m = V[0] & 0x7fff7fffu;
#pragma unroll
    for (int i = 1; i < 32; ++i) m = pkmax_u16(m, V[i] & 0x7fff7fffu);
    m = pkmax_u16(m, m >> 16) & 0xffffu;
    const u32 top = wave_max_u32(m) >> 8;
    const u32 base = top > 3u ? top - 3u : 0u;
    const u32 b4 = base * 0x01010101u;
    u32 U[16], C[4] = {0, 0, 0, 0}, Sg[2] = {0, 0}, E[2] = {0, 0};
#pragma unroll
    for (int k = 0; k < 16; ++k) {
        const int t = k >> 1, o = 2 * (k & 1);
        const u32 h = __builtin_amdgcn_perm(V[4 * t + o + 1], V[4 * t + o], 0x07050301u);
        O[k >> 2][k & 3] = __builtin_amdgcn_perm(V[4 * t + o + 1], V[4 * t + o], 0x06040200u);
        const u32 u = (h | 0x80808080u) - b4;
        const u32 e = (u ^ 0x80808080u) & 0xfcfcfcfcu;
        const u32 nz = (((e & 0x7c7c7c7cu) + 0x7c7c7c7cu) | e) & 0x80808080u;
        U[k] = u;
        C[k >> 2] |= (u & 0x03030303u) << (2 * (k & 3));
        Sg[k >> 3] |= (h & 0x80808080u) >> (7 - (k & 7));
        E[k >> 3] |= nz >> (7 - (k & 7));
    }
    const u32 n = __builtin_popcount(E[0]) + __builtin_popcount(E[1]);
    const u32 incl = wave_scan_incl(n);
    const u32 total = __builtin_amdgcn_readlane(incl, 63);
    if (total > 128u) return 0u;
    list = 0;
    if (total) {
#pragma unroll
        for (int j = 0; j < 4; ++j) { const u32x4 q = {U[4 * j], U[4 * j + 1], U[4 * j + 2], U[4 * j + 3]}; *(LAS u32x4*)(sc + j * 1024 + lane * 16) = q; }
        u32 pos = incl - n, e0 = E[0], e1 = E[1];
        bool far = false;
        while (__any((e0 | e1) != 0u)) {
            const bool has = (e0 | e1) != 0u;
            const u32 w = e0 ? 0u : 1u, mk = e0 ? e0 : e1;
            const u32 p = has ? __builtin_ctz(mk) : 0u;
            const u32 k = 8u * w + (p & 7u);
            const u32 ub = *(const LAS unsigned char*)(sc + (k >> 2) * 1024u + lane * 16u + (k & 3u) * 4u + (p >> 3));
            const u32 code = (0x7fu - ub) >> 2;
            if (has) {
                far |= code > 15u;
                *(LAS unsigned short*)(sc + 4096u + pos * 2u) = (unsigned short)((lane << 10) | ((w * 32u + p) << 4) | (code & 15u));
                ++pos;
            }
            if (e0) e0 &= e0 - 1u; else e1 &= e1 - 1u;
        }
        if (__any(far)) return 0u;
        list = *(const LAS u32*)(sc + 4096u + lane * 4u);
    }
    O[4].x = C[0]; O[4].y = C[1]; O[4].z = C[2]; O[4].w = C[3];
    S.x = Sg[0]; S.y = Sg[1];
    return base | META_TIGHT | (total << 20);
}
__device__ __forceinline__ void decode_tight(u32x4* B, u32x2 S, u32 list, u32 meta, LAS char* sc, u32 lane) {
    const u32 b4 = (meta & 0x7fu) * 0x01010101u;
    u32 H[16];
#pragma unroll
    for (int k = 0; k < 16; ++k)
        H[k] = (((B[4][k >> 2] >> (2 * (k & 3))) & 0x03030303u) + b4) | ((S[k >> 3] << (7 - (k & 7))) & 0x80808080u);
    const u32 total = (meta >> 20) & 0xffu;
    if (total) {
#pragma unroll
        for (int j = 0; j < 4; ++j) { const u32x4 q = {H[4 * j], H[4 * j + 1], H[4 * j + 2], H[4 * j + 3]}; *(LAS u32x4*)(sc + j * 1024 + lane * 16) = q; }
#pragma unroll
        for (int e = 0; e < 2; ++e) {
            const u32 ent = e ? list >> 16 : list & 0xffffu;
            const u32 p = (ent >> 4) & 31u, k = ((ent >> 9) & 1u) * 8u + (p & 7u);
            if (2u * lane + e < total)
                (void)__hip_atomic_fetch_sub((LAS u32*)(sc + (k >> 2) * 1024u + (ent >> 10) * 16u + (k & 3u) * 4u), (4u * ((ent & 15u) + 1u)) << (8u * (p >> 3)),
                                             __ATOMIC_RELAXED, __HIP_MEMORY_SCOPE_WORKGROUP);
        }
        asm volatile("" ::: "memory");
#pragma unroll
        for (int j = 0; j < 4; ++j) { const u32x4 q = *(const LAS u32x4*)(sc + j * 1024 + lane * 16); H[4 * j] = q.x; H[4 * j + 1] = q.y; H[4 * j + 2] = q.z; H[4 * j + 3] = q.w; }
    }
    u32 V[32];
#pragma unroll
    for (int t = 0; t < 8; ++t) {
        const u32 l01 = B[t >> 1][2 * (t & 1)], l23 = B[t >> 1][2 * (t & 1) + 1];
        V[4 * t + 0] = __builtin_amdgcn_perm(H[2 * t], l01, 0x05010400u);
        V[4 * t + 1] = __builtin_amdgcn_perm(H[2 * t], l01, 0x07030602u);
        V[4 * t + 2] = __builtin_amdgcn_perm(H[2 * t + 1], l23, 0x05010400u);
        V[4 * t + 3] = __builtin_amdgcn_perm(H[2 * t + 1], l23, 0x07030602u);
    }
#pragma unroll
    for (int t = 0; t < 8; ++t) { B[t].x = V[4 * t]; B[t].y = V[4 * t + 1]; B[t].z = V[4 * t + 2]; B[t].w = V[4 * t + 3]; }
}

__device__ __forceinline__ bool task_is_w(u32 u) { return (u % 96u) < 32u; }
__device__ __forceinline__ u32 task_xb(u32 src, u32 dst, u32 u) { return (u % 96u - 32u + ((src - dst) & 7u) * 8u) & 63u; }
__device__ __forceinline__ u32 task_slot(u32 src, u32 dst, u32 u) {
    const u32 st = src * 8u + u / 96u;
    return task_is_w(u) ? WS_REL + (st * 64u + 2u * (u % 96u)) * BLK : XS_REL + (st * 128u + 2u * task_xb(src, dst, u)) * BLK;
}
__device__ __forceinline__ u32 task_rdy(u32 src, u32 dst, u32 u) {
    const u32 st = src * 8u + u / 96u;
    if (task_is_w(u)) { const u32 wb = u % 96u; return RDW_OFF + ((wb >> 1) * 64u + st) * 8u + (wb & 1u) * 4u; }
    const u32 xb = task_xb(src, dst, u);
    return RDX_OFF + ((xb / 5u) * 64u + st) * 32u + (xb % 5u) * 4u;
}
__device__ __forceinline__ u32 task_pck(u32 src, u32 dst, u32 u) {
    const u32 j = u / 96u;
    return task_is_w(u) ? 512u + (dst * 8u + j) * 32u + u % 96u : j * 64u + task_xb(src, dst, u);
}
__device__ __forceinline__ u32 raw_lane_off(bool is_w, u32 lane) {
    return is_w ? ((lane >> 4) & 1u) * 4096u + (lane >> 5) * 512u + (lane & 15u) * 16u : (lane & 7u) * 512u + (lane >> 3) * 16u;
}
__device__ __forceinline__ u32 raw_vec_off(bool is_w, int v) { return is_w ? (v >> 1) * 1024u + (v & 1) * 256u : (v >> 2) * 4096u + (v & 3) * 128u; }

constexpr u32 NDEC = NGEMM + NDECWG * 8u;
__device__ __forceinline__ void decoder(char* myws, u32 rank, u32 call, u32 di, u32 lane, LAS char* sc) {
    const rsrc_t ws = mkr(myws, WS_USED);
    GAS u32* const tmo = (GAS u32*)(myws + TMO_OFF);
    const u32 sv = lane * 16u;
    const u32 fv = lane == 0 ? 0u : OOB;
    u32 owed = 0;
    constexpr u32 QE = 7u * (NTASK - NRAW);
    for (u32 q = di; q < QE;) {
        const u32 qn = q + NDEC;
        const u32 u = q / 7u, src = (rank + 1u + q % 7u) & 7u;
        GAS u64* arr = (GAS u64*)(myws + ARR_OFF + (src * NTASK + u) * 64u);
        const u64 t0 = rt();
        u64 f;
        for (u32 spins = 0;; ++spins) {
            f = __hip_atomic_load(arr, __ATOMIC_RELAXED, __HIP_MEMORY_SCOPE_SYSTEM);
            const u32 seen = rfl((u32)(f >> 32));
            if (owed) { VM0(); __builtin_amdgcn_raw_buffer_store_b32(call, ws, fv, owed, AUX_SYS); owed = 0; }
            if (seen >= call) break;
            if ((spins & 15u) == 15u && give_up(tmo, call, t0)) break;
            __builtin_amdgcn_s_sleep(1);
        }
        asm volatile("" ::: "memory");
        const u32 meta = rfl((u32)f);
        const u32 so = task_slot(src, rank, u);
        u32x4 B[8];
#pragma unroll
        for (int i = 0; i < 5; ++i) B[i] = __builtin_amdgcn_raw_buffer_load_b128(ws, sv + i * 1024, PST_OFF + so, AUX_SYS);
        if (meta & META_TIGHT) {
            const u32x2 S = __builtin_amdgcn_raw_buffer_load_b64(ws, lane * 8u + SGN_REL, PST_OFF + so, AUX_SYS);
            const u32 list = __builtin_amdgcn_raw_buffer_load_b32(ws, lane * 4u + LST_REL, PST_OFF + so, AUX_SYS);
            decode_tight(B, S, list, meta, sc, lane);
        } else {
            B[5] = __builtin_amdgcn_raw_buffer_load_b128(ws, sv + 5120, PST_OFF + so, AUX_SYS);
            if (meta & META_RAW) {
                B[6] = __builtin_amdgcn_raw_buffer_load_b128(ws, sv + 6144, PST_OFF + so, AUX_SYS);
                B[7] = __builtin_amdgcn_raw_buffer_load_b128(ws, sv + 7168, PST_OFF + so, AUX_SYS);
            } else {
                u32 exc = 0;
                if (meta & 0x780u) exc = __builtin_amdgcn_raw_buffer_load_b32(ws, lane * 4u + EXC_REL, PST_OFF + so, AUX_SYS);
                decode_task(B, meta, exc, lane);
            }
        }
        const bool isw = task_is_w(u);
        const u32 lo = raw_lane_off(isw, lane) + RAW_OFF + so;
#pragma unroll
        for (int i = 0; i < 8; ++i) st128(B[i], ws, lo + raw_vec_off(isw, i));
        owed = task_rdy(src, rank, u);
        if (qn >= QE || u >= NTASK - 96u) {
            VM0(); __builtin_amdgcn_raw_buffer_store_b32(call, ws, fv, owed, AUX_SYS); owed = 0;
        }
        q = qn;
    }
    if (owed) { VM0(); __builtin_amdgcn_raw_buffer_store_b32(call, ws, fv, owed, AUX_SYS); }
}


__global__ void __launch_bounds__(512) gemm_ag_q8(Params P) {
    __shared__ __attribute__((aligned(16))) char smem_[LDS_BYTES];
    LAS char* lds = (LAS char*)smem_;
    const int tid = threadIdx.x;
    const u32 lane = tid & 63;
    const u32 wave = rfl(tid >> 6);
    const u32 b = blockIdx.x;
    const u32 rank = P.rank;
    const u32 xcd = b & 7, idx = b >> 3;
    char* const myws = P.ws[rank];
    GAS u32* const ctr = (GAS u32*)(myws + CTR_OFF);
    GAS u32* const tmo = (GAS u32*)(myws + TMO_OFF);

    if (idx >= 26u) {
        const u32 ci = (idx - 26u) * 8u + xcd;
        if (ci >= NCOMM - NDECWG) {
            const u32 call = rfl(__hip_atomic_load(ctr, __ATOMIC_RELAXED, __HIP_MEMORY_SCOPE_AGENT)) + 1u;
            decoder(myws, rank, call, NGEMM + (ci - (NCOMM - NDECWG)) * 8u + wave, lane, lds + wave * SCR_BYTES);
            return;
        }
        if (wave == NPUSH) {
            if (ci != 0) return;
            const u32 call = rfl(__hip_atomic_load(ctr, __ATOMIC_RELAXED, __HIP_MEMORY_SCOPE_AGENT)) + 1u;
            const rsrc_t ws = mkr(myws, WS_USED);
            const u32 off = LMX_OFF + lane * 32u;
            const bool has = lane * 4u < NGEMM;
            const u64 t1 = rt();
            u32 gm = 0;
            for (u32 spins = 1;; ++spins) {
                const u32x4 a = __builtin_amdgcn_raw_buffer_load_b128(ws, off, 0, AUX_SYS);
                const u32x4 c = __builtin_amdgcn_raw_buffer_load_b128(ws, off + 16, 0, AUX_SYS);
                if (__all(!has || (a.y >= call && a.w >= call && c.y >= call && c.w >= call))) {
                    const u32 m1 = a.x > a.z ? a.x : a.z, m2 = c.x > c.z ? c.x : c.z;
                    gm = has ? (m1 > m2 ? m1 : m2) : 0u;
                    break;
                }
                if ((spins & 15u) == 0u && give_up(tmo, call, t1)) break;
                __builtin_amdgcn_s_sleep(1);
            }
            gm = wave_max_u32(gm);
            const u64 g = ((u64)call << 32) | gm;
            __hip_atomic_store((GAS u64*)(P.ws[lane >> 3] + AMX_OFF + (lane & 7) * 128 + rank * 8), g, __ATOMIC_RELAXED, __HIP_MEMORY_SCOPE_SYSTEM);
            return;
        }
        if (wave > NPUSH) return;
        const u32 sid = ci * NPUSH + wave;
        if (sid >= 7u * KPUSH) return;
        const u32 d = (rank + 1u + sid % 7u) & 7u, k = sid / 7u;
        const u32 call = rfl(__hip_atomic_load(ctr, __ATOMIC_RELAXED, __HIP_MEMORY_SCOPE_AGENT)) + 1u;
        const rsrc_t ws = mkr(myws, WS_USED), dst = mkr(P.ws[d], WS_USED);
        const u32 sv = lane * 16u;
        const u32 fv = lane == 0 ? 0u : OOB;
        GAS u32* const que = (GAS u32*)(myws + QUE_OFF + d * 128u);
        const u32 qbase = (call - 1u) * (NTASK - 4u * KPUSH) - 5u * KPUSH;
        auto prd_of = [&](u32 u) { return (GAS u64*)(myws + PRD_OFF + task_pck(rank, d, u) * 64u); };
        auto settle = [&](u32 u, u32 lo, u32 hi) {
            if (hi >= call) return lo;
            const u64 t0 = rt();
            for (u32 spins = 1;; ++spins) {
                const u64 f = __hip_atomic_load(prd_of(u), __ATOMIC_RELAXED, __HIP_MEMORY_SCOPE_SYSTEM);
                if (rfl((u32)(f >> 32)) >= call || ((spins & 15u) == 0u && give_up(tmo, call, t0))) return rfl((u32)f);
                __builtin_amdgcn_s_sleep(1);
            }
        };
        auto load_task = [&](u32x4* B, u32 u) {
            const u32 so = u >= NTASK - NRAW ? RAW_OFF + task_slot(rank, d, u) : PCK_OFF + task_pck(rank, d, u) * 8192u;
#pragma unroll
            for (int i = 0; i < 8; ++i) B[i] = __builtin_amdgcn_raw_buffer_load_b128(ws, sv + (i & 3) * 1024, so + (i >> 2) * 4096, AUX_SYS);
        };
        u32x4 A[8], B[8];
        static_assert(3 * KPUSH <= 96, "the tasks packed by the pushers lie in k slice 0");
        u32 c, n1, metac;
        {
            const rsrc_t xin = mkr(P.x, X_BYTES), win = mkr(P.w, W_BYTES);
            const u32 xv = (lane >> 3) * 1024u + (lane & 7u) * 16u;
            const u32 c16 = lane & 15u;
            const u32 wv = (lane >> 5) * 262144u + ((lane & 16u) + 8u * ((c16 >> 2) & 1u) + 4u * (c16 >> 3) + (c16 & 3u)) * 4u;
            u32 R[32], V[32];
            auto fetch = [&](u32 u) {
                if (u < 32u) {
                    const u32 base = (2048u * d + 64u * u) * 2u;
#pragma unroll
                    for (int i = 0; i < 32; ++i) R[i] = __builtin_amdgcn_raw_buffer_load_b32(win, wv, base + ((i >> 3) * 16u + (i & 7)) * 32768u, 0);
                } else {
                    const u32 base = task_xb(rank, d, u) * 65536u;
#pragma unroll
                    for (int i = 0; i < 8; ++i) {
                        const u32x4 v = __builtin_amdgcn_raw_buffer_load_b128(xin, xv, base + i * 8192u, 0);
                        R[4 * i] = v.x; R[4 * i + 1] = v.y; R[4 * i + 2] = v.z; R[4 * i + 3] = v.w;
                    }
                }
            };
            auto arrange = [&](u32 u) {
                if (u < 32u) {
#pragma unroll
                    for (int g = 0; g < 4; ++g)
#pragma unroll
                        for (int q = 0; q < 4; ++q) {
                            V[8 * g + q] = __builtin_amdgcn_perm(R[8 * g + 2 * q + 1], R[8 * g + 2 * q], 0x05040100u);
                            V[8 * g + 4 + q] = __builtin_amdgcn_perm(R[8 * g + 2 * q + 1], R[8 * g + 2 * q], 0x07060302u);
                        }
                } else {
#pragma unroll
                    for (int i = 0; i < 32; ++i) V[i] = R[i];
                }
            };
            auto send = [&](u32x4* O, u32 u, u32 meta) {
                const u32 slot = PST_OFF + task_slot(rank, d, u);
#pragma unroll
                for (int i = 0; i < 6; ++i) st128(O[i], dst, sv + slot + i * 1024u);
                if (meta & META_RAW) { st128(O[6], dst, sv + slot + 6144u); st128(O[7], dst, sv + slot + 7168u); }
            };
            auto flag = [&](u32 u, u32 meta) {
                VM0();
                u32x2 f; f.x = meta; f.y = call;
                __builtin_amdgcn_raw_buffer_store_b64(f, dst, fv, ARR_OFF + (rank * NTASK + u) * 64u, AUX_SYS);
            };
            const u32 u0 = k, u1 = k + KPUSH, u2 = k + 2u * KPUSH;
            fetch(u0);
            arrange(u0); fetch(u1);
            const u32 m0 = encode_task(V, A, dst, PST_OFF + task_slot(rank, d, u0), lane);
            send(A, u0, m0);
            arrange(u1); fetch(u2);
            const u32 m1 = encode_task(V, B, dst, PST_OFF + task_slot(rank, d, u1), lane);
            flag(u0, m0);
            send(B, u1, m1);
            arrange(u2);
            const u32 m2 = encode_task(V, A, dst, PST_OFF + task_slot(rank, d, u2), lane);
            flag(u1, m1);
            send(A, u2, m2);
            c = k + 3u * KPUSH; n1 = k + 4u * KPUSH;
            metac = settle(c, 0u, 0u);
            asm volatile("" ::: "memory");
            load_task(B, c);
            flag(u2, m2);
            asm volatile("s_waitcnt vmcnt(0)" : "+v"(B[0]), "+v"(B[1]), "+v"(B[2]), "+v"(B[3]), "+v"(B[4]), "+v"(B[5]), "+v"(B[6]), "+v"(B[7]) :: "memory");
        }

        u32 fa = OOB, fb = OOB, oa = 0, ob = 0;
        u32x2 fo; fo.x = 0; fo.y = call;
#define ST1(CUR, i) do { \
            if ((i) < 5) st128(CUR[i], dst, so + (i) * 1024u); \
            else if ((i) == 5) st128(CUR[5], dst, (!full && (metac & META_TIGHT) && lane >= 48u) ? OOB : so + 5120u); \
            else if ((i) == 6) st128(CUR[6], dst, (full || ((metac & 0x780u) && lane < 2u)) ? so + 6144u : OOB); \
            else st128(CUR[7], dst, full ? so + 7168u : OOB); } while (0)
#define PUSH(CUR, NXT) do { \
            const bool tail = c >= NTASK - NRAW; \
            const bool full = tail || (metac & META_RAW); \
            const u32 so = sv + (tail ? RAW_OFF : PST_OFF) + task_slot(rank, d, c); \
            const bool v1 = n1 < NTASK; \
            GAS u64* const pa = prd_of(v1 ? n1 : 0u); \
            u32 tk = 1u, meta1 = 0; \
            u64 pk; \
            asm volatile("global_load_dwordx2 %0, %1, off sc0 sc1" : "=v"(pk) : "v"(pa) : "memory"); \
            __builtin_amdgcn_raw_buffer_store_b64(fo, dst, fa, oa, AUX_SYS); \
            __builtin_amdgcn_raw_buffer_store_b32(call, dst, fb, ob, AUX_SYS); \
            _Pragma("unroll") for (int i = 0; i < PB; ++i) ST1(CUR, i); \
            asm volatile("s_waitcnt vmcnt(" STR(PB2) ")" : "+v"(pk) :: "memory"); \
            if (v1) { \
                meta1 = settle(n1, rfl((u32)pk), rfl((u32)(pk >> 32))); \
                asm volatile("" ::: "memory"); \
                load_task(NXT, n1); \
                if (PB < 8 && n1 < NTASK - KPUSH) asm volatile("s_waitcnt vmcnt(8)" ::: "memory");        \
                _Pragma("unroll") for (int i = PB; i < 8; ++i) ST1(CUR, i); \
                u64 sv_; \
                asm volatile("s_mov_b64 %1, exec\n\ts_mov_b64 exec, 1\n\tglobal_atomic_add %0, %2, %0, off sc0\n\ts_mov_b64 exec, %1\n\t" \
                             "s_waitcnt vmcnt(0)" : "+v"(tk), "=&s"(sv_) : "v"(que) : "memory"); \
                asm volatile("" : "+v"(NXT[0]), "+v"(NXT[1]), "+v"(NXT[2]), "+v"(NXT[3]), "+v"(NXT[4]), "+v"(NXT[5]), "+v"(NXT[6]), \
                             "+v"(NXT[7]) :: "memory");            \
            } else { \
                _Pragma("unroll") for (int i = PB; i < 8; ++i) ST1(CUR, i); \
                VM0(); \
            } \
            fo.x = metac; \
            fa = tail ? OOB : fv; fb = tail ? fv : OOB; \
            oa = ARR_OFF + (rank * NTASK + c) * 64u; ob = task_rdy(rank, d, c); \
            c = n1; metac = meta1; \
            if (v1) n1 = rfl(tk) - qbase; \
            } while (0)
        for (;;) {
            PUSH(B, A);
            if (c >= NTASK) break;
            PUSH(A, B);
            if (c >= NTASK) break;
        }
        __builtin_amdgcn_raw_buffer_store_b64(fo, dst, fa, oa, AUX_SYS);
        __builtin_amdgcn_raw_buffer_store_b32(call, dst, fb, ob, AUX_SYS);
        VM0();
        return;
    }

    const u32 ti4 = (xcd & 1u) * 26u + idx;
    const u32 mt = ti4 >> 2;
    const u32 nt = (xcd >> 1) * 4u + (ti4 & 3u);
    const u32 gi = mt * 16u + nt;
    if (tid < (int)((L_SCR - L_CTL) / 4)) *(LAS u32*)(lds + L_CTL + tid * 4) = 0u;
    const u32 call = rfl(__hip_atomic_load(ctr, __ATOMIC_RELAXED, __HIP_MEMORY_SCOPE_AGENT)) + 1u;
    LGKM0();
    __builtin_amdgcn_s_barrier();

    if (wave == 7) {
        decoder(myws, rank, call, gi, lane, lds + L_SCR + 3 * SCR_BYTES);
        return;
    }
    if (wave == 6) {
        const rsrc_t ws = mkr(myws, WS_USED);
        const u32 fx = RDX_OFF + (mt * 64u + lane) * 32u, fw = RDW_OFF + (nt * 64u + lane) * 8u;
        const bool five = mt < MT - 1;
        u64 pending = ~0ull;
        u32 n = 0;
        const u64 t0 = rt();
        u32 spins = 0;
        while (pending) {
            const bool mine = (pending >> lane) & 1ull;
            u32x4 a = {0, 0, 0, 0};
            u32 a4 = 0;
            u32x2 cc = {0, 0};
            if (mine) {
                a = __builtin_amdgcn_raw_buffer_load_b128(ws, fx, 0, AUX_SYS);
                a4 = __builtin_amdgcn_raw_buffer_load_b32(ws, fx + 16, 0, AUX_SYS);
                cc = __builtin_amdgcn_raw_buffer_load_b64(ws, fw, 0, AUX_SYS);
            }
            const bool ready = mine && a.x >= call && a.y >= call && a.z >= call && a.w >= call && (a4 >= call || !five) && cc.x >= call && cc.y >= call;
            u64 mask = __ballot(ready);
            if (!mask) {
                if ((++spins & 15u) == 0u && give_up(tmo, call, t0)) mask = pending;
                else { __builtin_amdgcn_s_sleep(1); continue; }
            }
            if ((mask >> lane) & 1ull) {
                const u32 pos = n + __builtin_amdgcn_mbcnt_hi((u32)(mask >> 32), __builtin_amdgcn_mbcnt_lo((u32)mask, 0u));
                *(LAS u32*)(lds + L_ORD + pos * 4) = lane;
            }
            LGKM0();
            n += __builtin_popcountll(mask);
            pending &= ~mask;
            if (lane == 0) lds_st(lds, L_OCNT, n);
        }
        const u32 row0 = 320u * mt;
        const u32 nrow8 = mt == MT - 1 ? 32u : 40u;
        for (u32 gq = 0; gq < 40u; ++gq) {
            const u32 i = gq >> 3, g = (gq & 1u) * 20u + 4u * i + ((gq >> 1) & 3u);
            if ((gq & 7u) == 0u) lds_wait_ge(lds, L_OUT + i * 4u, 4u);
            if (g >= nrow8) continue;
            const u32 row = 8 * g + (lane >> 3), ch = lane & 7;
            const u32x4 v = *(const LAS u32x4*)(lds + row * 128 + ((ch ^ (row & 7)) * 16));
            const u32 grow = row0 + 8 * g;
            const rsrc_t ob = mkr(P.out[(grow >> 9) & 7u], OUT_BYTES);
            st128(v, ob, ((grow & 511u) + (lane >> 3)) * NN + rank * NL + nt * 128 + ch * 16);
            asm volatile("s_waitcnt vmcnt(" STR(OUTWIN1) ")" ::: "memory");
        }
        VM0();
        const u32 d0 = row0 >> 9, d1 = (row0 + 8u * nrow8 - 1u) >> 9;
        if (lane <= d1 - d0) {
            const u32 d = (d0 + lane) & 7u;
            __hip_atomic_store((GAS u32*)(P.ws[d] + DON_OFF + ((rank * 3u + (mt - (512u * d) / 320u)) * 16u + nt) * 128u), call, __ATOMIC_RELAXED, __HIP_MEMORY_SCOPE_SYSTEM);
        }
        const u32 cnt = (512u * rank + 511u) / 320u - (512u * rank) / 320u + 1u;
        for (u32 e = gi; e < 8u * cnt * 16u; e += NGEMM) {
            GAS u32* f = (GAS u32*)(myws + DON_OFF + (((e / (cnt * 16u)) * 3u + (e / 16u) % cnt) * 16u + e % 16u) * 128u);
            const u64 t1 = rt();
            for (u32 sp = 1; rfl(__hip_atomic_load(f, __ATOMIC_RELAXED, __HIP_MEMORY_SCOPE_SYSTEM)) < call; ++sp) {
                if ((sp & 15u) == 0u && give_up(tmo, call, t1)) {
                    if (lane == 0) __hip_atomic_store((GAS unsigned char*)(P.out[rank] + rank * NL), (unsigned char)0x80, __ATOMIC_RELAXED, __HIP_MEMORY_SCOPE_SYSTEM);
                    break;
                }
                __builtin_amdgcn_s_sleep(1);
            }
        }
        return;
    }
    if (wave >= 4) {
        const u32 half = wave - 4;
        const char* xsrc = myws + RAW_OFF + XS_REL + (10u * mt) * BLK + half * 2048u + lane * 16u;
        const char* wsrc = myws + RAW_OFF + WS_REL + (4u * nt) * BLK + half * 2048u + lane * 16u;
        const u32 lbase = (u32)(uintptr_t)lds;
        auto rd = [&](u32 off) {
            u32 v;
            asm volatile("ds_read_b32 %0, %1\n\ts_waitcnt lgkmcnt(0)" : "=v"(v) : "v"(lbase + off) : "memory");
            return rfl(v);
        };
        u32 ni = 0, islot = half, iuse = 0;
        u32 np = 0, pslot = half;
        while (np < 64) {
            bool can = ni < 64 && ni - np < 2u && rd(L_OCNT) > ni;
            if (can) can = rd(L_FREE + islot * 4) >= 4u * iuse;
            if (can) {
                const u32 s_ = rd(L_ORD + ni * 4) & 63u;
                const char* gx = xsrc + (size_t)s_ * (128u * BLK);
                const char* gw = wsrc + (size_t)s_ * (64u * BLK);
                LAS char* sb = lds + islot * SLOT;
#define DMA2(g, l) do { \
                __builtin_amdgcn_global_load_lds((const GAS u32*)(g), (LAS u32*)(l), 16, 0, AUX_SYS); \
                __builtin_amdgcn_global_load_lds((const GAS u32*)(g), (LAS u32*)(l), 16, 1024, AUX_SYS); } while (0)
#pragma unroll
                for (int i = 0; i < 10; ++i) DMA2(gx + i * BLK, sb + i * 2048);
#pragma unroll
                for (int i = 0; i < 4; ++i) DMA2(gw + i * BLK, sb + (10 + i) * 2048);
                ++ni;
                islot += 2; if (islot >= NSLOT) { islot -= NSLOT; ++iuse; }
            } else if (np < ni) {
                if (ni - np == 2) asm volatile("s_waitcnt vmcnt(28)" ::: "memory"); else VM0();
                if (lane == 0) asm volatile("ds_add_u32 %0, %1" :: "v"(lbase + L_FULL + pslot * 4), "v"(1u) : "memory");
                ++np;
                pslot += 2; if (pslot >= NSLOT) pslot -= NSLOT;
            } else {
                __builtin_amdgcn_s_sleep(1);
            }
        }
        return;
    }

    {
        const rsrc_t xin = mkr(P.x, X_BYTES), win = mkr(P.w, W_BYTES), ws = mkr(myws, WS_USED);
        const u32 xv = (lane >> 3) * 1024u + (lane & 7u) * 16u;
        const u32 c16 = lane & 15u;
        const u32 wv = (lane >> 5) * 262144u + ((lane & 16u) + 8u * ((c16 >> 2) & 1u) + 4u * (c16 >> 3) + (c16 & 3u)) * 4u;
        const u32 sv = lane * 16u;
        const u32 fv = lane == 0 ? 0u : OOB;
        for (u32 e = wave * NGEMM + gi; wave < 3u && e < 2560u; e += 3u * NGEMM) {
            const u32 j = e / 320u, r = e % 320u;
            if (e < 3u * NGEMM) for (u32 i = 0; i < j; ++i) __builtin_amdgcn_s_sleep(STAGGER);
            const bool isw = r < 256u;
            const u32 d = isw ? (r & 7u) : rank, ti = isw ? (r >> 3) : r - 256u;
            if (isw && j == 0 && ti < 3u * KPUSH && d != rank) continue;
            u32 V[32];
            if (isw) {
                u32 R[32];
                const u32 base = (64u * j * 16384u + 2048u * d + 64u * ti) * 2u;
#pragma unroll
                for (int i = 0; i < 32; ++i)
                    R[i] = __builtin_amdgcn_raw_buffer_load_b32(win, wv, base + ((i >> 3) * 16u + (i & 7)) * 32768u, 0);
#pragma unroll
                for (int g = 0; g < 4; ++g)
#pragma unroll
                    for (int q = 0; q < 4; ++q) {
                        V[8 * g + q] = __builtin_amdgcn_perm(R[8 * g + 2 * q + 1], R[8 * g + 2 * q], 0x05040100u);
                        V[8 * g + 4 + q] = __builtin_amdgcn_perm(R[8 * g + 2 * q + 1], R[8 * g + 2 * q], 0x07060302u);
                    }
            } else {
                const u32 base = ti * 65536u + j * 128u;
#pragma unroll
                for (int i = 0; i < 8; ++i) {
                    const u32x4 v = __builtin_amdgcn_raw_buffer_load_b128(xin, xv, base + i * 8192u, 0);
                    V[4 * i] = v.x; V[4 * i + 1] = v.y; V[4 * i + 2] = v.z; V[4 * i + 3] = v.w;
                }
            }
            const u32 st = rank * 8u + j;
            if (d == rank) {
                const u32 rel = isw ? WS_REL + (st * 64u + 2u * ti) * BLK : XS_REL + (st * 128u + 2u * ti) * BLK;
                const u32 lo = raw_lane_off(isw, lane) + RAW_OFF + rel;
#pragma unroll
                for (int i = 0; i < 8; ++i) { const u32x4 v = {V[4 * i], V[4 * i + 1], V[4 * i + 2], V[4 * i + 3]}; st128(v, ws, lo + raw_vec_off(isw, i)); }
            }
            u32 meta = 0;
            const u32 pk = isw ? 512u + (d * 8u + j) * 32u + ti : j * 64u + ti;
            const bool pack = !isw || d != rank;
            if (pack) {
                u32x4 O[8];
                const u32 slot = PCK_OFF + pk * 8192u, so = sv + slot;
                u32x2 S; u32 list;
                meta = encode_tight(V, O, S, list, lds + L_SCR + wave * SCR_BYTES, lane);
                if (meta) {
#pragma unroll
                    for (int i = 0; i < 5; ++i) st128(O[i], ws, so + i * 1024u);
                    __builtin_amdgcn_raw_buffer_store_b64(S, ws, lane * 8u + SGN_REL, slot, AUX_SYS);
                    __builtin_amdgcn_raw_buffer_store_b32(list, ws, lane * 4u + LST_REL, slot, AUX_SYS);
                } else {
                    meta = encode_task(V, O, ws, slot, lane);
#pragma unroll
                    for (int i = 0; i < 6; ++i) st128(O[i], ws, so + i * 1024u);
                    if (meta & META_RAW) { st128(O[6], ws, so + 6144u); st128(O[7], ws, so + 7168u); }
                }
            }
            VM0();
            if (pack) { u32x2 f; f.x = meta; f.y = call; __builtin_amdgcn_raw_buffer_store_b64(f, ws, fv, PRD_OFF + pk * 64u, AUX_SYS); }
            if (d == rank) {
                const u32 rdy = isw ? RDW_OFF + ((ti >> 1) * 64u + st) * 8u + (ti & 1u) * 4u : RDX_OFF + ((ti / 5u) * 64u + st) * 32u + (ti % 5u) * 4u;
                __builtin_amdgcn_raw_buffer_store_b32(call, ws, fv, rdy, AUX_SYS);
            }
        }
    }
    const u32 wm = wave >> 1, wn = wave & 1;
    f32x16 acc[2][5];
#pragma unroll
    for (int f = 0; f < 2; ++f)
#pragma unroll
        for (int i = 0; i < 5; ++i)
#pragma unroll
            for (int r = 0; r < 16; ++r) acc[f][i][r] = 0.f;
    {
        const LAS char* xb0 = lds + (5 * wm) * 2048 + lane * 16;
        const LAS char* wb0 = lds + (10 + 2 * wn) * 2048 + lane * 16;
        u32x4 F0[7], F1[7];
#define MM(F) _Pragma("unroll") for (int i = 0; i < 5; ++i) _Pragma("unroll") for (int f = 0; f < 2; ++f) \
            acc[f][i] = __builtin_amdgcn_mfma_f32_32x32x16_bf16(__builtin_bit_cast(bf16x8, F[5 + f]), __builtin_bit_cast(bf16x8, F[i]), acc[f][i], 0, 0, 0)
#define SB() __builtin_amdgcn_sched_barrier(0)
#define MF(F, i, f) acc[f][i] = __builtin_amdgcn_mfma_f32_32x32x16_bf16(__builtin_bit_cast(bf16x8, F[5 + f]), __builtin_bit_cast(bf16x8, F[i]), acc[f][i], 0, 0, 0)
#define RX(F, i, sl, t) F[i] = *(const LAS u32x4*)(xb0 + (sl) + (i) * 2048 + (t) * 1024)
#define RW(F, f, sl, t) F[5 + f] = *(const LAS u32x4*)(wb0 + (sl) + (f) * 2048 + (t) * 1024)
        const u32 lbase = (u32)(uintptr_t)lds;
        u32 slot = 0, use = 1;
        lds_wait_ge(lds, L_FULL, 1u);
        SB();
        RW(F0, 0, 0u, 0); SB(); RX(F0, 0, 0u, 0); SB(); RW(F0, 1, 0u, 0); SB(); RX(F0, 1, 0u, 0); SB();
        RX(F0, 2, 0u, 0); SB(); RX(F0, 3, 0u, 0); SB(); RX(F0, 4, 0u, 0); SB();
        u32 cur = 0, nxt = 1;
        for (u32 h = 0; h < 127; ++h) {
            const u32 co = cur * SLOT, no = nxt * SLOT;
            SB();
            MF(F0, 0, 0); const u32 fl = lds_ld(lds, L_FULL + nxt * 4); RW(F1, 0, co, 1); SB();
            MF(F0, 0, 1); RX(F1, 0, co, 1); SB();
            MF(F0, 1, 0); RW(F1, 1, co, 1); SB();
            MF(F0, 1, 1); RX(F1, 1, co, 1); SB();
            MF(F0, 2, 0); RX(F1, 2, co, 1); SB();
            MF(F0, 2, 1); RX(F1, 3, co, 1); SB();
            MF(F0, 3, 0); RX(F1, 4, co, 1); SB();
            MF(F0, 3, 1); SB();
            MF(F0, 4, 0); SB();
            MF(F0, 4, 1); SB();
            if (rfl(fl) < use) lds_wait_ge(lds, L_FULL + nxt * 4, use);
            asm volatile("" ::: "memory");
            SB();
            MF(F1, 0, 0); RW(F0, 0, no, 0); SB();
            MF(F1, 0, 1); RX(F0, 0, no, 0); SB();
            MF(F1, 1, 0); RW(F0, 1, no, 0); SB();
            MF(F1, 1, 1); RX(F0, 1, no, 0); SB();
            MF(F1, 2, 0); RX(F0, 2, no, 0); SB();
            {
                u64 sv_;
                asm volatile("s_waitcnt lgkmcnt(5)\n\ts_mov_b64 %0, exec\n\ts_mov_b64 exec, 1\n\tds_add_u32 %1, %2\n\ts_mov_b64 exec, %0"
                             : "=&s"(sv_) : "v"(lbase + L_FREE + cur * 4), "v"(1u) : "memory");
            }
            SB();
            MF(F1, 2, 1); RX(F0, 3, no, 0); SB();
            MF(F1, 3, 0); RX(F0, 4, no, 0); SB();
            MF(F1, 3, 1); SB();
            MF(F1, 4, 0);
            cur = nxt;
            if (++nxt == NSLOT) { nxt = 0; ++use; }
            SB();
            MF(F1, 4, 1); SB();
        }
        slot = cur;
        const u32 slot_last = slot * SLOT;
        RW(F1, 0, slot_last, 1); RW(F1, 1, slot_last, 1);
#pragma unroll
        for (int i = 0; i < 5; ++i) RX(F1, i, slot_last, 1);
        MM(F0);
        MM(F1);
    }
    const u32 ngrp = (mt == MT - 1 && wm == 1) ? 3u : 5u;
    float mx = 0.f;
#pragma unroll
    for (int i = 0; i < 5; ++i) {
        float m = 0.f;
#pragma unroll
        for (int f = 0; f < 2; ++f)
#pragma unroll
            for (int r = 0; r < 16; ++r) m = fmaxf(m, fabsf(acc[f][i][r]));
        mx = (u32)i < ngrp ? fmaxf(mx, m) : mx;
    }
    const u32 mb = wave_max_u32(__float_as_uint(mx));
    u32 arrived = 0;
    if (lane == 0) {
        (void)__hip_atomic_fetch_max((LAS u32*)(lds + L_WMAX), mb, __ATOMIC_RELAXED, __HIP_MEMORY_SCOPE_WORKGROUP);
        arrived = __hip_atomic_fetch_add((LAS u32*)(lds + L_WCNT), 1u, __ATOMIC_RELAXED, __HIP_MEMORY_SCOPE_WORKGROUP);
    }
    arrived = rfl(arrived);
    if (arrived == 3) {
        const u32 wmax = rfl(lds_ld(lds, L_WMAX));
        if (lane == 0)
            __hip_atomic_store((GAS u64*)(myws + LMX_OFF + gi * 8), ((u64)call << 32) | wmax, __ATOMIC_RELAXED, __HIP_MEMORY_SCOPE_AGENT);
    }
    if (wave == 0) {
        GAS u64* g = (GAS u64*)(myws + AMX_OFF + xcd * 128 + (lane & 7) * 8);
        const u64 t0 = rt();
        u32 gm = 0, bad = 0;
        for (u32 spins = 1;; ++spins) {
            const u64 v = __hip_atomic_load(g, __ATOMIC_RELAXED, __HIP_MEMORY_SCOPE_SYSTEM);
            const u32 tm = __hip_atomic_load(tmo, __ATOMIC_RELAXED, __HIP_MEMORY_SCOPE_AGENT);
            bad = rfl(tm) == call;
            if (bad || __all((u32)(v >> 32) >= call)) { gm = (u32)v; break; }
            if ((spins & 15u) == 0u && give_up(tmo, call, t0)) { bad = 1; break; }
            __builtin_amdgcn_s_sleep(1);
        }
        gm = wave_max_u32(gm);
        if (lane == 0) { lds_st(lds, L_GMAX, gm); lds_st(lds, L_BAD, bad); }
        LGKM0();
        if (lane == 0) lds_st(lds, L_GOK, 1u);
        if (gi == 0 && lane == 0) __hip_atomic_store(ctr, call, __ATOMIC_RELAXED, __HIP_MEMORY_SCOPE_AGENT);
    }
    lds_wait_ge(lds, L_GOK, 1u);
    const float amax = __uint_as_float(rfl(lds_ld(lds, L_GMAX)));
    const bool poison = rfl(lds_ld(lds, L_BAD)) != 0u;
    {
        const float scale = amax / 127.0f;
        const float inv = scale > 0.f ? 1.0f / scale : 0.f;
        const f32x2 inv2 = {inv, inv}, magic2 = {12582912.f + 128.f, 12582912.f + 128.f};
#pragma unroll
        for (int i = 0; i < 5; ++i) {
#pragma unroll
            for (int f = 0; f < 2; ++f) {
                const u32 ch = 4 * wn + 2 * f + (lane >> 5);
                u32x4 v;
#pragma unroll
                for (int q = 0; q < 4; ++q) {
                    constexpr float MAGIC = 12582912.f + 128.f;
                    const f32x2 lo2 = {acc[f][i][2 * q], acc[f][i][2 * q + 1]}, hi2 = {acc[f][i][2 * q + 8], acc[f][i][2 * q + 9]};
                    const f32x2 tl = __builtin_elementwise_fma(lo2, inv2, magic2), th = __builtin_elementwise_fma(hi2, inv2, magic2);
                    const u32 b0 = __float_as_uint(__builtin_amdgcn_fmed3f(tl.x, MAGIC - 127.f, MAGIC + 127.f));
                    const u32 b1 = __float_as_uint(__builtin_amdgcn_fmed3f(th.x, MAGIC - 127.f, MAGIC + 127.f));
                    const u32 b2 = __float_as_uint(__builtin_amdgcn_fmed3f(tl.y, MAGIC - 127.f, MAGIC + 127.f));
                    const u32 b3 = __float_as_uint(__builtin_amdgcn_fmed3f(th.y, MAGIC - 127.f, MAGIC + 127.f));
                    const u32 d = __builtin_amdgcn_perm(__builtin_amdgcn_perm(b3, b2, 0x0c0c0400u), __builtin_amdgcn_perm(b1, b0, 0x0c0c0400u), 0x05040100u) ^ 0x80808080u;
                    v[q] = poison ? 0x80808080u : d;
                }
                const u32 row = 160 * wm + 32 * i + (lane & 31);
                *(LAS u32x4*)(lds + row * 128 + ((ch ^ (row & 7)) * 16)) = v;
            }
            LGKM0();
            if (lane == 0) lds_add(lds, L_OUT + i * 4, 1u);
            __builtin_amdgcn_sched_barrier(0);
        }
    }
}

extern "C" void kernel_launch(int rank, int nranks, ncclComm_t comm, hipStream_t stream,
                              void* const* d_in, const int* in_sizes, int n_in,
                              void* d_out, int out_size, const int* in_dtypes, int out_dtype,
                              void* const* d_out_all, void* const* d_ws_all, size_t ws_bytes) {
    (void)comm; (void)in_dtypes; (void)out_dtype; (void)d_out;
    if (nranks != NR || (unsigned)rank >= (unsigned)NR || n_in != 2 || in_sizes[0] != MM * KSH || in_sizes[1] != KSH * NN ||
        out_size != (int)OUT_BYTES || ws_bytes < WS_USED) {
        fprintf(stderr, "kernel_launch: unexpected shapes\n");
        return;
    }
    Params P;
    P.x = d_in[0];
    P.w = d_in[1];
    for (int p = 0; p < NR; ++p) { P.ws[p] = (char*)d_ws_all[p]; P.out[p] = (char*)d_out_all[p]; }
    P.rank = rank;
    P.pad = 0;
    hipLaunchKernelGGL(gemm_ag_q8, dim3(256), dim3(512), 0, stream, P);
}
```

```cpp
#include <hip/hip_runtime.h>
#include <rccl/rccl.h>
#include <cstdio>
#include <cstdint>

typedef unsigned u32;
typedef unsigned long long u64;
typedef u32 u32x4 __attribute__((ext_vector_type(4)));
typedef u32 u32x2 __attribute__((ext_vector_type(2)));
typedef float f32x16 __attribute__((ext_vector_type(16)));
typedef float f32x2 __attribute__((ext_vector_type(2)));
typedef __bf16 bf16x8 __attribute__((ext_vector_type(8)));
typedef __amdgpu_buffer_rsrc_t rsrc_t;
#define LAS __attribute__((address_space(3)))
#define GAS __attribute__((address_space(1)))

constexpr int NR = 8;
constexpr int MM = 4096;
constexpr int KSH = 512;
constexpr int NN = 16384;
constexpr int NL = NN / NR;
constexpr u32 X_BYTES = MM * KSH * 2;
constexpr u32 W_BYTES = KSH * NN * 2;
constexpr u32 OUT_BYTES = (MM / NR) * NN;
constexpr u32 BLK = 4096;
constexpr u32 XS_REL = 0;
constexpr u32 WS_REL = 64u * 128u * BLK;
constexpr u32 STG_BYTES = WS_REL + 64u * 64u * BLK;
constexpr u32 PST_OFF = 0;
constexpr u32 RAW_OFF = STG_BYTES;
constexpr u32 NTASK = 768;
constexpr u32 PCK_OFF = 2u * STG_BYTES;
constexpr u32 PRD_OFF = PCK_OFF + 2560u * 8192u;
constexpr u32 ARR_OFF = PRD_OFF + 2560u * 64u;
constexpr u32 MT = 13;
constexpr u32 NGEMM = MT * 16;
constexpr u32 NCOMM = 256 - NGEMM;
constexpr u32 RDX_OFF = ARR_OFF + 8u * NTASK * 64u;
constexpr u32 RDW_OFF = RDX_OFF + MT * 64u * 32u;
constexpr u32 LMX_OFF = RDW_OFF + 16u * 64u * 8u;
constexpr u32 AMX_OFF = LMX_OFF + 256u * 8u;
constexpr u32 DON_OFF = AMX_OFF + 8u * 128u;
constexpr u32 QUE_OFF = DON_OFF + 384u * 128u;
constexpr u32 CTR_OFF = QUE_OFF + 8u * 128u;
constexpr u32 TMO_OFF = CTR_OFF + 128u;
constexpr u32 WS_USED = CTR_OFF + 4096u;
constexpr int AUX_SYS = 17;
constexpr u64 DEADLINE = 100000000ull;
constexpr u32 OOB = 0x40000000u;

constexpr u32 NDECWG = 16;
constexpr u32 NPUSH = 6;
constexpr u32 KPUSH = 27;
#define PB 3
#define PB2 5
constexpr u32 NRAW = 48;
#define STAGGER 40
#define OUTWIN1 1
static_assert(7 * KPUSH <= (NCOMM - NDECWG) * NPUSH && NPUSH < 8, "pushers");
static_assert(NRAW <= 64, "the unpacked tail consists of x tasks");

constexpr u32 SLOT = 14 * 2048;
constexpr u32 NSLOT = 5;
constexpr u32 L_CTL = NSLOT * SLOT;
constexpr u32 L_FULL = L_CTL;
constexpr u32 L_FREE = L_CTL + 32;
constexpr u32 L_OCNT = L_CTL + 64;
constexpr u32 L_WMAX = L_CTL + 68;
constexpr u32 L_WCNT = L_CTL + 72;
constexpr u32 L_GMAX = L_CTL + 76;
constexpr u32 L_BAD = L_CTL + 80;
constexpr u32 L_GOK = L_CTL + 84;
constexpr u32 L_OUT = L_CTL + 96;
constexpr u32 L_ORD = L_CTL + 128;
constexpr u32 L_SCR = L_ORD + 256;
constexpr u32 LDS_BYTES = L_SCR + 4 * 4352;

struct Params {
    const void* x;
    const void* w;
    char* ws[NR];
    char* out[NR];
    int rank;
    int pad;
};

__device__ __forceinline__ rsrc_t mkr(const void* p, u32 bytes) {
    return __builtin_amdgcn_make_buffer_rsrc((void*)p, 0, bytes, 0x00020000);
}
__device__ __forceinline__ u64 rt() { return __builtin_amdgcn_s_memrealtime(); }
__device__ __forceinline__ u32 rfl(u32 v) { return __builtin_amdgcn_readfirstlane(v); }
__device__ __forceinline__ void st128(u32x4 v, rsrc_t r, u32 off) { __builtin_amdgcn_raw_buffer_store_b128(v, r, off, 0, 17); }
#define STR2(x) #x
#define STR(x) STR2(x)
#define LGKM0() asm volatile("s_waitcnt lgkmcnt(0)" ::: "memory")
#define VM0() asm volatile("s_waitcnt vmcnt(0)" ::: "memory")

__device__ __forceinline__ u32 lds_ld(const LAS char* l, u32 off) {
    return __hip_atomic_load((const LAS u32*)(l + off), __ATOMIC_RELAXED, __HIP_MEMORY_SCOPE_WORKGROUP);
}
__device__ __forceinline__ void lds_st(LAS char* l, u32 off, u32 v) {
    __hip_atomic_store((LAS u32*)(l + off), v, __ATOMIC_RELAXED, __HIP_MEMORY_SCOPE_WORKGROUP);
}
__device__ __forceinline__ void lds_add(LAS char* l, u32 off, u32 v) {
    (void)__hip_atomic_fetch_add((LAS u32*)(l + off), v, __ATOMIC_RELAXED, __HIP_MEMORY_SCOPE_WORKGROUP);
}
__device__ __forceinline__ void lds_wait_ge(const LAS char* l, u32 off, u32 want) {
    while (rfl(lds_ld(l, off)) < want) __builtin_amdgcn_s_sleep(1);
    asm volatile("" ::: "memory");
}
__device__ __forceinline__ bool give_up(GAS u32* tmo, u32 call, u64 t0) {
    if (rfl(__hip_atomic_load(tmo, __ATOMIC_RELAXED, __HIP_MEMORY_SCOPE_AGENT)) == call || rt() - t0 > DEADLINE) {
        __hip_atomic_store(tmo, call, __ATOMIC_RELAXED, __HIP_MEMORY_SCOPE_AGENT);
        return true;
    }
    return false;
}
__device__ __forceinline__ u32 wave_max_u32(u32 v) {
    u32 o;
    o = __builtin_amdgcn_update_dpp(0u, v, 0x111, 0xf, 0xf, false); v = v > o ? v : o;
    o = __builtin_amdgcn_update_dpp(0u, v, 0x112, 0xf, 0xf, false); v = v > o ? v : o;
    o = __builtin_amdgcn_update_dpp(0u, v, 0x114, 0xf, 0xf, false); v = v > o ? v : o;
    o = __builtin_amdgcn_update_dpp(0u, v, 0x118, 0xf, 0xf, false); v = v > o ? v : o;
    o = __builtin_amdgcn_update_dpp(0u, v, 0x142, 0xa, 0xf, false); v = v > o ? v : o;
    o = __builtin_amdgcn_update_dpp(0u, v, 0x143, 0xc, 0xf, false); v = v > o ? v : o;
    return __builtin_amdgcn_readlane(v, 63);
}

constexpr u32 EXC_CAP = 8;
constexpr u32 META_RAW = 0x800u;
constexpr u32 EXC_REL = 6144;
__device__ __forceinline__ u32 pkmax_u16(u32 a, u32 b) {
    typedef unsigned short u16x2 __attribute__((ext_vector_type(2)));
    return __builtin_bit_cast(u32, __builtin_elementwise_max(__builtin_bit_cast(u16x2, a), __builtin_bit_cast(u16x2, b)));
}
__device__ __forceinline__ u32 encode_task(const u32* V, u32x4* O, rsrc_t dst, u32 slot_off, u32 lane) {
    u32 m = V[0] & 0x7fff7fffu;
#pragma unroll
    for (int i = 1; i < 32; ++i) m = pkmax_u16(m, V[i] & 0x7fff7fffu);
    m = pkmax_u16(m, m >> 16) & 0xffffu;
    const u32 top = wave_max_u32(m) >> 8;
    const u32 base = top > 7u ? top - 7u : 0u;
    const u32 b4 = base * 0x01010101u;
    u32 bad = 0;
    u32 Lp[16], X[8];
#pragma unroll
    for (int t = 0; t < 8; ++t) {
        const u32 h01 = __builtin_amdgcn_perm(V[4 * t + 1], V[4 * t], 0x07050301u), h23 = __builtin_amdgcn_perm(V[4 * t + 3], V[4 * t + 2], 0x07050301u);
        Lp[2 * t] = __builtin_amdgcn_perm(V[4 * t + 1], V[4 * t], 0x06040200u);
        Lp[2 * t + 1] = __builtin_amdgcn_perm(V[4 * t + 3], V[4 * t + 2], 0x06040200u);
        const u32 u01 = (h01 | 0x80808080u) - b4, u23 = (h23 | 0x80808080u) - b4;
        bad |= (u01 ^ 0x80808080u) | (u23 ^ 0x80808080u);
        X[t] = (u01 & 0x07070707u) | ((h01 >> 4) & 0x08080808u) | ((u23 & 0x07070707u) << 4) | (h23 & 0x80808080u);
    }
    u32 meta = base;
    if (__any((bad & 0xf8f8f8f8u) != 0u)) {
        u64 bm = 0;
#pragma unroll
        for (int i = 0; i < 64; ++i) {
            const u32 h7 = (V[i >> 1] >> (8 + 16 * (i & 1))) & 0x7fu;
            bm |= (u64)(h7 - base > 7u ? 1u : 0u) << i;
        }
        u32 n = 0;
        for (;;) {
            const u64 act = __ballot(bm != 0ull);
            if (!act) break;
            const u32 cnt = __builtin_popcountll(act);
            if (n + cnt > EXC_CAP) { n = EXC_CAP + 1u; break; }
            const u32 pos = n + __builtin_amdgcn_mbcnt_hi((u32)(act >> 32), __builtin_amdgcn_mbcnt_lo((u32)act, 0u));
            const u32 i = bm ? __builtin_ctzll(bm) : 0u;
            u32 dw = V[0];
#pragma unroll
            for (int q = 1; q < 32; ++q) dw = (i >> 1) == (u32)q ? V[q] : dw;
            const u32 val = (dw >> (16u * (i & 1u))) & 0xffffu;
            const u32 ent = (lane << 22) | (i << 16) | val;
            __builtin_amdgcn_raw_buffer_store_b32(ent, dst, bm ? EXC_REL + pos * 4u : OOB, slot_off, AUX_SYS);
            bm &= bm - 1ull;
            n += cnt;
        }
        meta = n > EXC_CAP ? META_RAW : (base | (n << 7));
    }
    if (meta & META_RAW) {
#pragma unroll
        for (int t = 0; t < 8; ++t) { O[t].x = V[4 * t]; O[t].y = V[4 * t + 1]; O[t].z = V[4 * t + 2]; O[t].w = V[4 * t + 3]; }
    } else {
#pragma unroll
        for (int t = 0; t < 4; ++t) { O[t].x = Lp[4 * t]; O[t].y = Lp[4 * t + 1]; O[t].z = Lp[4 * t + 2]; O[t].w = Lp[4 * t + 3]; }
#pragma unroll
        for (int t = 0; t < 2; ++t) { O[4 + t].x = X[4 * t]; O[4 + t].y = X[4 * t + 1]; O[4 + t].z = X[4 * t + 2]; O[4 + t].w = X[4 * t + 3]; }
    }
    return meta;
}
__device__ __forceinline__ void decode_task(u32x4* B, u32 meta, u32 exc, u32 lane) {
    const u32 b4 = (meta & 0x7fu) * 0x01010101u;
    u32 V[32];
#pragma unroll
    for (int t = 0; t < 8; ++t) {
        const u32 x = B[4 + (t >> 2)][t & 3], l01 = B[t >> 1][2 * (t & 1)], l23 = B[t >> 1][2 * (t & 1) + 1];
        const u32 he = ((x & 0x07070707u) + b4) | ((x << 4) & 0x80808080u);
        const u32 ho = (((x >> 4) & 0x07070707u) + b4) | (x & 0x80808080u);
        V[4 * t + 0] = __builtin_amdgcn_perm(he, l01, 0x05010400u);
        V[4 * t + 1] = __builtin_amdgcn_perm(he, l01, 0x07030602u);
        V[4 * t + 2] = __builtin_amdgcn_perm(ho, l23, 0x05010400u);
        V[4 * t + 3] = __builtin_amdgcn_perm(ho, l23, 0x07030602u);
    }
    const u32 cnt = (meta >> 7) & 15u;
    for (u32 e = 0; e < cnt; ++e) {
        const u32 ent = __builtin_amdgcn_readlane(exc, e);
        const u32 i = (ent >> 16) & 63u, sh = 16u * (i & 1u);
        const bool me = lane == (ent >> 22);
        const u32 keep = ~(0xffffu << sh), put = (ent & 0xffffu) << sh;
#pragma unroll
        for (int q = 0; q < 32; ++q) V[q] = (me && (i >> 1) == (u32)q) ? ((V[q] & keep) | put) : V[q];
    }
#pragma unroll
    for (int t = 0; t < 8; ++t) { B[t].x = V[4 * t]; B[t].y = V[4 * t + 1]; B[t].z = V[4 * t + 2]; B[t].w = V[4 * t + 3]; }
}

constexpr u32 META_TIGHT = 0x10000u;
constexpr u32 SGN_REL = 5120, LST_REL = 5632;
constexpr u32 SCR_BYTES = 4352;
__device__ __forceinline__ u32 wave_scan_incl(u32 v) {
    v += __builtin_amdgcn_update_dpp(0u, v, 0x111, 0xf, 0xf, false);
    v += __builtin_amdgcn_update_dpp(0u, v, 0x112, 0xf, 0xf, false);
    v += __builtin_amdgcn_update_dpp(0u, v, 0x114, 0xf, 0xf, false);
    v += __builtin_amdgcn_update_dpp(0u, v, 0x118, 0xf, 0xf, false);
    v += __builtin_amdgcn_update_dpp(0u, v, 0x142, 0xa, 0xf, false);
    v += __builtin_amdgcn_update_dpp(0u, v, 0x143, 0xc, 0xf, false);
    return v;
}
__device__ __forceinline__ u32 encode_tight(const u32* V, u32x4* O, u32x2& S, u32& list, LAS char* sc, u32 lane) {
    u32 m = V[0] & 0x7fff7fffu;
#pragma unroll
    for (int i = 1; i < 32; ++i) m = pkmax_u16(m, V[i] & 0x7fff7fffu);
    m = pkmax_u16(m, m >> 16) & 0xffffu;
    const u32 top = wave_max_u32(m) >> 8;
    const u32 base = top > 3u ? top - 3u : 0u;
    const u32 b4 = base * 0x01010101u;
    u32 U[16], C[4] = {0, 0, 0, 0}, Sg[2] = {0, 0}, E[2] = {0, 0};
#pragma unroll
    for (int k = 0; k < 16; ++k) {
        const int t = k >> 1, o = 2 * (k & 1);
        const u32 h = __builtin_amdgcn_perm(V[4 * t + o + 1], V[4 * t + o], 0x07050301u);
        O[k >> 2][k & 3] = __builtin_amdgcn_perm(V[4 * t + o + 1], V[4 * t + o], 0x06040200u);
        const u32 u = (h | 0x80808080u) - b4;
        const u32 e = (u ^ 0x80808080u) & 0xfcfcfcfcu;
        const u32 nz = (((e & 0x7c7c7c7cu) + 0x7c7c7c7cu) | e) & 0x80808080u;
        U[k] = u;
        C[k >> 2] |= (u & 0x03030303u) << (2 * (k & 3));
        Sg[k >> 3] |= (h & 0x80808080u) >> (7 - (k & 7));
        E[k >> 3] |= nz >> (7 - (k & 7));
    }
    const u32 n = __builtin_popcount(E[0]) + __builtin_popcount(E[1]);
    const u32 incl = wave_scan_incl(n);
    const u32 total = __builtin_amdgcn_readlane(incl, 63);
    if (total > 128u) return 0u;
    list = 0;
    if (total) {
#pragma unroll
        for (int j = 0; j < 4; ++j) { const u32x4 q = {U[4 * j], U[4 * j + 1], U[4 * j + 2], U[4 * j + 3]}; *(LAS u32x4*)(sc + j * 1024 + lane * 16) = q; }
        u32 pos = incl - n, e0 = E[0], e1 = E[1];
        bool far = false;
        while (__any((e0 | e1) != 0u)) {
            const bool has = (e0 | e1) != 0u;
            const u32 w = e0 ? 0u : 1u, mk = e0 ? e0 : e1;
            const u32 p = has ? __builtin_ctz(mk) : 0u;
            const u32 k = 8u * w + (p & 7u);
            const u32 ub = *(const LAS unsigned char*)(sc + (k >> 2) * 1024u + lane * 16u + (k & 3u) * 4u + (p >> 3));
            const u32 code = (0x7fu - ub) >> 2;
            if (has) {
                far |= code > 15u;
                *(LAS unsigned short*)(sc + 4096u + pos * 2u) = (unsigned short)((lane << 10) | ((w * 32u + p) << 4) | (code & 15u));
                ++pos;
            }
            if (e0) e0 &= e0 - 1u; else e1 &= e1 - 1u;
        }
        if (__any(far)) return 0u;
        list = *(const LAS u32*)(sc + 4096u + lane * 4u);
    }
    O[4].x = C[0]; O[4].y = C[1]; O[4].z = C[2]; O[4].w = C[3];
    S.x = Sg[0]; S.y = Sg[1];
    return base | META_TIGHT | (total << 20);
}
__device__ __forceinline__ void decode_tight(u32x4* B, u32x2 S, u32 list, u32 meta, LAS char* sc, u32 lane) {
    const u32 b4 = (meta & 0x7fu) * 0x01010101u;
    u32 H[16];
#pragma unroll
    for (int k = 0; k < 16; ++k)
        H[k] = (((B[4][k >> 2] >> (2 * (k & 3))) & 0x03030303u) + b4) | ((S[k >> 3] << (7 - (k & 7))) & 0x80808080u);
    const u32 total = (meta >> 20) & 0xffu;
    if (total) {
#pragma unroll
        for (int j = 0; j < 4; ++j) { const u32x4 q = {H[4 * j], H[4 * j + 1], H[4 * j + 2], H[4 * j + 3]}; *(LAS u32x4*)(sc + j * 1024 + lane * 16) = q; }
#pragma unroll
        for (int e = 0; e < 2; ++e) {
            const u32 ent = e ? list >> 16 : list & 0xffffu;
            const u32 p = (ent >> 4) & 31u, k = ((ent >> 9) & 1u) * 8u + (p & 7u);
            if (2u * lane + e < total)
                (void)__hip_atomic_fetch_sub((LAS u32*)(sc + (k >> 2) * 1024u + (ent >> 10) * 16u + (k & 3u) * 4u), (4u * ((ent & 15u) + 1u)) << (8u * (p >> 3)),
                                             __ATOMIC_RELAXED, __HIP_MEMORY_SCOPE_WORKGROUP);
        }
        asm volatile("" ::: "memory");
#pragma unroll
        for (int j = 0; j < 4; ++j) { const u32x4 q = *(const LAS u32x4*)(sc + j * 1024 + lane * 16); H[4 * j] = q.x; H[4 * j + 1] = q.y; H[4 * j + 2] = q.z; H[4 * j + 3] = q.w; }
    }
    u32 V[32];
#pragma unroll
    for (int t = 0; t < 8; ++t) {
        const u32 l01 = B[t >> 1][2 * (t & 1)], l23 = B[t >> 1][2 * (t & 1) + 1];
        V[4 * t + 0] = __builtin_amdgcn_perm(H[2 * t], l01, 0x05010400u);
        V[4 * t + 1] = __builtin_amdgcn_perm(H[2 * t], l01, 0x07030602u);
        V[4 * t + 2] = __builtin_amdgcn_perm(H[2 * t + 1], l23, 0x05010400u);
        V[4 * t + 3] = __builtin_amdgcn_perm(H[2 * t + 1], l23, 0x07030602u);
    }
#pragma unroll
    for (int t = 0; t < 8; ++t) { B[t].x = V[4 * t]; B[t].y = V[4 * t + 1]; B[t].z = V[4 * t + 2]; B[t].w = V[4 * t + 3]; }
}

__device__ __forceinline__ bool task_is_w(u32 u) { return (u % 96u) < 32u; }
__device__ __forceinline__ u32 task_xb(u32 src, u32 dst, u32 u) { return (u % 96u - 32u + ((src - dst) & 7u) * 8u) & 63u; }
__device__ __forceinline__ u32 task_slot(u32 src, u32 dst, u32 u) {
    const u32 st = src * 8u + u / 96u;
    return task_is_w(u) ? WS_REL + (st * 64u + 2u * (u % 96u)) * BLK : XS_REL + (st * 128u + 2u * task_xb(src, dst, u)) * BLK;
}
__device__ __forceinline__ u32 task_rdy(u32 src, u32 dst, u32 u) {
    const u32 st = src * 8u + u / 96u;
    if (task_is_w(u)) { const u32 wb = u % 96u; return RDW_OFF + ((wb >> 1) * 64u + st) * 8u + (wb & 1u) * 4u; }
    const u32 xb = task_xb(src, dst, u);
    return RDX_OFF + ((xb / 5u) * 64u + st) * 32u + (xb % 5u) * 4u;
}
__device__ __forceinline__ u32 task_pck(u32 src, u32 dst, u32 u) {
    const u32 j = u / 96u;
    return task_is_w(u) ? 512u + (dst * 8u + j) * 32u + u % 96u : j * 64u + task_xb(src, dst, u);
}
__device__ __forceinline__ u32 raw_lane_off(bool is_w, u32 lane) {
    return is_w ? ((lane >> 4) & 1u) * 4096u + (lane >> 5) * 512u + (lane & 15u) * 16u : (lane & 7u) * 512u + (lane >> 3) * 16u;
}
__device__ __forceinline__ u32 raw_vec_off(bool is_w, int v) { return is_w ? (v >> 1) * 1024u + (v & 1) * 256u : (v >> 2) * 4096u + (v & 3) * 128u; }

constexpr u32 NDEC = NGEMM + NDECWG * 8u;
__device__ __forceinline__ void decoder(char* myws, u32 rank, u32 call, u32 di, u32 lane, LAS char* sc) {
    const rsrc_t ws = mkr(myws, WS_USED);
    GAS u32* const tmo = (GAS u32*)(myws + TMO_OFF);
    const u32 sv = lane * 16u;
    const u32 fv = lane == 0 ? 0u : OOB;
    u32 owed = 0;
    constexpr u32 QE = 7u * (NTASK - NRAW);
    for (u32 q = di; q < QE;) {
        const u32 qn = q + NDEC;
        const u32 u = q / 7u, src = (rank + 1u + q % 7u) & 7u;
        GAS u64* arr = (GAS u64*)(myws + ARR_OFF + (src * NTASK + u) * 64u);
        const u64 t0 = rt();
        u64 f;
        for (u32 spins = 0;; ++spins) {
            f = __hip_atomic_load(arr, __ATOMIC_RELAXED, __HIP_MEMORY_SCOPE_SYSTEM);
            const u32 seen = rfl((u32)(f >> 32));
            if (owed) { VM0(); __builtin_amdgcn_raw_buffer_store_b32(call, ws, fv, owed, AUX_SYS); owed = 0; }
            if (seen >= call) break;
            if ((spins & 15u) == 15u && give_up(tmo, call, t0)) break;
            __builtin_amdgcn_s_sleep(1);
        }
        asm volatile("" ::: "memory");
        const u32 meta = rfl((u32)f);
        const u32 so = task_slot(src, rank, u);
        u32x4 B[8];
#pragma unroll
        for (int i = 0; i < 5; ++i) B[i] = __builtin_amdgcn_raw_buffer_load_b128(ws, sv + i * 1024, PST_OFF + so, AUX_SYS);
        if (meta & META_TIGHT) {
            const u32x2 S = __builtin_amdgcn_raw_buffer_load_b64(ws, lane * 8u + SGN_REL, PST_OFF + so, AUX_SYS);
            const u32 list = __builtin_amdgcn_raw_buffer_load_b32(ws, lane * 4u + LST_REL, PST_OFF + so, AUX_SYS);
            decode_tight(B, S, list, meta, sc, lane);
        } else {
            B[5] = __builtin_amdgcn_raw_buffer_load_b128(ws, sv + 5120, PST_OFF + so, AUX_SYS);
            if (meta & META_RAW) {
                B[6] = __builtin_amdgcn_raw_buffer_load_b128(ws, sv + 6144, PST_OFF + so, AUX_SYS);
                B[7] = __builtin_amdgcn_raw_buffer_load_b128(ws, sv + 7168, PST_OFF + so, AUX_SYS);
            } else {
                u32 exc = 0;
                if (meta & 0x780u) exc = __builtin_amdgcn_raw_buffer_load_b32(ws, lane * 4u + EXC_REL, PST_OFF + so, AUX_SYS);
                decode_task(B, meta, exc, lane);
            }
        }
        const bool isw = task_is_w(u);
        const u32 lo = raw_lane_off(isw, lane) + RAW_OFF + so;
#pragma unroll
        for (int i = 0; i < 8; ++i) st128(B[i], ws, lo + raw_vec_off(isw, i));
        owed = task_rdy(src, rank, u);
        if (qn >= QE || u >= NTASK - 96u) {
            VM0(); __builtin_amdgcn_raw_buffer_store_b32(call, ws, fv, owed, AUX_SYS); owed = 0;
        }
        q = qn;
    }
    if (owed) { VM0(); __builtin_amdgcn_raw_buffer_store_b32(call, ws, fv, owed, AUX_SYS); }
}


__global__ void __launch_bounds__(512) gemm_ag_q8(Params P) {
    __shared__ __attribute__((aligned(16))) char smem_[LDS_BYTES];
    LAS char* lds = (LAS char*)smem_;
    const int tid = threadIdx.x;
    const u32 lane = tid & 63;
    const u32 wave = rfl(tid >> 6);
    const u32 b = blockIdx.x;
    const u32 rank = P.rank;
    const u32 xcd = b & 7, idx = b >> 3;
    char* const myws = P.ws[rank];
    GAS u32* const ctr = (GAS u32*)(myws + CTR_OFF);
    GAS u32* const tmo = (GAS u32*)(myws + TMO_OFF);

    if (idx >= 26u) {
        const u32 ci = (idx - 26u) * 8u + xcd;
        if (ci >= NCOMM - NDECWG) {
            const u32 call = rfl(__hip_atomic_load(ctr, __ATOMIC_RELAXED, __HIP_MEMORY_SCOPE_AGENT)) + 1u;
            decoder(myws, rank, call, NGEMM + (ci - (NCOMM - NDECWG)) * 8u + wave, lane, lds + wave * SCR_BYTES);
            return;
        }
        if (wave == NPUSH) {
            if (ci != 0) return;
            const u32 call = rfl(__hip_atomic_load(ctr, __ATOMIC_RELAXED, __HIP_MEMORY_SCOPE_AGENT)) + 1u;
            const rsrc_t ws = mkr(myws, WS_USED);
            const u32 off = LMX_OFF + lane * 32u;
            const bool has = lane * 4u < NGEMM;
            const u64 t1 = rt();
            u32 gm = 0;
            for (u32 spins = 1;; ++spins) {
                const u32x4 a = __builtin_amdgcn_raw_buffer_load_b128(ws, off, 0, AUX_SYS);
                const u32x4 c = __builtin_amdgcn_raw_buffer_load_b128(ws, off + 16, 0, AUX_SYS);
                if (__all(!has || (a.y >= call && a.w >= call && c.y >= call && c.w >= call))) {
                    const u32 m1 = a.x > a.z ? a.x : a.z, m2 = c.x > c.z ? c.x : c.z;
                    gm = has ? (m1 > m2 ? m1 : m2) : 0u;
                    break;
                }
                if ((spins & 15u) == 0u && give_up(tmo, call, t1)) break;
                __builtin_amdgcn_s_sleep(1);
            }
            gm = wave_max_u32(gm);
            const u64 g = ((u64)call << 32) | gm;
            __hip_atomic_store((GAS u64*)(P.ws[lane >> 3] + AMX_OFF + (lane & 7) * 128 + rank * 8), g, __ATOMIC_RELAXED, __HIP_MEMORY_SCOPE_SYSTEM);
            return;
        }
        if (wave > NPUSH) return;
        const u32 sid = ci * NPUSH + wave;
        if (sid >= 7u * KPUSH) return;
        const u32 d = (rank + 1u + sid % 7u) & 7u, k = sid / 7u;
        const rsrc_t ws = mkr(myws, WS_USED), dst = mkr(P.ws[d], WS_USED);
        const u32 sv = lane * 16u;
        const u32 fv = lane == 0 ? 0u : OOB;
        const rsrc_t xin = mkr(P.x, X_BYTES), win = mkr(P.w, W_BYTES);
        const u32 c16 = lane & 15u;
        const u32 wv = (lane >> 5) * 262144u + ((lane & 16u) + 8u * ((c16 >> 2) & 1u) + 4u * (c16 >> 3) + (c16 & 3u)) * 4u;
        u32 R[32];
#pragma unroll
        for (int i = 0; i < 32; ++i) R[i] = __builtin_amdgcn_raw_buffer_load_b32(win, wv, (2048u * d + 64u * k) * 2u + ((i >> 3) * 16u + (i & 7)) * 32768u, 0);
        const u32 call = rfl(__hip_atomic_load(ctr, __ATOMIC_RELAXED, __HIP_MEMORY_SCOPE_AGENT)) + 1u;
        GAS u32* const que = (GAS u32*)(myws + QUE_OFF + d * 128u);
        const u32 qbase = (call - 1u) * (NTASK - 4u * KPUSH) - 5u * KPUSH;
        auto prd_of = [&](u32 u) { return (GAS u64*)(myws + PRD_OFF + task_pck(rank, d, u) * 64u); };
        auto settle = [&](u32 u, u32 lo, u32 hi) {
            if (hi >= call) return lo;
            const u64 t0 = rt();
            for (u32 spins = 1;; ++spins) {
                const u64 f = __hip_atomic_load(prd_of(u), __ATOMIC_RELAXED, __HIP_MEMORY_SCOPE_SYSTEM);
                if (rfl((u32)(f >> 32)) >= call || ((spins & 15u) == 0u && give_up(tmo, call, t0))) return rfl((u32)f);
                __builtin_amdgcn_s_sleep(1);
            }
        };
        auto load_task = [&](u32x4* B, u32 u) {
            const u32 so = u >= NTASK - NRAW ? RAW_OFF + task_slot(rank, d, u) : PCK_OFF + task_pck(rank, d, u) * 8192u;
#pragma unroll
            for (int i = 0; i < 8; ++i) B[i] = __builtin_amdgcn_raw_buffer_load_b128(ws, sv + (i & 3) * 1024, so + (i >> 2) * 4096, AUX_SYS);
        };
        u32x4 A[8], B[8];
        static_assert(3 * KPUSH <= 96 && KPUSH <= 32, "the tasks packed by the pushers lie in k slice 0");
        u32 c, n1, metac;
        {
            const u32 xv = (lane >> 3) * 1024u + (lane & 7u) * 16u;
            u32 V[32];
            auto fetch = [&](u32 u) {
                if (u < 32u) {
                    const u32 base = (2048u * d + 64u * u) * 2u;
#pragma unroll
                    for (int i = 0; i < 32; ++i) R[i] = __builtin_amdgcn_raw_buffer_load_b32(win, wv, base + ((i >> 3) * 16u + (i & 7)) * 32768u, 0);
                } else {
                    const u32 base = task_xb(rank, d, u) * 65536u;
#pragma unroll
                    for (int i = 0; i < 8; ++i) {
                        const u32x4 v = __builtin_amdgcn_raw_buffer_load_b128(xin, xv, base + i * 8192u, 0);
                        R[4 * i] = v.x; R[4 * i + 1] = v.y; R[4 * i + 2] = v.z; R[4 * i + 3] = v.w;
                    }
                }
            };
            auto arrange = [&](u32 u) {
                if (u < 32u) {
#pragma unroll
                    for (int g = 0; g < 4; ++g)
#pragma unroll
                        for (int q = 0; q < 4; ++q) {
                            V[8 * g + q] = __builtin_amdgcn_perm(R[8 * g + 2 * q + 1], R[8 * g + 2 * q], 0x05040100u);
                            V[8 * g + 4 + q] = __builtin_amdgcn_perm(R[8 * g + 2 * q + 1], R[8 * g + 2 * q], 0x07060302u);
                        }
                } else {
#pragma unroll
                    for (int i = 0; i < 32; ++i) V[i] = R[i];
                }
            };
            auto send = [&](u32x4* O, u32 u, u32 meta) {
                const u32 slot = PST_OFF + task_slot(rank, d, u);
#pragma unroll
                for (int i = 0; i < 6; ++i) st128(O[i], dst, sv + slot + i * 1024u);
                if (meta & META_RAW) { st128(O[6], dst, sv + slot + 6144u); st128(O[7], dst, sv + slot + 7168u); }
            };
            auto flag = [&](u32 u, u32 meta) {
                VM0();
                u32x2 f; f.x = meta; f.y = call;
                __builtin_amdgcn_raw_buffer_store_b64(f, dst, fv, ARR_OFF + (rank * NTASK + u) * 64u, AUX_SYS);
            };
            const u32 u0 = k, u1 = k + KPUSH, u2 = k + 2u * KPUSH;
            arrange(u0); fetch(u1);
            const u32 m0 = encode_task(V, A, dst, PST_OFF + task_slot(rank, d, u0), lane);
            send(A, u0, m0);
            arrange(u1); fetch(u2);
            const u32 m1 = encode_task(V, B, dst, PST_OFF + task_slot(rank, d, u1), lane);
            flag(u0, m0);
            send(B, u1, m1);
            arrange(u2);
            const u32 m2 = encode_task(V, A, dst, PST_OFF + task_slot(rank, d, u2), lane);
            flag(u1, m1);
            send(A, u2, m2);
            c = k + 3u * KPUSH; n1 = k + 4u * KPUSH;
            metac = settle(c, 0u, 0u);
            asm volatile("" ::: "memory");
            load_task(B, c);
            flag(u2, m2);
            asm volatile("s_waitcnt vmcnt(0)" : "+v"(B[0]), "+v"(B[1]), "+v"(B[2]), "+v"(B[3]), "+v"(B[4]), "+v"(B[5]), "+v"(B[6]), "+v"(B[7]) :: "memory");
        }

        u32 fa = OOB, fb = OOB, oa = 0, ob = 0;
        u32x2 fo; fo.x = 0; fo.y = call;
#define ST1(CUR, i) do { \
            if ((i) < 5) st128(CUR[i], dst, so + (i) * 1024u); \
            else if ((i) == 5) st128(CUR[5], dst, (!full && (metac & META_TIGHT) && lane >= 48u) ? OOB : so + 5120u); \
            else if ((i) == 6) st128(CUR[6], dst, (full || ((metac & 0x780u) && lane < 2u)) ? so + 6144u : OOB); \
            else st128(CUR[7], dst, full ? so + 7168u : OOB); } while (0)
#define PUSH(CUR, NXT) do { \
            const bool tail = c >= NTASK - NRAW; \
            const bool full = tail || (metac & META_RAW); \
            const u32 so = sv + (tail ? RAW_OFF : PST_OFF) + task_slot(rank, d, c); \
            const bool v1 = n1 < NTASK; \
            GAS u64* const pa = prd_of(v1 ? n1 : 0u); \
            u32 tk = 1u, meta1 = 0; \
            u64 pk; \
            asm volatile("global_load_dwordx2 %0, %1, off sc0 sc1" : "=v"(pk) : "v"(pa) : "memory"); \
            __builtin_amdgcn_raw_buffer_store_b64(fo, dst, fa, oa, AUX_SYS); \
            __builtin_amdgcn_raw_buffer_store_b32(call, dst, fb, ob, AUX_SYS); \
            _Pragma("unroll") for (int i = 0; i < PB; ++i) ST1(CUR, i); \
            asm volatile("s_waitcnt vmcnt(" STR(PB2) ")" : "+v"(pk) :: "memory"); \
            if (v1) { \
                meta1 = settle(n1, rfl((u32)pk), rfl((u32)(pk >> 32))); \
                asm volatile("" ::: "memory"); \
                load_task(NXT, n1); \
                if (PB < 8 && n1 < NTASK - KPUSH) asm volatile("s_waitcnt vmcnt(8)" ::: "memory");        \
                _Pragma("unroll") for (int i = PB; i < 8; ++i) ST1(CUR, i); \
                u64 sv_; \
                asm volatile("s_mov_b64 %1, exec\n\ts_mov_b64 exec, 1\n\tglobal_atomic_add %0, %2, %0, off sc0\n\ts_mov_b64 exec, %1\n\t" \
                             "s_waitcnt vmcnt(0)" : "+v"(tk), "=&s"(sv_) : "v"(que) : "memory"); \
                asm volatile("" : "+v"(NXT[0]), "+v"(NXT[1]), "+v"(NXT[2]), "+v"(NXT[3]), "+v"(NXT[4]), "+v"(NXT[5]), "+v"(NXT[6]), \
                             "+v"(NXT[7]) :: "memory");            \
            } else { \
                _Pragma("unroll") for (int i = PB; i < 8; ++i) ST1(CUR, i); \
                VM0(); \
            } \
            fo.x = metac; \
            fa = tail ? OOB : fv; fb = tail ? fv : OOB; \
            oa = ARR_OFF + (rank * NTASK + c) * 64u; ob = task_rdy(rank, d, c); \
            c = n1; metac = meta1; \
            if (v1) n1 = rfl(tk) - qbase; \
            } while (0)
        for (;;) {
            PUSH(B, A);
            if (c >= NTASK) break;
            PUSH(A, B);
            if (c >= NTASK) break;
        }
        __builtin_amdgcn_raw_buffer_store_b64(fo, dst, fa, oa, AUX_SYS);
        __builtin_amdgcn_raw_buffer_store_b32(call, dst, fb, ob, AUX_SYS);
        VM0();
        return;
    }

    const u32 ti4 = (xcd & 1u) * 26u + idx;
    const u32 mt = ti4 >> 2;
    const u32 nt = (xcd >> 1) * 4u + (ti4 & 3u);
    const u32 gi = mt * 16u + nt;
    if (tid < (int)((L_SCR - L_CTL) / 4)) *(LAS u32*)(lds + L_CTL + tid * 4) = 0u;
    const u32 call = rfl(__hip_atomic_load(ctr, __ATOMIC_RELAXED, __HIP_MEMORY_SCOPE_AGENT)) + 1u;
    LGKM0();
    __builtin_amdgcn_s_barrier();

    if (wave == 7) {
        decoder(myws, rank, call, gi, lane, lds + L_SCR + 3 * SCR_BYTES);
        return;
    }
    if (wave == 6) {
        const rsrc_t ws = mkr(myws, WS_USED);
        const u32 fx = RDX_OFF + (mt * 64u + lane) * 32u, fw = RDW_OFF + (nt * 64u + lane) * 8u;
        const bool five = mt < MT - 1;
        u64 pending = ~0ull;
        u32 n = 0;
        const u64 t0 = rt();
        u32 spins = 0;
        while (pending) {
            const bool mine = (pending >> lane) & 1ull;
            u32x4 a = {0, 0, 0, 0};
            u32 a4 = 0;
            u32x2 cc = {0, 0};
            if (mine) {
                a = __builtin_amdgcn_raw_buffer_load_b128(ws, fx, 0, AUX_SYS);
                a4 = __builtin_amdgcn_raw_buffer_load_b32(ws, fx + 16, 0, AUX_SYS);
                cc = __builtin_amdgcn_raw_buffer_load_b64(ws, fw, 0, AUX_SYS);
            }
            const bool ready = mine && a.x >= call && a.y >= call && a.z >= call && a.w >= call && (a4 >= call || !five) && cc.x >= call && cc.y >= call;
            u64 mask = __ballot(ready);
            if (!mask) {
                if ((++spins & 15u) == 0u && give_up(tmo, call, t0)) mask = pending;
                else { __builtin_amdgcn_s_sleep(1); continue; }
            }
            if ((mask >> lane) & 1ull) {
                const u32 pos = n + __builtin_amdgcn_mbcnt_hi((u32)(mask >> 32), __builtin_amdgcn_mbcnt_lo((u32)mask, 0u));
                *(LAS u32*)(lds + L_ORD + pos * 4) = lane;
            }
            LGKM0();
            n += __builtin_popcountll(mask);
            pending &= ~mask;
            if (lane == 0) lds_st(lds, L_OCNT, n);
        }
        const u32 row0 = 320u * mt;
        const u32 nrow8 = mt == MT - 1 ? 32u : 40u;
        for (u32 gq = 0; gq < 40u; ++gq) {
            const u32 i = gq >> 3, g = (gq & 1u) * 20u + 4u * i + ((gq >> 1) & 3u);
            if ((gq & 7u) == 0u) lds_wait_ge(lds, L_OUT + i * 4u, 4u);
            if (g >= nrow8) continue;
            const u32 row = 8 * g + (lane >> 3), ch = lane & 7;
            const u32x4 v = *(const LAS u32x4*)(lds + row * 128 + ((ch ^ (row & 7)) * 16));
            const u32 grow = row0 + 8 * g;
            const rsrc_t ob = mkr(P.out[(grow >> 9) & 7u], OUT_BYTES);
            st128(v, ob, ((grow & 511u) + (lane >> 3)) * NN + rank * NL + nt * 128 + ch * 16);
            asm volatile("s_waitcnt vmcnt(" STR(OUTWIN1) ")" ::: "memory");
        }
        VM0();
        const u32 d0 = row0 >> 9, d1 = (row0 + 8u * nrow8 - 1u) >> 9;
        if (lane <= d1 - d0) {
            const u32 d = (d0 + lane) & 7u;
            __hip_atomic_store((GAS u32*)(P.ws[d] + DON_OFF + ((rank * 3u + (mt - (512u * d) / 320u)) * 16u + nt) * 128u), call, __ATOMIC_RELAXED, __HIP_MEMORY_SCOPE_SYSTEM);
        }
        const u32 cnt = (512u * rank + 511u) / 320u - (512u * rank) / 320u + 1u;
        for (u32 e = gi; e < 8u * cnt * 16u; e += NGEMM) {
            GAS u32* f = (GAS u32*)(myws + DON_OFF + (((e / (cnt * 16u)) * 3u + (e / 16u) % cnt) * 16u + e % 16u) * 128u);
            const u64 t1 = rt();
            for (u32 sp = 1; rfl(__hip_atomic_load(f, __ATOMIC_RELAXED, __HIP_MEMORY_SCOPE_SYSTEM)) < call; ++sp) {
                if ((sp & 15u) == 0u && give_up(tmo, call, t1)) {
                    if (lane == 0) __hip_atomic_store((GAS unsigned char*)(P.out[rank] + rank * NL), (unsigned char)0x80, __ATOMIC_RELAXED, __HIP_MEMORY_SCOPE_SYSTEM);
                    break;
                }
                __builtin_amdgcn_s_sleep(1);
            }
        }
        return;
    }
    if (wave >= 4) {
        const u32 half = wave - 4;
        const char* xsrc = myws + RAW_OFF + XS_REL + (10u * mt) * BLK + half * 2048u + lane * 16u;
        const char* wsrc = myws + RAW_OFF + WS_REL + (4u * nt) * BLK + half * 2048u + lane * 16u;
        const u32 lbase = (u32)(uintptr_t)lds;
        auto rd = [&](u32 off) {
            u32 v;
            asm volatile("ds_read_b32 %0, %1\n\ts_waitcnt lgkmcnt(0)" : "=v"(v) : "v"(lbase + off) : "memory");
            return rfl(v);
        };
        u32 ni = 0, islot = half, iuse = 0;
        u32 np = 0, pslot = half;
        while (np < 64) {
            bool can = ni < 64 && ni - np < 2u && rd(L_OCNT) > ni;
            if (can) can = rd(L_FREE + islot * 4) >= 4u * iuse;
            if (can) {
                const u32 s_ = rd(L_ORD + ni * 4) & 63u;
                const char* gx = xsrc + (size_t)s_ * (128u * BLK);
                const char* gw = wsrc + (size_t)s_ * (64u * BLK);
                LAS char* sb = lds + islot * SLOT;
#define DMA2(g, l) do { \
                __builtin_amdgcn_global_load_lds((const GAS u32*)(g), (LAS u32*)(l), 16, 0, AUX_SYS); \
                __builtin_amdgcn_global_load_lds((const GAS u32*)(g), (LAS u32*)(l), 16, 1024, AUX_SYS); } while (0)
#pragma unroll
                for (int i = 0; i < 10; ++i) DMA2(gx + i * BLK, sb + i * 2048);
#pragma unroll
                for (int i = 0; i < 4; ++i) DMA2(gw + i * BLK, sb + (10 + i) * 2048);
                ++ni;
                islot += 2; if (islot >= NSLOT) { islot -= NSLOT; ++iuse; }
            } else if (np < ni) {
                if (ni - np == 2) asm volatile("s_waitcnt vmcnt(28)" ::: "memory"); else VM0();
                if (lane == 0) asm volatile("ds_add_u32 %0, %1" :: "v"(lbase + L_FULL + pslot * 4), "v"(1u) : "memory");
                ++np;
                pslot += 2; if (pslot >= NSLOT) pslot -= NSLOT;
            } else {
                __builtin_amdgcn_s_sleep(1);
            }
        }
        return;
    }

    {
        const rsrc_t xin = mkr(P.x, X_BYTES), win = mkr(P.w, W_BYTES), ws = mkr(myws, WS_USED);
        const u32 xv = (lane >> 3) * 1024u + (lane & 7u) * 16u;
        const u32 c16 = lane & 15u;
        const u32 wv = (lane >> 5) * 262144u + ((lane & 16u) + 8u * ((c16 >> 2) & 1u) + 4u * (c16 >> 3) + (c16 & 3u)) * 4u;
        const u32 sv = lane * 16u;
        const u32 fv = lane == 0 ? 0u : OOB;
        for (u32 e = wave * NGEMM + gi; wave < 3u && e < 2560u; e += 3u * NGEMM) {
            const u32 j = e / 320u, r = e % 320u;
            if (e < 3u * NGEMM) for (u32 i = 0; i < j; ++i) __builtin_amdgcn_s_sleep(STAGGER);
            const bool isw = r < 256u;
            const u32 d = isw ? (r & 7u) : rank, ti = isw ? (r >> 3) : r - 256u;
            if (isw && j == 0 && ti < 3u * KPUSH && d != rank) continue;
            u32 V[32];
            if (isw) {
                u32 R[32];
                const u32 base = (64u * j * 16384u + 2048u * d + 64u * ti) * 2u;
#pragma unroll
                for (int i = 0; i < 32; ++i)
                    R[i] = __builtin_amdgcn_raw_buffer_load_b32(win, wv, base + ((i >> 3) * 16u + (i & 7)) * 32768u, 0);
#pragma unroll
                for (int g = 0; g < 4; ++g)
#pragma unroll
                    for (int q = 0; q < 4; ++q) {
                        V[8 * g + q] = __builtin_amdgcn_perm(R[8 * g + 2 * q + 1], R[8 * g + 2 * q], 0x05040100u);
                        V[8 * g + 4 + q] = __builtin_amdgcn_perm(R[8 * g + 2 * q + 1], R[8 * g + 2 * q], 0x07060302u);
                    }
            } else {
                const u32 base = ti * 65536u + j * 128u;
#pragma unroll
                for (int i = 0; i < 8; ++i) {
                    const u32x4 v = __builtin_amdgcn_raw_buffer_load_b128(xin, xv, base + i * 8192u, 0);
                    V[4 * i] = v.x; V[4 * i + 1] = v.y; V[4 * i + 2] = v.z; V[4 * i + 3] = v.w;
                }
            }
            const u32 st = rank * 8u + j;
            if (d == rank) {
                const u32 rel = isw ? WS_REL + (st * 64u + 2u * ti) * BLK : XS_REL + (st * 128u + 2u * ti) * BLK;
                const u32 lo = raw_lane_off(isw, lane) + RAW_OFF + rel;
#pragma unroll
                for (int i = 0; i < 8; ++i) { const u32x4 v = {V[4 * i], V[4 * i + 1], V[4 * i + 2], V[4 * i + 3]}; st128(v, ws, lo + raw_vec_off(isw, i)); }
            }
            u32 meta = 0;
            const u32 pk = isw ? 512u + (d * 8u + j) * 32u + ti : j * 64u + ti;
            const bool pack = !isw || d != rank;
            if (pack) {
                u32x4 O[8];
                const u32 slot = PCK_OFF + pk * 8192u, so = sv + slot;
                u32x2 S; u32 list;
                meta = encode_tight(V, O, S, list, lds + L_SCR + wave * SCR_BYTES, lane);
                if (meta) {
#pragma unroll
                    for (int i = 0; i < 5; ++i) st128(O[i], ws, so + i * 1024u);
                    __builtin_amdgcn_raw_buffer_store_b64(S, ws, lane * 8u + SGN_REL, slot, AUX_SYS);
                    __builtin_amdgcn_raw_buffer_store_b32(list, ws, lane * 4u + LST_REL, slot, AUX_SYS);
                } else {
                    meta = encode_task(V, O, ws, slot, lane);
#pragma unroll
                    for (int i = 0; i < 6; ++i) st128(O[i], ws, so + i * 1024u);
                    if (meta & META_RAW) { st128(O[6], ws, so + 6144u); st128(O[7], ws, so + 7168u); }
                }
            }
            VM0();
            if (pack) { u32x2 f; f.x = meta; f.y = call; __builtin_amdgcn_raw_buffer_store_b64(f, ws, fv, PRD_OFF + pk * 64u, AUX_SYS); }
            if (d == rank) {
                const u32 rdy = isw ? RDW_OFF + ((ti >> 1) * 64u + st) * 8u + (ti & 1u) * 4u : RDX_OFF + ((ti / 5u) * 64u + st) * 32u + (ti % 5u) * 4u;
                __builtin_amdgcn_raw_buffer_store_b32(call, ws, fv, rdy, AUX_SYS);
            }
        }
    }
    const u32 wm = wave >> 1, wn = wave & 1;
    f32x16 acc[2][5];
#pragma unroll
    for (int f = 0; f < 2; ++f)
#pragma unroll
        for (int i = 0; i < 5; ++i)
#pragma unroll
            for (int r = 0; r < 16; ++r) acc[f][i][r] = 0.f;
    {
        const LAS char* xb0 = lds + (5 * wm) * 2048 + lane * 16;
        const LAS char* wb0 = lds + (10 + 2 * wn) * 2048 + lane * 16;
        u32x4 F0[7], F1[7];
#define MM(F) _Pragma("unroll") for (int i = 0; i < 5; ++i) _Pragma("unroll") for (int f = 0; f < 2; ++f) \
            acc[f][i] = __builtin_amdgcn_mfma_f32_32x32x16_bf16(__builtin_bit_cast(bf16x8, F[5 + f]), __builtin_bit_cast(bf16x8, F[i]), acc[f][i], 0, 0, 0)
#define SB() __builtin_amdgcn_sched_barrier(0)
#define MF(F, i, f) acc[f][i] = __builtin_amdgcn_mfma_f32_32x32x16_bf16(__builtin_bit_cast(bf16x8, F[5 + f]), __builtin_bit_cast(bf16x8, F[i]), acc[f][i], 0, 0, 0)
#define RX(F, i, sl, t) F[i] = *(const LAS u32x4*)(xb0 + (sl) + (i) * 2048 + (t) * 1024)
#define RW(F, f, sl, t) F[5 + f] = *(const LAS u32x4*)(wb0 + (sl) + (f) * 2048 + (t) * 1024)
        const u32 lbase = (u32)(uintptr_t)lds;
        u32 slot = 0, use = 1;
        lds_wait_ge(lds, L_FULL, 1u);
        SB();
        RW(F0, 0, 0u, 0); SB(); RX(F0, 0, 0u, 0); SB(); RW(F0, 1, 0u, 0); SB(); RX(F0, 1, 0u, 0); SB();
        RX(F0, 2, 0u, 0); SB(); RX(F0, 3, 0u, 0); SB(); RX(F0, 4, 0u, 0); SB();
        u32 cur = 0, nxt = 1;
        for (u32 h = 0; h < 127; ++h) {
            const u32 co = cur * SLOT, no = nxt * SLOT;
            SB();
            MF(F0, 0, 0); const u32 fl = lds_ld(lds, L_FULL + nxt * 4); RW(F1, 0, co, 1); SB();
            MF(F0, 0, 1); RX(F1, 0, co, 1); SB();
            MF(F0, 1, 0); RW(F1, 1, co, 1); SB();
            MF(F0, 1, 1); RX(F1, 1, co, 1); SB();
            MF(F0, 2, 0); RX(F1, 2, co, 1); SB();
            MF(F0, 2, 1); RX(F1, 3, co, 1); SB();
            MF(F0, 3, 0); RX(F1, 4, co, 1); SB();
            MF(F0, 3, 1); SB();
            MF(F0, 4, 0); SB();
            MF(F0, 4, 1); SB();
            if (rfl(fl) < use) lds_wait_ge(lds, L_FULL + nxt * 4, use);
            asm volatile("" ::: "memory");
            SB();
            MF(F1, 0, 0); RW(F0, 0, no, 0); SB();
            MF(F1, 0, 1); RX(F0, 0, no, 0); SB();
            MF(F1, 1, 0); RW(F0, 1, no, 0); SB();
            MF(F1, 1, 1); RX(F0, 1, no, 0); SB();
            MF(F1, 2, 0); RX(F0, 2, no, 0); SB();
            {
                u64 sv_;
                asm volatile("s_waitcnt lgkmcnt(5)\n\ts_mov_b64 %0, exec\n\ts_mov_b64 exec, 1\n\tds_add_u32 %1, %2\n\ts_mov_b64 exec, %0"
                             : "=&s"(sv_) : "v"(lbase + L_FREE + cur * 4), "v"(1u) : "memory");
            }
            SB();
            MF(F1, 2, 1); RX(F0, 3, no, 0); SB();
            MF(F1, 3, 0); RX(F0, 4, no, 0); SB();
            MF(F1, 3, 1); SB();
            MF(F1, 4, 0);
            cur = nxt;
            if (++nxt == NSLOT) { nxt = 0; ++use; }
            SB();
            MF(F1, 4, 1); SB();
        }
        slot = cur;
        const u32 slot_last = slot * SLOT;
        RW(F1, 0, slot_last, 1); RW(F1, 1, slot_last, 1);
#pragma unroll
        for (int i = 0; i < 5; ++i) RX(F1, i, slot_last, 1);
        MM(F0);
        MM(F1);
    }
    const u32 ngrp = (mt == MT - 1 && wm == 1) ? 3u : 5u;
    float mx = 0.f;
#pragma unroll
    for (int i = 0; i < 5; ++i) {
        float m = 0.f;
#pragma unroll
        for (int f = 0; f < 2; ++f)
#pragma unroll
            for (int r = 0; r < 16; ++r) m = fmaxf(m, fabsf(acc[f][i][r]));
        mx = (u32)i < ngrp ? fmaxf(mx, m) : mx;
    }
    const u32 mb = wave_max_u32(__float_as_uint(mx));
    u32 arrived = 0;
    if (lane == 0) {
        (void)__hip_atomic_fetch_max((LAS u32*)(lds + L_WMAX), mb, __ATOMIC_RELAXED, __HIP_MEMORY_SCOPE_WORKGROUP);
        arrived = __hip_atomic_fetch_add((LAS u32*)(lds + L_WCNT), 1u, __ATOMIC_RELAXED, __HIP_MEMORY_SCOPE_WORKGROUP);
    }
    arrived = rfl(arrived);
    if (arrived == 3) {
        const u32 wmax = rfl(lds_ld(lds, L_WMAX));
        if (lane == 0)
            __hip_atomic_store((GAS u64*)(myws + LMX_OFF + gi * 8), ((u64)call << 32) | wmax, __ATOMIC_RELAXED, __HIP_MEMORY_SCOPE_AGENT);
    }
    if (wave == 0) {
        GAS u64* g = (GAS u64*)(myws + AMX_OFF + xcd * 128 + (lane & 7) * 8);
        const u64 t0 = rt();
        u32 gm = 0, bad = 0;
        for (u32 spins = 1;; ++spins) {
            const u64 v = __hip_atomic_load(g, __ATOMIC_RELAXED, __HIP_MEMORY_SCOPE_SYSTEM);
            const u32 tm = __hip_atomic_load(tmo, __ATOMIC_RELAXED, __HIP_MEMORY_SCOPE_AGENT);
            bad = rfl(tm) == call;
            if (bad || __all((u32)(v >> 32) >= call)) { gm = (u32)v; break; }
            if ((spins & 15u) == 0u && give_up(tmo, call, t0)) { bad = 1; break; }
            __builtin_amdgcn_s_sleep(1);
        }
        gm = wave_max_u32(gm);
        if (lane == 0) { lds_st(lds, L_GMAX, gm); lds_st(lds, L_BAD, bad); }
        LGKM0();
        if (lane == 0) lds_st(lds, L_GOK, 1u);
        if (gi == 0 && lane == 0) __hip_atomic_store(ctr, call, __ATOMIC_RELAXED, __HIP_MEMORY_SCOPE_AGENT);
    }
    lds_wait_ge(lds, L_GOK, 1u);
    const float amax = __uint_as_float(rfl(lds_ld(lds, L_GMAX)));
    const bool poison = rfl(lds_ld(lds, L_BAD)) != 0u;
    {
        const float scale = amax / 127.0f;
        const float inv = scale > 0.f ? 1.0f / scale : 0.f;
        const f32x2 inv2 = {inv, inv}, magic2 = {12582912.f + 128.f, 12582912.f + 128.f};
#pragma unroll
        for (int i = 0; i < 5; ++i) {
#pragma unroll
            for (int f = 0; f < 2; ++f) {
                const u32 ch = 4 * wn + 2 * f + (lane >> 5);
                u32x4 v;
#pragma unroll
                for (int q = 0; q < 4; ++q) {
                    constexpr float MAGIC = 12582912.f + 128.f;
                    const f32x2 lo2 = {acc[f][i][2 * q], acc[f][i][2 * q + 1]}, hi2 = {acc[f][i][2 * q + 8], acc[f][i][2 * q + 9]};
                    const f32x2 tl = __builtin_elementwise_fma(lo2, inv2, magic2), th = __builtin_elementwise_fma(hi2, inv2, magic2);
                    const u32 b0 = __float_as_uint(__builtin_amdgcn_fmed3f(tl.x, MAGIC - 127.f, MAGIC + 127.f));
                    const u32 b1 = __float_as_uint(__builtin_amdgcn_fmed3f(th.x, MAGIC - 127.f, MAGIC + 127.f));
                    const u32 b2 = __float_as_uint(__builtin_amdgcn_fmed3f(tl.y, MAGIC - 127.f, MAGIC + 127.f));
                    const u32 b3 = __float_as_uint(__builtin_amdgcn_fmed3f(th.y, MAGIC - 127.f, MAGIC + 127.f));
                    const u32 d = __builtin_amdgcn_perm(__builtin_amdgcn_perm(b3, b2, 0x0c0c0400u), __builtin_amdgcn_perm(b1, b0, 0x0c0c0400u), 0x05040100u) ^ 0x80808080u;
                    v[q] = poison ? 0x80808080u : d;
                }
                const u32 row = 160 * wm + 32 * i + (lane & 31);
                *(LAS u32x4*)(lds + row * 128 + ((ch ^ (row & 7)) * 16)) = v;
            }
            LGKM0();
            if (lane == 0) lds_add(lds, L_OUT + i * 4, 1u);
            __builtin_amdgcn_sched_barrier(0);
        }
    }
}

extern "C" void kernel_launch(int rank, int nranks, ncclComm_t comm, hipStream_t stream,
                              void* const* d_in, const int* in_sizes, int n_in,
                              void* d_out, int out_size, const int* in_dtypes, int out_dtype,
                              void* const* d_out_all, void* const* d_ws_all, size_t ws_bytes) {
    (void)comm; (void)in_dtypes; (void)out_dtype; (void)d_out;
    if (nranks != NR || (unsigned)rank >= (unsigned)NR || n_in != 2 || in_sizes[0] != MM * KSH || in_sizes[1] != KSH * NN ||
        out_size != (int)OUT_BYTES || ws_bytes < WS_USED) {
        fprintf(stderr, "kernel_launch: unexpected shapes\n");
        return;
    }
    Params P;
    P.x = d_in[0];
    P.w = d_in[1];
    for (int p = 0; p < NR; ++p) { P.ws[p] = (char*)d_ws_all[p]; P.out[p] = (char*)d_out_all[p]; }
    P.rank = rank;
    P.pad = 0;
    hipLaunchKernelGGL(gemm_ag_q8, dim3(256), dim3(512), 0, stream, P);
}
```
